```python
import numpy as np

kernel_name = "gpu_ag_gemm_m4096_k4096_n16384_e5m2e5m2_dqsilu"


ABSMAX_RTOL = 0.05

COMPUTE_DTYPE = "float32"
EPILOGUE = "dequant_silu"
IN_QUANT = None
OUT_QUANT = None
IN_FP8 = "e5m2e5m2"
IN_DTYPES = {"x": "e5m2", "w_mat": "e5m2",
             "scale_x": "float32", "scale_w": "float32"}
WORLD_SIZE = 8

def _snap_fp8(x, mant_bits, max_abs, min_normal):
    a = np.abs(x.astype(np.float64))
    _m, _e = np.frexp(a)
    q = np.where(a >= min_normal, np.ldexp(1.0, _e - 1 - mant_bits),
                 min_normal * 2.0 ** -mant_bits)
    snapped = np.minimum(np.round(a / q) * q, max_abs)
    return (np.sign(x) * snapped).astype(np.float32)

def setup_inputs():

    x = _snap_fp8(np.random.randn(4096, 4096).astype(np.float32), 2, 57344.0, 2.0 ** -14)
    w_mat = _snap_fp8(np.random.randn(4096, 16384).astype(np.float32), 2, 57344.0, 2.0 ** -14)
    scale_x = (0.02 * (1.0 + np.random.rand(1))).astype(np.float32)
    scale_w = (0.02 * (1.0 + np.random.rand(1))).astype(np.float32)
    return {"x": x, "w_mat": w_mat, "scale_x": scale_x, "scale_w": scale_w}

def reference(x, w_mat, scale_x, scale_w):
    acc = (x @ w_mat).astype(np.float32)
    y = (acc * (scale_x * scale_w)).astype(np.float32)
    return (y / (1.0 + np.exp(-np.clip(y, -60.0, 60.0)))).astype(np.float32)
```

```cpp
#include <hip/hip_runtime.h>
#include <rccl/rccl.h>
#include <cstdio>
#include <cstdint>
#include <utility>

namespace {

constexpr int NR = 8, M_ = 4096, K_ = 4096, NL = 2048, MS = M_ / NR;
constexpr int BM = 128, BN = 256, BK = 128, NK = K_ / BK;
constexpr unsigned A_BYTES = BM * BK, B_BYTES = BK * BN, STAGE = A_BYTES + B_BYTES, NSTAGE = 3;
constexpr unsigned SYNC_OFF = NSTAGE * STAGE;
constexpr unsigned LDS_BYTES = SYNC_OFF + 64;
constexpr int NWG = 256;
constexpr size_t WS_CNT = 0;
constexpr size_t WS_TMO = 64 << 10;
constexpr size_t WS_FLAG = 128 << 10;
constexpr size_t WS_DATA = 1 << 20;
constexpr size_t SHARD = (size_t)MS * K_;
constexpr size_t WS_STAMP = WS_DATA + 2 * NR * SHARD;
constexpr size_t WS_NEED = WS_STAMP + (size_t)NWG * 256 * 8;
constexpr unsigned long long DEADLINE_TICKS = 30000000ull;

typedef unsigned u32x2 __attribute__((ext_vector_type(2)));
typedef unsigned u32x4 __attribute__((ext_vector_type(4)));
typedef float f32x4 __attribute__((ext_vector_type(4)));
typedef float f32x16 __attribute__((ext_vector_type(16)));
#define LAS __attribute__((address_space(3)))
#define GAS __attribute__((address_space(1)))

struct Params {
    const unsigned char* x; const unsigned char* w; const float* sx; const float* sw; float* out;
    unsigned char* ws[NR];
    unsigned rank, pad;
};

__device__ __forceinline__ unsigned rfl(unsigned v) { return __builtin_amdgcn_readfirstlane(v); }
__device__ __forceinline__ void lds_st(unsigned addr, unsigned v) { asm volatile("ds_write_b32 %0, %1" :: "v"(addr), "v"(v) : "memory"); }
__device__ __forceinline__ u32x4 lds_ld4(unsigned addr) { u32x4 r; asm volatile("ds_read_b128 %0, %1\n\ts_waitcnt lgkmcnt(0)" : "=&v"(r) : "v"(addr) : "memory"); return r; }
__device__ __forceinline__ unsigned min4(u32x4 v) { return rfl(min(min(v.x, v.y), min(v.z, v.w))); }

#define DMA_PIECE(V, SC) "buffer_load_dwordx4 " V ", %[rs], %[so] offen" SC " lds\n\ts_add_u32 m0, m0, 0x400\n\ts_add_u32 %[so], %[so], %[st]\n\t"
#define DMA_PAIR(SC) DMA_PIECE("%[ve]", SC) DMA_PIECE("%[vo]", SC)
#define DMA16(SC) "s_mov_b32 %[kp], m0\n\ts_mov_b32 m0, %[ld]\n\ts_nop 0\n\t" DMA_PAIR(SC) DMA_PAIR(SC) DMA_PAIR(SC) DMA_PAIR(SC) DMA_PAIR(SC) DMA_PAIR(SC) DMA_PAIR(SC) DMA_PAIR(SC) "s_mov_b32 m0, %[kp]"
template <bool SYS> __device__ __forceinline__ void dma16(__amdgpu_buffer_rsrc_t rs, unsigned ve, unsigned vo, unsigned soff, unsigned step, unsigned lds) {
    unsigned keep;
    if (SYS) asm volatile(DMA16(" sc0 sc1") : [kp] "=&s"(keep), [so] "+s"(soff) : [ve] "v"(ve), [vo] "v"(vo), [rs] "s"(rs), [st] "s"(step), [ld] "s"(lds) : "memory", "scc");
    else     asm volatile(DMA16("")         : [kp] "=&s"(keep), [so] "+s"(soff) : [ve] "v"(ve), [vo] "v"(vo), [rs] "s"(rs), [st] "s"(step), [ld] "s"(lds) : "memory", "scc");
}
__device__ __forceinline__ void wait_free(unsigned sync, unsigned need) {
    for (unsigned sp = 0; sp < (1u << 24); ++sp) {
        if (min4(lds_ld4(sync + 16)) >= need) break;
        __builtin_amdgcn_s_sleep(1);
    }
}

#define STAMP(var, idx) do {} while (0)

#define NSEND 256
#define WIN 1
constexpr int NJ = 7 * 512 / NSEND;
static_assert(NJ * NSEND == 7 * 512, "jobs must divide");
struct Sched { int n; int op[NJ * 12 + 16], a[NJ * 12 + 16], b[NJ * 12 + 16]; };
constexpr Sched make_sched() {
    Sched s{}; int n = 0, issued = 4;
    int posL[NJ + 1] = {}, posS[4 * NJ] = {};
    posL[0] = 4; int done = 0, flagged = 0;
    for (int ns = 0; ns < 4 * NJ + WIN; ++ns) {
        const int j = ns / 4, pc = ns % 4;
        if (ns < 4 * NJ && pc == 0 && j + 1 < NJ) { s.op[n] = 0; s.a[n] = j + 1; ++n; issued += 4; posL[j + 1] = issued; }
        int need = 0;
        if (ns < 4 * NJ) need = posL[j];
        if (ns >= WIN && posS[ns - WIN] > need) need = posS[ns - WIN];
        if (need > done) { s.op[n] = 3; s.a[n] = issued - need; ++n; done = need; }
        if (ns < 4 * NJ) { s.op[n] = 1; s.a[n] = j; s.b[n] = pc; ++n; ++issued; posS[ns] = issued; }
        while (flagged < NJ && posS[4 * flagged + 3] != 0 && posS[4 * flagged + 3] <= done) { s.op[n] = 2; s.a[n] = flagged; ++n; ++issued; ++flagged; }
    }
    s.n = n; return s;
}
constexpr Sched SCHED = make_sched();
struct Sender {
    __amdgpu_buffer_rsrc_t rs; unsigned v0, v1, v2, v3, vst, c, rank, call, lane;
    u32x4 d[2][4];
    unsigned long long st0 = 0;
    template <int J> __device__ __forceinline__ void load() {
        const unsigned t = (c + NSEND * J) / 7u, ks = t >> 4, jrb = (t >> 2) & 3, jq = t & 3;
        const unsigned so = (jrb * BM + jq * 32) * K_ + ks * BK;
        u32x4 (&b)[4] = d[J & 1];
        asm volatile("buffer_load_dwordx4 %0, %4, %8, %9 offen\n\tbuffer_load_dwordx4 %1, %5, %8, %9 offen\n\t"
                     "buffer_load_dwordx4 %2, %6, %8, %9 offen\n\tbuffer_load_dwordx4 %3, %7, %8, %9 offen"
                     : "=&v"(b[0]), "=&v"(b[1]), "=&v"(b[2]), "=&v"(b[3]) : "v"(v0), "v"(v1), "v"(v2), "v"(v3), "s"(rs), "s"(so) : "memory");
    }
    template <int J, int PC> __device__ __forceinline__ void store(const Params& p) {
        const unsigned i = c + NSEND * J, t = i / 7u, peer = (rank + 1 + (i - 7 * t)) & 7, ks = t >> 4, jrb = (t >> 2) & 3, jq = t & 3;
        unsigned char* dst = p.ws[peer] + WS_DATA + ((size_t)(call & 1) * NR + rank) * SHARD + (jrb * NK + ks) * A_BYTES + jq * 4096;
        asm volatile("global_store_dwordx4 %1, %0, %2 offset:%3 sc0 sc1\n\ts_nop 1" :: "v"(d[J & 1][PC]), "v"(vst), "s"(dst), "n"(PC * 1024) : "memory");
    }
    template <int J> __device__ __forceinline__ void flag(const Params& p) {
        const unsigned i = c + NSEND * J, t = i / 7u, peer = (rank + 1 + (i - 7 * t)) & 7, ks = t >> 4, jrb = (t >> 2) & 3, jq = t & 3;
        unsigned char* fp = p.ws[peer] + WS_FLAG + ((rank * 4 + jrb) * 128 + ks * 4 + jq) * 4;
        asm volatile("s_mov_b64 exec, 1\n\tglobal_store_dword %0, %1, %2 sc0 sc1\n\ts_mov_b64 exec, -1" :: "v"(0u), "v"(call), "s"(fp) : "memory");
        if constexpr (J % (NJ / 14) == NJ / 14 - 1) STAMP(st0, J / (NJ / 14));
    }
    template <int I> __device__ __forceinline__ void step(const Params& p) {
        constexpr int op = SCHED.op[I], a = SCHED.a[I], b = SCHED.b[I];
        if constexpr (op == 0) load<a>();
        else if constexpr (op == 1) store<a, b>(p);
        else if constexpr (op == 2) flag<a>(p);
        else asm volatile("s_waitcnt vmcnt(%0)" :: "n"(a) : "memory");
    }
    template <int LO, int HI> __device__ __forceinline__ void run(const Params& p) {
        if constexpr (HI - LO == 1) step<LO>(p);
        else { run<LO, (LO + HI) / 2>(p); run<(LO + HI) / 2, HI>(p); }
    }
};

#define MF(ACC, W, X) "v_mfma_f32_32x32x64_f8f6f4 " ACC ", " W ", " X ", " ACC " cbsz:1 blgp:1\n\t"
#define RX(R, A, OFF) "ds_read_b128 " R ", " A " offset:" OFF "\n\t"
#define RW(R, A, OFF) "ds_read_b64_tr_b8 " R ", " A " offset:" OFF "\n\t"
#define R0_W0a RW("v[64:65]", "%[b0]", "16384") RW("v[66:67]", "%[b0]", "18432") RW("v[68:69]", "%[b0]", "20480")
#define R0_W0b RW("v[70:71]", "%[b0]", "22528") RX("v[32:35]", "%[a0]", "0") RX("v[36:39]", "%[a1]", "0")
#define R0_W1a RW("v[72:73]", "%[b1]", "16384") RW("v[74:75]", "%[b1]", "18432") RW("v[76:77]", "%[b1]", "20480")
#define R0_W1b RW("v[78:79]", "%[b1]", "22528") RX("v[40:43]", "%[a0]", "4096") RX("v[44:47]", "%[a1]", "4096")
#define R0_X2  RX("v[48:51]", "%[a0]", "8192") RX("v[52:55]", "%[a1]", "8192")
#define R0_X3  RX("v[56:59]", "%[a0]", "12288") RX("v[60:63]", "%[a1]", "12288")
#define R1_W0a RW("v[112:113]", "%[b0]", "32768") RW("v[114:115]", "%[b0]", "34816") RW("v[116:117]", "%[b0]", "36864")
#define R1_W0b RW("v[118:119]", "%[b0]", "38912") RX("v[80:83]", "%[a2]", "0") RX("v[84:87]", "%[a3]", "0")
#define R1_W1a RW("v[120:121]", "%[b1]", "32768") RW("v[122:123]", "%[b1]", "34816") RW("v[124:125]", "%[b1]", "36864")
#define R1_W1b RW("v[126:127]", "%[b1]", "38912") RX("v[88:91]", "%[a2]", "4096") RX("v[92:95]", "%[a3]", "4096")
#define R1_X2  RX("v[96:99]", "%[a2]", "8192") RX("v[100:103]", "%[a3]", "8192")
#define R1_X3  RX("v[104:107]", "%[a2]", "12288") RX("v[108:111]", "%[a3]", "12288")
#define CLOB8(a,b,c,d,e,f,g,h) "v" #a, "v" #b, "v" #c, "v" #d, "v" #e, "v" #f, "v" #g, "v" #h
#define FRAG_CLOBBERS CLOB8(28,29,30,31,32,33,34,35), CLOB8(36,37,38,39,40,41,42,43), CLOB8(44,45,46,47,48,49,50,51), CLOB8(52,53,54,55,56,57,58,59), \
    CLOB8(60,61,62,63,64,65,66,67), CLOB8(68,69,70,71,72,73,74,75), CLOB8(76,77,78,79,80,81,82,83), CLOB8(84,85,86,87,88,89,90,91), \
    CLOB8(92,93,94,95,96,97,98,99), CLOB8(100,101,102,103,104,105,106,107), CLOB8(108,109,110,111,112,113,114,115), CLOB8(116,117,118,119,120,121,122,123), \
    "v124", "v125", "v126", "v127"

__global__ void __launch_bounds__(512, 2) ag_gemm_kernel(Params p) {
    __shared__ __attribute__((aligned(1024))) unsigned char smem[LDS_BYTES];
    const unsigned lds0 = (unsigned)(size_t)(LAS unsigned char*)smem;
    const unsigned sync = lds0 + SYNC_OFF;
    const unsigned lane = threadIdx.x & 63, wid = rfl(threadIdx.x >> 6), bid = blockIdx.x;
    const unsigned rank = p.rank & 7;
    const unsigned src = bid & 7, rb = (bid >> 3) & 3, cb = bid >> 5;
    unsigned char* const myws = p.ws[rank];
    if (threadIdx.x < 8) lds_st(sync + 4 * threadIdx.x, 0u);
    asm volatile("s_waitcnt lgkmcnt(0)\n\ts_barrier" ::: "memory");

    if (wid < 4) {
        const float scale = p.sx[0] * p.sw[0];
        const unsigned r = lane & 31, h = lane >> 5, key = (r >> 1) & 7;
        unsigned a0 = lds0 + r * 128 + (((2 * h + 0) ^ key) << 4), a1 = lds0 + r * 128 + (((2 * h + 1) ^ key) << 4);
        unsigned a2 = lds0 + r * 128 + (((4 + 2 * h + 0) ^ key) << 4), a3 = lds0 + r * 128 + (((4 + 2 * h + 1) ^ key) << 4);
        const unsigned ch = (lane >> 4) & 1, q = (lane & 15) >> 1, pp = lane & 1;
        unsigned b0 = lds0 + (32 * h + q) * 256 + (((4 * wid + 0 + ch) ^ (2 * q)) << 4) + 8 * pp;
        unsigned b1 = lds0 + (32 * h + q) * 256 + (((4 * wid + 2 + ch) ^ (2 * q)) << 4) + 8 * pp;
        f32x16 acc[8];
#pragma unroll
        for (int i = 0; i < 8; ++i)
#pragma unroll
            for (int j = 0; j < 16; ++j) acc[i][j] = 0.f;
        unsigned kk = 0, slot = 0, dl, tmp, spin = 0;
        const unsigned vsync = sync, vcons = sync + 16 + 4 * wid;
        asm volatile(
            "s_nop 1\n\t"
            "s_branch .Lspin%=\n"
            ".Lloop%=:\n\t"
            "s_waitcnt lgkmcnt(0)\n\t"
            MF("%0", "v[64:71]", "v[32:39]") R1_W0a
            MF("%1", "v[72:79]", "v[32:39]") R1_W0b
            MF("%2", "v[64:71]", "v[40:47]") R1_W1a
            MF("%3", "v[72:79]", "v[40:47]") R1_W1b
            MF("%4", "v[64:71]", "v[48:55]") R1_X2
            "s_add_u32 %[kk], %[kk], 1\n\t"
            "v_mov_b32 v28, %[kk]\n\t"
            MF("%5", "v[72:79]", "v[48:55]") R1_X3
            "ds_write_b32 %[vc], v28\n\t"
            "s_add_u32 %[sl], %[sl], 1\n\t"
            "s_cmp_eq_u32 %[sl], 3\n\t"
            "s_cselect_b32 %[dl], %[dn], %[dp]\n\t"
            "s_cselect_b32 %[sl], 0, %[sl]\n\t"
            MF("%6", "v[64:71]", "v[56:63]")
            "ds_read_b128 v[28:31], %[vs]\n\t"
            "v_add_u32 %[a0], %[dl], %[a0]\n\tv_add_u32 %[a1], %[dl], %[a1]\n\tv_add_u32 %[a2], %[dl], %[a2]\n\t"
            "v_add_u32 %[a3], %[dl], %[a3]\n\tv_add_u32 %[b0], %[dl], %[b0]\n\tv_add_u32 %[b1], %[dl], %[b1]\n\t"
            MF("%7", "v[72:79]", "v[56:63]")
            "s_waitcnt lgkmcnt(0)\n\t"
            MF("%0", "v[112:119]", "v[80:87]")
            "v_min3_u32 v28, v28, v29, v30\n\t"
            "s_nop 0\n\t"
            "v_readfirstlane_b32 %[tm], v28\n\t"
            "s_nop 0\n\t"
            "s_cmp_gt_u32 %[tm], %[kk]\n\t"
            "s_cbranch_scc0 .Lslow%=\n\t"
            R0_W0a
            MF("%1", "v[120:127]", "v[80:87]") R0_W0b
            MF("%2", "v[112:119]", "v[88:95]") R0_W1a
            MF("%3", "v[120:127]", "v[88:95]") R0_W1b
            MF("%4", "v[112:119]", "v[96:103]") R0_X2
            MF("%5", "v[120:127]", "v[96:103]") R0_X3
            MF("%6", "v[112:119]", "v[104:111]")
            MF("%7", "v[120:127]", "v[104:111]")
            "s_branch .Lloop%=\n"
            ".Lslow%=:\n\t"
            MF("%1", "v[120:127]", "v[80:87]")
            MF("%2", "v[112:119]", "v[88:95]")
            MF("%3", "v[120:127]", "v[88:95]")
            MF("%4", "v[112:119]", "v[96:103]")
            MF("%5", "v[120:127]", "v[96:103]")
            MF("%6", "v[112:119]", "v[104:111]")
            MF("%7", "v[120:127]", "v[104:111]")
            "s_cmp_ge_u32 %[kk], 32\n\t"
            "s_cbranch_scc1 .Ldone%=\n"
            ".Lspin%=:\n\t"
            "ds_read_b128 v[28:31], %[vs]\n\t"
            "s_add_u32 %[sp], %[sp], 1\n\t"
            "s_waitcnt lgkmcnt(0)\n\t"
            "v_min3_u32 v28, v28, v29, v30\n\t"
            "s_nop 0\n\t"
            "v_readfirstlane_b32 %[tm], v28\n\t"
            "s_cmp_gt_u32 %[sp], 0x2000000\n\t"
            "s_cbranch_scc1 .Ldone%=\n\t"
            "s_cmp_gt_u32 %[tm], %[kk]\n\t"
            "s_cbranch_scc0 .Lspin%=\n\t"
            R0_W0a R0_W0b R0_W1a R0_W1b R0_X2 R0_X3
            "s_branch .Lloop%=\n"
            ".Ldone%=:\n\t"
            "s_nop 15\n\ts_nop 7"
            : "+a"(acc[0]), "+a"(acc[1]), "+a"(acc[2]), "+a"(acc[3]), "+a"(acc[4]), "+a"(acc[5]), "+a"(acc[6]), "+a"(acc[7]),
              [a0] "+v"(a0), [a1] "+v"(a1), [a2] "+v"(a2), [a3] "+v"(a3), [b0] "+v"(b0), [b1] "+v"(b1),
              [kk] "+s"(kk), [sl] "+s"(slot), [dl] "=&s"(dl), [tm] "=&s"(tmp), [sp] "+s"(spin)
            : [vs] "v"(vsync), [vc] "v"(vcons), [dp] "s"(STAGE), [dn] "s"(0u - (NSTAGE - 1) * STAGE)
            : "memory", "scc", FRAG_CLOBBERS);
        STAMP(st0, 0);
        const u32x4 fl = lds_ld4(sync);
        const bool bad = rfl(fl.w) != 0u || kk != (unsigned)NK;
        float* orow = p.out + (size_t)(src * MS + rb * BM + r) * NL + cb * BN + wid * 64 + 4 * h;
        const float nl2e = -1.44269504088896340736f;
#pragma unroll
        for (int mi = 0; mi < 4; ++mi)
#pragma unroll
            for (int ni = 0; ni < 2; ++ni)
#pragma unroll
                for (int g = 0; g < 4; ++g) {
                    f32x4 o;
#pragma unroll
                    for (int j = 0; j < 4; ++j) {
                        const float y = acc[2 * mi + ni][4 * g + j] * scale;
                        const float c = __builtin_amdgcn_fmed3f(y, -60.f, 60.f);
                        const float e = __builtin_amdgcn_exp2f(c * nl2e);
                        o[j] = y * __builtin_amdgcn_rcpf(1.f + e);
                    }
                    if (bad) o = (f32x4){__builtin_nanf(""), __builtin_nanf(""), __builtin_nanf(""), __builtin_nanf("")};
                    *(f32x4*)(orow + (size_t)mi * 32 * NL + ni * 32 + g * 8) = o;
                }
    } else if (wid == 4) {
        GAS unsigned* cntp = (GAS unsigned*)(myws + WS_CNT + bid * 64);
        const unsigned call = rfl(__hip_atomic_load(cntp, __ATOMIC_RELAXED, __HIP_MEMORY_SCOPE_AGENT)) + 1u;
        if (lane == 0) __hip_atomic_store(cntp, call, __ATOMIC_RELAXED, __HIP_MEMORY_SCOPE_AGENT);
        GAS unsigned* tmo = (GAS unsigned*)(myws + WS_TMO);
        const bool local = src == rank;
        const unsigned char* base = local ? p.x : myws + WS_DATA + ((size_t)(call & 1) * NR + src) * SHARD;
        const __amdgpu_buffer_rsrc_t rs = __builtin_amdgcn_make_buffer_rsrc((void*)base, 0, (unsigned)SHARD, 0x00020000);
        const unsigned ve = local ? (lane >> 3) * K_ + (((lane & 7) ^ (lane >> 4)) << 4) : lane * 16;
        const unsigned vo = local ? (lane >> 3) * K_ + (((lane & 7) ^ (4 + (lane >> 4))) << 4) : lane * 16;
        const unsigned step = local ? 8 * K_ : 1024, kstep = local ? BK : A_BYTES;
        const unsigned s0 = rb * BM * K_;
        const unsigned char* flagp = myws + WS_FLAG + (src * 4 + rb) * 512;
        unsigned ready = local ? NK : 0, slot = 0;
        bool dead = false;
        asm volatile("s_waitcnt vmcnt(0)" ::: "memory");
        for (unsigned k = 0; k < NK; ++k) {
            if (k >= ready) {
                const unsigned long long t0 = __builtin_amdgcn_s_memrealtime();
                for (unsigned sp = 0;; ++sp) {
                    u32x2 f;
                    asm volatile("global_load_dwordx2 %0, %1, %2 sc0 sc1\n\ts_waitcnt vmcnt(0)" : "=&v"(f) : "v"(lane * 8), "s"(flagp) : "memory");
                    if (sp == 0) lds_st(sync, k);
                    const unsigned long long mk = __ballot(f.x >= call && f.y >= call);
                    ready = (~mk == 0ull ? 64u : (unsigned)__builtin_ctzll(~mk)) >> 1;
                    if (ready > k) break;
                    if ((sp & 15) == 15) {
                        if (rfl(__hip_atomic_load(tmo, __ATOMIC_RELAXED, __HIP_MEMORY_SCOPE_AGENT)) == call || __builtin_amdgcn_s_memrealtime() - t0 > DEADLINE_TICKS) {
                            if (lane == 0) __hip_atomic_store(tmo, call, __ATOMIC_RELAXED, __HIP_MEMORY_SCOPE_AGENT);
                            dead = true; break;
                        }
                    }
                    __builtin_amdgcn_s_sleep(1);
                }
                if (dead) break;
            }
            STAMP(st0, k);
            if (k >= NSTAGE) wait_free(sync, k - NSTAGE + 1);
            dma16<true>(rs, ve, vo, s0 + k * kstep, step, lds0 + slot * STAGE);
            asm volatile("s_waitcnt vmcnt(16)" ::: "memory");
            lds_st(sync, k);
            slot = slot == NSTAGE - 1 ? 0 : slot + 1;
        }
        if (dead || rfl(__hip_atomic_load(tmo, __ATOMIC_RELAXED, __HIP_MEMORY_SCOPE_AGENT)) == call) lds_st(sync + 12, 1u);
        asm volatile("s_waitcnt vmcnt(0)" ::: "memory");
        lds_st(sync, NK);
    } else if (wid < 7) {
        const unsigned hb = wid - 5;
        const __amdgpu_buffer_rsrc_t rs = __builtin_amdgcn_make_buffer_rsrc((void*)p.w, 0, (unsigned)(K_ * NL), 0x00020000);
        const unsigned ve = (lane >> 4) * NL + (((lane & 15) ^ (2 * (lane >> 4))) << 4);
        const unsigned vo = (lane >> 4) * NL + (((lane & 15) ^ (2 * (4 + (lane >> 4)))) << 4);
        unsigned slot = 0;
        for (unsigned k = 0; k < NK; ++k) {
            if (k >= NSTAGE) wait_free(sync, k - NSTAGE + 1);
            dma16<false>(rs, ve, vo, (k * BK + hb * 64) * NL + cb * BN, 4 * NL, lds0 + slot * STAGE + A_BYTES + hb * 16384);
            asm volatile("s_waitcnt vmcnt(16)" ::: "memory");
            lds_st(sync + 4 + 4 * hb, k);
            slot = slot == NSTAGE - 1 ? 0 : slot + 1;
        }
        asm volatile("s_waitcnt vmcnt(0)" ::: "memory");
        lds_st(sync + 4 + 4 * hb, NK);
    } else {
        if (bid < NSEND) {
            Sender s;
            GAS unsigned* cntp = (GAS unsigned*)(myws + WS_CNT + NWG * 64 + bid * 64);
            unsigned cv;
            asm volatile("global_load_dword %0, %1, %2 sc1" : "=&v"(cv) : "v"(0u), "s"(cntp) : "memory");
            s.rs = __builtin_amdgcn_make_buffer_rsrc((void*)p.x, 0, (unsigned)SHARD, 0x00020000);
            s.v0 = (lane >> 3) * K_ + (((lane & 7) ^ (lane >> 4)) << 4);
            s.v1 = (lane >> 3) * K_ + (((lane & 7) ^ (4 + (lane >> 4))) << 4) + 8 * K_;
            s.v2 = s.v0 + 16 * K_; s.v3 = s.v1 + 16 * K_;
            s.vst = lane * 16; s.c = bid; s.rank = rank; s.lane = lane;
            s.load<0>();
            asm volatile("s_waitcnt vmcnt(4)" : "+v"(cv) :: "memory");
            s.call = rfl(cv) + 1u;
            s.run<0, SCHED.n>(p);
            if (lane == 0) __hip_atomic_store(cntp, s.call, __ATOMIC_RELAXED, __HIP_MEMORY_SCOPE_AGENT);
        }
    }
}

}

extern "C" void kernel_launch(int rank, int nranks, ncclComm_t comm, hipStream_t stream,
                              void* const* d_in, const int* in_sizes, int n_in,
                              void* d_out, int out_size, const int* in_dtypes, int out_dtype,
                              void* const* d_out_all, void* const* d_ws_all, size_t ws_bytes) {
    (void)comm; (void)in_dtypes; (void)out_dtype; (void)d_out_all;
    if (nranks != NR || n_in != 4 || rank < 0 || rank >= NR || in_sizes[0] != MS * K_ || in_sizes[1] != K_ * NL || out_size != M_ * NL || ws_bytes < WS_NEED) {
        fprintf(stderr, "ag_gemm: unexpected shapes\n");
        return;
    }
    Params p;
    p.x = (const unsigned char*)d_in[0]; p.w = (const unsigned char*)d_in[1];
    p.sx = (const float*)d_in[2]; p.sw = (const float*)d_in[3]; p.out = (float*)d_out;
    for (int i = 0; i < NR; ++i) p.ws[i] = (unsigned char*)d_ws_all[i];
    p.rank = (unsigned)rank; p.pad = 0;
    hipLaunchKernelGGL(ag_gemm_kernel, dim3(NWG), dim3(512), 0, stream, p);
    hipError_t e = hipGetLastError();
    if (e != hipSuccess) fprintf(stderr, "ag_gemm launch: %s\n", hipGetErrorString(e));
}
```

```cpp
#include <hip/hip_runtime.h>
#include <rccl/rccl.h>
#include <cstdio>
#include <cstdint>
#include <utility>

namespace {

constexpr int NR = 8, M_ = 4096, K_ = 4096, NL = 2048, MS = M_ / NR;
constexpr int BM = 128, BN = 256, BK = 128, NK = K_ / BK;
constexpr unsigned A_BYTES = BM * BK, B_BYTES = BK * BN, STAGE = A_BYTES + B_BYTES, NSTAGE = 3;
constexpr unsigned ENC_OFF = NSTAGE * STAGE;
constexpr unsigned LIST_OFF = ENC_OFF + 6144;
constexpr unsigned SYNC_OFF = LIST_OFF + 512;
constexpr unsigned LDS_BYTES = SYNC_OFF + 64;
constexpr int NWG = 256;
constexpr unsigned EXC_CAP = 128;
constexpr size_t SHARD = (size_t)MS * K_;
constexpr size_t WS_CNT = 0;
constexpr size_t WS_TMO = 64 << 10;
constexpr size_t WS_FLAG = 128 << 10;
constexpr size_t WS_DATA = 1 << 20;
constexpr unsigned QU = 3072 + 256;
constexpr unsigned PACK_PAR = 512 * QU;
constexpr unsigned RAW_OFF = 4 << 20;
constexpr unsigned DATA_BYTES = RAW_OFF + 2 * (unsigned)SHARD;
static_assert(2 * PACK_PAR <= RAW_OFF, "workspace map");
constexpr size_t WS_STAMP = WS_DATA + DATA_BYTES;
constexpr size_t WS_NEED = WS_STAMP + (size_t)NWG * 256 * 8;
constexpr unsigned long long DEADLINE_TICKS = 30000000ull;

typedef unsigned u32x2 __attribute__((ext_vector_type(2)));
typedef unsigned u32x4 __attribute__((ext_vector_type(4)));
typedef float f32x4 __attribute__((ext_vector_type(4)));
typedef float f32x16 __attribute__((ext_vector_type(16)));
#define LAS __attribute__((address_space(3)))
#define GAS __attribute__((address_space(1)))

struct Params {
    const unsigned char* x; const unsigned char* w; const float* sx; const float* sw; float* out;
    unsigned char* ws[NR];
    unsigned rank, pad;
};

__device__ __forceinline__ unsigned rfl(unsigned v) { return __builtin_amdgcn_readfirstlane(v); }
__device__ __forceinline__ void lds_st(unsigned addr, unsigned v) { asm volatile("ds_write_b32 %0, %1" :: "v"(addr), "v"(v) : "memory"); }
__device__ __forceinline__ u32x4 lds_ld4(unsigned addr) { u32x4 r; asm volatile("ds_read_b128 %0, %1\n\ts_waitcnt lgkmcnt(0)" : "=&v"(r) : "v"(addr) : "memory"); return r; }
__device__ __forceinline__ unsigned min4(u32x4 v) { return rfl(min(min(v.x, v.y), min(v.z, v.w))); }

#define DMA_PIECE(V, SC) "buffer_load_dwordx4 " V ", %[rs], %[so] offen" SC " lds\n\ts_add_u32 m0, m0, 0x400\n\ts_add_u32 %[so], %[so], %[st]\n\t"
#define DMA_PAIR(SC) DMA_PIECE("%[ve]", SC) DMA_PIECE("%[vo]", SC)
#define DMA16(SC) "s_mov_b32 %[kp], m0\n\ts_mov_b32 m0, %[ld]\n\ts_nop 0\n\t" DMA_PAIR(SC) DMA_PAIR(SC) DMA_PAIR(SC) DMA_PAIR(SC) DMA_PAIR(SC) DMA_PAIR(SC) DMA_PAIR(SC) DMA_PAIR(SC) "s_mov_b32 m0, %[kp]"
__device__ __forceinline__ void dma16(__amdgpu_buffer_rsrc_t rs, unsigned ve, unsigned vo, unsigned soff, unsigned step, unsigned lds) {
    unsigned keep;
    asm volatile(DMA16("") : [kp] "=&s"(keep), [so] "+s"(soff) : [ve] "v"(ve), [vo] "v"(vo), [rs] "s"(rs), [st] "s"(step), [ld] "s"(lds) : "memory", "scc");
}
#define DMA8 "s_mov_b32 %[kp], m0\n\ts_mov_b32 m0, %[ld]\n\ts_nop 0\n\t" DMA_PAIR("") DMA_PAIR("") DMA_PAIR("") DMA_PAIR("") "s_mov_b32 m0, %[kp]"
__device__ __forceinline__ void dma8(__amdgpu_buffer_rsrc_t rs, unsigned ve, unsigned vo, unsigned soff, unsigned step, unsigned lds) {
    unsigned keep;
    asm volatile(DMA8 : [kp] "=&s"(keep), [so] "+s"(soff) : [ve] "v"(ve), [vo] "v"(vo), [rs] "s"(rs), [st] "s"(step), [ld] "s"(lds) : "memory", "scc");
}
#define DMA4 "s_mov_b32 %[kp], m0\n\ts_mov_b32 m0, %[ld]\n\ts_nop 0\n\t" DMA_PAIR("") DMA_PAIR("") "s_mov_b32 m0, %[kp]"
__device__ __forceinline__ void dma4(__amdgpu_buffer_rsrc_t rs, unsigned ve, unsigned vo, unsigned soff, unsigned step, unsigned lds) {
    unsigned keep;
    asm volatile(DMA4 : [kp] "=&s"(keep), [so] "+s"(soff) : [ve] "v"(ve), [vo] "v"(vo), [rs] "s"(rs), [st] "s"(step), [ld] "s"(lds) : "memory", "scc");
}
__device__ __forceinline__ unsigned wave_sum(unsigned v) {
    v += __builtin_amdgcn_update_dpp(0u, v, 0x111, 0xf, 0xf, true);
    v += __builtin_amdgcn_update_dpp(0u, v, 0x112, 0xf, 0xf, true);
    v += __builtin_amdgcn_update_dpp(0u, v, 0x114, 0xf, 0xf, true);
    v += __builtin_amdgcn_update_dpp(0u, v, 0x118, 0xf, 0xf, true);
    v += __builtin_amdgcn_update_dpp(0u, v, 0x142, 0xa, 0xf, false);
    v += __builtin_amdgcn_update_dpp(0u, v, 0x143, 0xc, 0xf, false);
    return __builtin_amdgcn_readlane(v, 63);
}
__device__ __forceinline__ unsigned bfi(unsigned m, unsigned a, unsigned b) { return (a & m) | (b & ~m); }
__device__ __forceinline__ unsigned lds_ld1(unsigned addr) { unsigned r; asm volatile("ds_read_b32 %0, %1\n\ts_waitcnt lgkmcnt(0)" : "=&v"(r) : "v"(addr) : "memory"); return rfl(r); }
__device__ __forceinline__ unsigned wait_word(unsigned addr, unsigned mask, unsigned need) {
    unsigned v = 0;
#pragma nounroll
    for (unsigned sp = 0; sp < (1u << 25); ++sp) {
        v = lds_ld1(addr);
        if ((v & mask) >= need) break;
        __builtin_amdgcn_s_sleep(1);
    }
    return v;
}
__device__ __forceinline__ void wait_free(unsigned sync, unsigned need) {
#pragma nounroll
    for (unsigned sp = 0; sp < (1u << 25); ++sp) {
        if (min4(lds_ld4(sync + 16)) >= need) break;
        __builtin_amdgcn_s_sleep(1);
    }
}

#define STAMPH(var, idx) do {} while (0)
#define STAMP(var, idx) do {} while (0)

#define MF(ACC, W, X) "v_mfma_f32_32x32x64_f8f6f4 " ACC ", " W ", " X ", " ACC " cbsz:1 blgp:1\n\t"
#define RX(R, A, OFF) "ds_read_b128 " R ", " A " offset:" OFF "\n\t"
#define RW(R, A, OFF) "ds_read_b64_tr_b8 " R ", " A " offset:" OFF "\n\t"
#define R0_W0a RW("v[64:65]", "%[b0]", "16384") RW("v[66:67]", "%[b0]", "18432") RW("v[68:69]", "%[b0]", "20480")
#define R0_W0b RW("v[70:71]", "%[b0]", "22528") RX("v[32:35]", "%[a0]", "0") RX("v[36:39]", "%[a1]", "0")
#define R0_W1a RW("v[72:73]", "%[b1]", "16384") RW("v[74:75]", "%[b1]", "18432") RW("v[76:77]", "%[b1]", "20480")
#define R0_W1b RW("v[78:79]", "%[b1]", "22528") RX("v[40:43]", "%[a0]", "4096") RX("v[44:47]", "%[a1]", "4096")
#define R0_X2  RX("v[48:51]", "%[a0]", "8192") RX("v[52:55]", "%[a1]", "8192")
#define R0_X3  RX("v[56:59]", "%[a0]", "12288") RX("v[60:63]", "%[a1]", "12288")
#define R1_W0a RW("v[112:113]", "%[b0]", "32768") RW("v[114:115]", "%[b0]", "34816") RW("v[116:117]", "%[b0]", "36864")
#define R1_W0b RW("v[118:119]", "%[b0]", "38912") RX("v[80:83]", "%[a2]", "0") RX("v[84:87]", "%[a3]", "0")
#define R1_W1a RW("v[120:121]", "%[b1]", "32768") RW("v[122:123]", "%[b1]", "34816") RW("v[124:125]", "%[b1]", "36864")
#define R1_W1b RW("v[126:127]", "%[b1]", "38912") RX("v[88:91]", "%[a2]", "4096") RX("v[92:95]", "%[a3]", "4096")
#define R1_X2  RX("v[96:99]", "%[a2]", "8192") RX("v[100:103]", "%[a3]", "8192")
#define R1_X3  RX("v[104:107]", "%[a2]", "12288") RX("v[108:111]", "%[a3]", "12288")
#define CLOB8(a,b,c,d,e,f,g,h) "v" #a, "v" #b, "v" #c, "v" #d, "v" #e, "v" #f, "v" #g, "v" #h
#define FRAG_CLOBBERS CLOB8(28,29,30,31,32,33,34,35), CLOB8(36,37,38,39,40,41,42,43), CLOB8(44,45,46,47,48,49,50,51), CLOB8(52,53,54,55,56,57,58,59), \
    CLOB8(60,61,62,63,64,65,66,67), CLOB8(68,69,70,71,72,73,74,75), CLOB8(76,77,78,79,80,81,82,83), CLOB8(84,85,86,87,88,89,90,91), \
    CLOB8(92,93,94,95,96,97,98,99), CLOB8(100,101,102,103,104,105,106,107), CLOB8(108,109,110,111,112,113,114,115), CLOB8(116,117,118,119,120,121,122,123), \
    "v124", "v125", "v126", "v127"

#define OUT_BITS " sc1"
#define DF 1
#define DL 2
#define EARLY 1
#define BROT 0
#define HPRIO 0
#define CSLEEP
#define FIRSTMASK 0xfu
__device__ __forceinline__ void epi_block(const LAS unsigned char* blk, unsigned lane, __amdgpu_buffer_rsrc_t ro, unsigned voff, unsigned soff, float scale, float nsl) {
    typedef float f32x2 __attribute__((ext_vector_type(2)));
    f32x4 v[8];
#pragma unroll
    for (int i = 0; i < 8; ++i) { const unsigned row = 4 * i + (lane >> 4); v[i] = *(const LAS f32x4*)(blk + row * 256 + (((lane & 15) ^ (row & 7)) << 4)); }
#pragma unroll
    for (int i = 0; i < 8; ++i) {
        const f32x4 y = v[i] * scale, t = v[i] * nsl;
        f32x4 d;
#pragma unroll
        for (int j = 0; j < 4; ++j) d[j] = __builtin_amdgcn_exp2f(__builtin_amdgcn_fmed3f(t[j], -86.5617024533378f, 86.5617024533378f));
        d = d + 1.f;
#pragma unroll
        for (int j = 0; j < 4; ++j) d[j] = __builtin_amdgcn_rcpf(d[j]);
        const f32x4 o = y * d;
        asm volatile("buffer_store_dwordx4 %0, %1, %2, %3 offen" OUT_BITS "\n\ts_nop 1" :: "v"(o), "v"(voff), "s"(ro), "s"(soff + i * 4 * NL * 4) : "memory");
    }
}

struct Helper {
    __amdgpu_buffer_rsrc_t rs, rw; const unsigned char* flagp; GAS unsigned* tmo; LAS unsigned char* ring;
    unsigned lane, call, sync, hq, s0, raw0, w0, vq, ve, vf, we, wo, fcall, fmeta, ip, cb, m60, csh;
    bool dead;
    u32x4 P[3][3]; unsigned E[3][2]; u32x2 F;
    unsigned long long st0, st1;
    __device__ __forceinline__ void poll(unsigned mask, const unsigned char* inv) {
        const unsigned long long t0 = __builtin_amdgcn_s_memrealtime();
        bool inv_ok = inv == nullptr;
#pragma nounroll
        for (unsigned sp = 0;; ++sp) {
            u32x2 f; unsigned iv = 0;
            if (inv != nullptr) asm volatile("global_load_dwordx2 %0, %2, %3 sc0 sc1\n\tglobal_load_dword %1, %4, %5 sc1\n\ts_waitcnt vmcnt(0)" : "=&v"(f), "=&v"(iv) : "v"(vf), "s"(flagp), "v"(0u), "s"(inv) : "memory");
            else asm volatile("global_load_dwordx2 %0, %1, %2 sc0 sc1\n\ts_waitcnt vmcnt(0)" : "=&v"(f) : "v"(vf), "s"(flagp) : "memory");
            fcall = f.x; fmeta = f.y;
            if (!inv_ok && rfl(iv) >= call) inv_ok = true;
            if (inv_ok && ((unsigned)__ballot(f.x >= call) & mask) == mask) break;
            if ((sp & 15) == 15) {
                const unsigned long long dt = __builtin_amdgcn_s_memrealtime() - t0;
                if (!inv_ok && dt > 2000) { asm volatile("buffer_inv sc0 sc1\n\ts_waitcnt vmcnt(0)" ::: "memory"); inv_ok = true; }
                if (rfl(__hip_atomic_load(tmo, __ATOMIC_RELAXED, __HIP_MEMORY_SCOPE_AGENT)) == call || dt > DEADLINE_TICKS) {
                    __hip_atomic_store(tmo, call, __ATOMIC_RELAXED, __HIP_MEMORY_SCOPE_AGENT);
                    dead = true; break;
                }
            }
        }
    }
    template <int D> __device__ __forceinline__ void issue(unsigned k) {
        const unsigned cnt = (__builtin_amdgcn_readlane(fmeta, k) >> csh) & 0xff;
        const unsigned so = s0 + k * QU, so2 = so + (cnt > 64 ? 128u : 0u);
        asm volatile("buffer_load_dwordx4 %0, %5, %7, %8 offen\n\tbuffer_load_dwordx4 %1, %5, %7, %8 offen offset:1024\n\tbuffer_load_dwordx4 %2, %5, %7, %8 offen offset:2048\n\t"
                     "buffer_load_ushort %3, %6, %7, %8 offen offset:3072\n\tbuffer_load_ushort %4, %6, %7, %9 offen offset:3072"
                     : "+v"(P[D][0]), "+v"(P[D][1]), "+v"(P[D][2]), "+v"(E[D][0]), "+v"(E[D][1])
                     : "v"(vq), "v"(ve), "s"(rs), "s"(so), "s"(so2) : "memory");
    }
    __device__ __forceinline__ void quarter(const u32x4& A, const u32x4& B, const u32x4& C, unsigned KK, LAS unsigned char* dst) {
        const unsigned lut = KK + 0x90801000u;
        u32x4 a, b, c, d;
#pragma unroll
        for (int j = 0; j < 4; j += 2) {
            unsigned t0, t1, t2, t3, t4, t5;
            asm("v_and_b32 %8, %20, %14\n\tv_and_b32 %9, %20, %15\n\tv_and_b32 %10, %20, %16\n\tv_and_b32 %11, %20, %17\n\tv_and_b32 %12, %20, %18\n\tv_and_b32 %13, %20, %19\n\t"
                "v_xad_u32 %0, %14, %8, %21\n\tv_xad_u32 %1, %15, %9, %21\n\tv_xad_u32 %2, %16, %10, %21\n\tv_xad_u32 %3, %17, %11, %21\n\tv_xad_u32 %4, %18, %12, %21\n\tv_xad_u32 %5, %19, %13, %21\n\t"
                "v_lshrrev_b32 %12, 5, %12\n\tv_lshrrev_b32 %13, 5, %13\n\tv_lshrrev_b32 %8, 5, %8\n\tv_lshrrev_b32 %9, 5, %9\n\t"
                "v_perm_b32 %12, %22, %22, %12\n\tv_perm_b32 %13, %22, %22, %13\n\tv_lshrrev_b32 %10, 3, %10\n\tv_lshrrev_b32 %11, 3, %11\n\t"
                "v_add3_u32 %6, %8, %10, %12\n\tv_add3_u32 %7, %9, %11, %13"
                : "=&v"(a[j]), "=&v"(a[j + 1]), "=&v"(b[j]), "=&v"(b[j + 1]), "=&v"(c[j]), "=&v"(c[j + 1]), "=&v"(d[j]), "=&v"(d[j + 1]),
                  "=&v"(t0), "=&v"(t1), "=&v"(t2), "=&v"(t3), "=&v"(t4), "=&v"(t5)
                : "v"(A[j]), "v"(A[j + 1]), "v"(B[j]), "v"(B[j + 1]), "v"(C[j]), "v"(C[j + 1]), "s"(m60), "s"(KK), "s"(lut));
        }
        *(LAS u32x4*)(dst) = a; *(LAS u32x4*)(dst + 1024) = b; *(LAS u32x4*)(dst + 2048) = c; *(LAS u32x4*)(dst + 3072) = d;
    }
    __device__ __forceinline__ void patch(unsigned e, unsigned base) {
        const int j = __builtin_amdgcn_sbfe((int)e, 13, 3);
        if (j != 0) asm volatile("ds_add_u32 %0, %1" :: "v"((e & 0xffcu) | base), "v"((unsigned)j << (((e << 3) + 5) & 31)) : "memory");
    }
    template <int D> __device__ __forceinline__ void ahead(unsigned k) {
        unsigned tg = k + DF;
        if (DL > DF && ((k + DL) & 7) == cb) tg = k + DL;
        if (tg >= NK) tg = NK - 1;
        if (ip <= tg) {
            if (__builtin_amdgcn_readlane(fcall, tg) < call || __builtin_amdgcn_readlane(fcall, ip) < call) { poll((1u << tg) | (1u << ip), nullptr); if (dead) return; }
            if (ip == k + 1) { issue<(D + 1) % 3>(ip); ++ip; }
            if (ip == k + 2 && ip <= tg) { issue<(D + 2) % 3>(ip); ++ip; }
        }
    }
    __device__ __forceinline__ void fill_w(unsigned k, unsigned slot) { dma8(rw, we, wo, w0 + k * (BK * NL), 4 * NL, (unsigned)(size_t)ring + slot * STAGE + A_BYTES + hq * 8192); }
    template <int D> __device__ __forceinline__ void stage(unsigned k) {
        if (k == 16) STAMPH(st0, 8);
        const unsigned sel = k == NK - 1 ? 0u : ip - k;
        asm volatile("s_cmp_eq_u32 %6, 1\n\ts_cbranch_scc1 .Lw1%=\n\ts_cmp_eq_u32 %6, 2\n\ts_cbranch_scc1 .Lw2%=\n\ts_waitcnt vmcnt(0)\n\ts_branch .Lwe%=\n"
                     ".Lw2%=:\n\ts_waitcnt vmcnt(13)\n\ts_branch .Lwe%=\n.Lw1%=:\n\ts_waitcnt vmcnt(8)\n.Lwe%=:"
                     : "+v"(P[D][0]), "+v"(P[D][1]), "+v"(P[D][2]), "+v"(E[D][0]), "+v"(E[D][1]), "+v"(F) : "s"(sel) : "memory", "scc");
        if (k == 31 || k == 0) STAMP(st1, k);
        if (k == 16) STAMPH(st0, 9);
        if (D == 1 && k == 1) asm volatile("global_load_dwordx2 %0, %1, %2 sc0 sc1" : "+v"(F) : "v"(vf), "s"(flagp) : "memory");
        if (D == 2 && k == 2) { const bool newer = F.x >= call; fcall = newer ? F.x : fcall; fmeta = newer ? F.y : fmeta; }
        const unsigned meta = __builtin_amdgcn_readlane(fmeta, k), KK = (meta & 0x7f) * 0x01010101u, cnt = (meta >> csh) & 0xff;
        ahead<D>(k);
        LAS unsigned char* dst = ring + D * STAGE + hq * 4096;
        const unsigned base = (unsigned)(size_t)dst;
        if (cnt <= EXC_CAP) {
            quarter(P[D][0], P[D][1], P[D][2], KK, dst + lane * 16);
            patch(E[D][0], base);
            if (cnt > 64) patch(E[D][1], base);
        } else {
            dma4(rs, vq, vq, raw0 + k * 4096, 1024, rfl(base));
            asm volatile("s_waitcnt vmcnt(0)" ::: "memory");
        }
        lds_st(sync + 4 * hq, k + 1);
        if (k == 16) STAMPH(st0, 10);
        if (k + 2 < NK) {
            if (k >= 1) wait_free(sync, k);
            if (k == 16) STAMPH(st0, 11);
            fill_w(k + 2, (D + 2) % 3);
        }
        if (k == 16) STAMPH(st0, 12);
    }
    __device__ __forceinline__ void drain() {
#pragma unroll
        for (int D = 0; D < 3; ++D)
            asm volatile("s_waitcnt vmcnt(0)" : "+v"(P[D][0]), "+v"(P[D][1]), "+v"(P[D][2]), "+v"(E[D][0]), "+v"(E[D][1]), "+v"(F) :: "memory");
    }
};

__global__ void __launch_bounds__(512, 2) ag_gemm_kernel(Params p) {
    __shared__ __attribute__((aligned(1024))) unsigned char smem[LDS_BYTES];
    const unsigned lds0 = (unsigned)(size_t)(LAS unsigned char*)smem;
    const unsigned sync = lds0 + SYNC_OFF;
    const unsigned lane = threadIdx.x & 63, wid = rfl(threadIdx.x >> 6), bid = blockIdx.x;
    const unsigned rank = p.rank & 7;
    const unsigned src = bid & 7, rb = (bid >> 3) & 3, cb = bid >> 5;
    unsigned char* const myws = p.ws[rank];
    const unsigned cks = ((bid >> 3) + 31) & 31, crb = (bid >> 1) & 3, chf = bid & 1;
    GAS unsigned* const cntp = (GAS unsigned*)(myws + WS_CNT + bid * 64);
    unsigned char* const invp = myws + WS_TMO + 128 + (__builtin_amdgcn_s_getreg((3 << 11) | 20) & 15u) * 128;
    const __amdgpu_buffer_rsrc_t rx = __builtin_amdgcn_make_buffer_rsrc((void*)p.x, 0, (unsigned)SHARD, 0x00020000);
    const unsigned xso = (crb * BM + chf * 64) * K_ + cks * BK;
    unsigned call, Kb;
    {
        if (bid < 8 && wid == 1) asm volatile("buffer_inv sc0 sc1" ::: "memory");
        const unsigned q2 = wid >> 2, sl = wid & 3;
        const unsigned cv = __hip_atomic_load(cntp, __ATOMIC_RELAXED, __HIP_MEMORY_SCOPE_AGENT);
        const u32x4 smp = __builtin_amdgcn_raw_buffer_load_b128(rx, lane * K_ + (lane & 7) * 16, xso, 0);
        unsigned x[4];
#pragma unroll
        for (int i = 0; i < 4; ++i)
            x[i] = __builtin_amdgcn_raw_buffer_load_b32(rx, (2 * sl + (lane >> 5)) * K_ + ((((lane >> 2) & 7) ^ (4 * (i & 1) + sl)) << 4) + 4 * (lane & 3), xso + (32 * q2 + 8 * i) * K_, 0);
        if (threadIdx.x < 144) lds_st(lds0 + LIST_OFF + 4 * threadIdx.x, 0u);
        asm volatile("s_waitcnt lgkmcnt(0)\n\ts_barrier" ::: "memory");
        call = rfl(cv) + 1u;
        unsigned sum = 0;
#pragma unroll
        for (int i = 0; i < 4; ++i) sum = __builtin_amdgcn_sad_u8(smp[i] & 0x7f7f7f7fu, 0u, sum);
        const int kq = ((int)wave_sum(sum) - 20582) >> 10;
        Kb = kq < 0 ? 0u : kq > 96 ? 96u : (unsigned)kq;
        const unsigned KK = Kb * 0x01010101u;
        unsigned e[4], c[4], M = 0;
#pragma unroll
        for (int i = 0; i < 4; ++i) {
            const unsigned t = (x[i] | 0x80808080u) - KK;
            e[i] = (t ^ 0x80808080u) & 0xe0e0e0e0u;
            c[i] = bfi(0x1f1f1f1fu, t, x[i]);
            M |= (((e[i] >> 1) + 0x70707070u) & 0x80808080u) >> (7 - i);
        }
        LAS unsigned* enc = (LAS unsigned*)((LAS unsigned char*)smem + ENC_OFF + q2 * 3072 + sl * 256) + lane;
        enc[0] = bfi(0x60606060u, c[3] << 5, c[0]);
        enc[256] = bfi(0x60606060u, c[3] << 3, c[1]);
        enc[512] = bfi(0x20202020u, c[3] << 1, bfi(0x40404040u, c[3] >> 1, c[2]));
        const unsigned tot = wave_sum(__builtin_popcount(M));
        unsigned idx;
        asm volatile("s_mov_b64 exec, 1\n\tds_add_rtn_u32 %0, %1, %2\n\ts_mov_b64 exec, -1\n\ts_waitcnt lgkmcnt(0)" : "=&v"(idx) : "v"(sync + 36 + 4 * q2), "v"(tot) : "memory");
        idx = rfl(idx);
        LAS unsigned short* list = (LAS unsigned short*)((LAS unsigned char*)smem + LIST_OFF + 256 * q2);
#pragma nounroll
        for (;;) {
            const unsigned long long act = __ballot(M != 0);
            if (act == 0ull) break;
            if (M != 0) {
                const unsigned bit = __builtin_ctz(M), i = bit & 3, by = bit >> 3;
                const unsigned ei = i == 0 ? e[0] : i == 1 ? e[1] : i == 2 ? e[2] : e[3];
                const unsigned my = idx + __builtin_amdgcn_mbcnt_hi((unsigned)(act >> 32), __builtin_amdgcn_mbcnt_lo((unsigned)act, 0u));
                if (my < EXC_CAP) list[my] = (unsigned short)((i * 1024 + sl * 256 + lane * 4 + by) | (((ei >> (8 * by + 5)) & 7) << 13));
                M &= M - 1;
            }
            idx += __builtin_popcountll(act);
        }
        if (bid < 8 && wid == 1) {
            asm volatile("s_waitcnt vmcnt(0)" ::: "memory");
            if (lane == 0) __hip_atomic_store((GAS unsigned*)invp, call, __ATOMIC_RELAXED, __HIP_MEMORY_SCOPE_AGENT);
        }
        asm volatile("s_waitcnt lgkmcnt(0)\n\ts_barrier" ::: "memory");
    }
    if (wid == 0) {
        STAMP(st0, 10);
        const u32x4 cw = lds_ld4(sync + 32);
        const unsigned c0 = min(rfl(cw.y), 255u), c1 = min(rfl(cw.z), 255u), hu = (crb * 2 + chf) * 32 + cks, qu = (crb * 4 + chf * 2) * 32 + cks;
        const unsigned meta = Kb | (c0 << 8) | (c1 << 16);
        {
            const LAS u32x4* enc = (const LAS u32x4*)((LAS unsigned char*)smem + ENC_OFF) + lane;
            const u32x4 d0 = enc[0], d1 = enc[64], d2 = enc[128], d3 = enc[192], d4 = enc[256], d5 = enc[320];
            const u32x4 d6 = *((const LAS u32x4*)((LAS unsigned char*)smem + LIST_OFF) + (lane & 31));
            unsigned char* dst = myws + WS_DATA + (call & 1) * PACK_PAR + qu * QU;
            asm volatile("global_store_dwordx4 %7, %0, %10 sc0 sc1\n\tglobal_store_dwordx4 %7, %1, %10 offset:1024 sc0 sc1\n\tglobal_store_dwordx4 %7, %2, %10 offset:2048 sc0 sc1\n\t"
                         "global_store_dwordx4 %8, %3, %10 sc0 sc1\n\tglobal_store_dwordx4 %8, %4, %10 offset:1024 sc0 sc1\n\tglobal_store_dwordx4 %8, %5, %10 offset:2048 sc0 sc1\n\t"
                         "s_mov_b64 exec, 0xffffffff\n\tglobal_store_dwordx4 %9, %6, %10 offset:3072 sc0 sc1\n\ts_mov_b64 exec, -1\n\ts_nop 1"
                         :: "v"(d0), "v"(d1), "v"(d2), "v"(d3), "v"(d4), "v"(d5), "v"(d6), "v"(lane * 16), "v"(lane * 16 + 32 * QU), "v"((lane & 15) * 16 + ((lane >> 4) & 1) * 32 * QU), "s"(dst) : "memory");
        }
        if (c0 > EXC_CAP || c1 > EXC_CAP) {
            unsigned char* dst = myws + WS_DATA + RAW_OFF + (call & 1) * (unsigned)SHARD + qu * 4096;
#pragma nounroll
            for (unsigned i = 0; i < 8; ++i) {
                const u32x4 d = __builtin_amdgcn_raw_buffer_load_b128(rx, (lane >> 3) * K_ + (((lane & 7) ^ (4 * (i & 1) + (lane >> 4))) << 4), xso + 8 * i * K_, 0);
                asm volatile("global_store_dwordx4 %1, %0, %2 sc0 sc1\n\ts_nop 1" :: "v"(d), "v"(lane * 16 + (i & 3) * 1024 + (i >> 2) * 32 * 4096), "s"(dst) : "memory");
            }
        }
        asm volatile("s_waitcnt vmcnt(0)" ::: "memory");
        const u32x2 fv = {call, meta};
#pragma unroll
        for (int i = 1; i < NR; ++i) {
            unsigned char* fp = p.ws[(rank + i) & 7] + WS_FLAG + (((call & 1) * NR + rank) * 256 + hu) * 8;
            asm volatile("s_mov_b64 exec, 1\n\tglobal_store_dwordx2 %0, %1, %2 sc0 sc1\n\ts_mov_b64 exec, -1" :: "v"(0u), "v"(fv), "s"(fp) : "memory");
        }
        if (lane == 0) __hip_atomic_store(cntp, call, __ATOMIC_RELAXED, __HIP_MEMORY_SCOPE_AGENT);
        STAMP(st0, 11);
    }

    float scale;
    if (wid < 4) {
        scale = p.sx[0] * p.sw[0];
        asm volatile("" : "+v"(scale));
        const unsigned r = lane & 31, h = lane >> 5, key = (r >> 1) & 7;
        unsigned a0 = lds0 + r * 128 + (((2 * h + 0) ^ key) << 4), a1 = lds0 + r * 128 + (((2 * h + 1) ^ key) << 4);
        unsigned a2 = lds0 + r * 128 + (((4 + 2 * h + 0) ^ key) << 4), a3 = lds0 + r * 128 + (((4 + 2 * h + 1) ^ key) << 4);
        const unsigned ch = (lane >> 4) & 1, q = (lane & 15) >> 1, pp = lane & 1;
        unsigned b0 = lds0 + (32 * h + q) * 256 + (((4 * wid + 0 + ch) ^ (2 * q)) << 4) + 8 * pp;
        unsigned b1 = lds0 + (32 * h + q) * 256 + (((4 * wid + 2 + ch) ^ (2 * q)) << 4) + 8 * pp;
        f32x16 acc[8];
#pragma unroll
        for (int i = 0; i < 8; ++i)
#pragma unroll
            for (int j = 0; j < 16; ++j) acc[i][j] = 0.f;
        unsigned kk = 0, slot = 0, dl, tmp, spin = 0;
        const unsigned vsync = sync, vcons = sync + 16 + 4 * wid;
        asm volatile(
            "s_nop 1\n\t"
            "s_branch .Lspin%=\n"
            ".Lloop%=:\n\t"
            "s_waitcnt lgkmcnt(0)\n\t"
            MF("%0", "v[64:71]", "v[32:39]") R1_W0a
            MF("%1", "v[72:79]", "v[32:39]") R1_W0b
            MF("%2", "v[64:71]", "v[40:47]") R1_W1a
            MF("%3", "v[72:79]", "v[40:47]") R1_W1b
            MF("%4", "v[64:71]", "v[48:55]") R1_X2
            "s_add_u32 %[kk], %[kk], 1\n\t"
            "v_mov_b32 v28, %[kk]\n\t"
            MF("%5", "v[72:79]", "v[48:55]") R1_X3
            "ds_write_b32 %[vc], v28\n\t"
            "s_add_u32 %[sl], %[sl], 1\n\t"
            "s_cmp_eq_u32 %[sl], 3\n\t"
            "s_cselect_b32 %[dl], %[dn], %[dp]\n\t"
            "s_cselect_b32 %[sl], 0, %[sl]\n\t"
            MF("%6", "v[64:71]", "v[56:63]")
            "ds_read_b128 v[28:31], %[vs]\n\t"
            "v_add_u32 %[a0], %[dl], %[a0]\n\tv_add_u32 %[a1], %[dl], %[a1]\n\tv_add_u32 %[a2], %[dl], %[a2]\n\t"
            "v_add_u32 %[a3], %[dl], %[a3]\n\tv_add_u32 %[b0], %[dl], %[b0]\n\tv_add_u32 %[b1], %[dl], %[b1]\n\t"
            MF("%7", "v[72:79]", "v[56:63]")
            "s_waitcnt lgkmcnt(0)\n\t"
            MF("%0", "v[112:119]", "v[80:87]")
            "v_min3_u32 v28, v28, v29, v30\n\t"
            "v_min_u32 v28, v28, v31\n\t"
            "s_nop 0\n\t"
            "v_readfirstlane_b32 %[tm], v28\n\t"
            "s_nop 0\n\t"
            "s_cmp_gt_u32 %[tm], %[kk]\n\t"
            "s_cbranch_scc0 .Lslow%=\n\t"
            R0_W0a
            MF("%1", "v[120:127]", "v[80:87]") R0_W0b
            MF("%2", "v[112:119]", "v[88:95]") R0_W1a
            MF("%3", "v[120:127]", "v[88:95]") R0_W1b
            MF("%4", "v[112:119]", "v[96:103]") R0_X2
            MF("%5", "v[120:127]", "v[96:103]") R0_X3
            MF("%6", "v[112:119]", "v[104:111]")
            MF("%7", "v[120:127]", "v[104:111]")
            "s_branch .Lloop%=\n"
            ".Lslow%=:\n\t"
            MF("%1", "v[120:127]", "v[80:87]")
            MF("%2", "v[112:119]", "v[88:95]")
            MF("%3", "v[120:127]", "v[88:95]")
            MF("%4", "v[112:119]", "v[96:103]")
            MF("%5", "v[120:127]", "v[96:103]")
            MF("%6", "v[112:119]", "v[104:111]")
            MF("%7", "v[120:127]", "v[104:111]")
            "s_cmp_ge_u32 %[kk], 32\n\t"
            "s_cbranch_scc1 .Ldone%=\n"
            ".Lspin%=:\n\t"
            CSLEEP
            "ds_read_b128 v[28:31], %[vs]\n\t"
            "s_add_u32 %[sp], %[sp], 1\n\t"
            "s_waitcnt lgkmcnt(0)\n\t"
            "v_min3_u32 v28, v28, v29, v30\n\t"
            "v_min_u32 v28, v28, v31\n\t"
            "s_nop 0\n\t"
            "v_readfirstlane_b32 %[tm], v28\n\t"
            "s_cmp_gt_u32 %[sp], 0x2000000\n\t"
            "s_cbranch_scc1 .Ldone%=\n\t"
            "s_cmp_gt_u32 %[tm], %[kk]\n\t"
            "s_cbranch_scc0 .Lspin%=\n\t"
            R0_W0a R0_W0b R0_W1a R0_W1b R0_X2 R0_X3
            "s_branch .Lloop%=\n"
            ".Ldone%=:\n\t"
            "s_nop 15\n\ts_nop 7"
            : "+a"(acc[0]), "+a"(acc[1]), "+a"(acc[2]), "+a"(acc[3]), "+a"(acc[4]), "+a"(acc[5]), "+a"(acc[6]), "+a"(acc[7]),
              [a0] "+v"(a0), [a1] "+v"(a1), [a2] "+v"(a2), [a3] "+v"(a3), [b0] "+v"(b0), [b1] "+v"(b1),
              [kk] "+s"(kk), [sl] "+s"(slot), [dl] "=&s"(dl), [tm] "=&s"(tmp), [sp] "+s"(spin)
            : [vs] "v"(vsync), [vc] "v"(vcons), [dp] "s"(STAGE), [dn] "s"(0u - (NSTAGE - 1) * STAGE)
            : "memory", "scc", FRAG_CLOBBERS);
        STAMP(st0, 0);
        if (kk != (unsigned)NK) lds_st(sync + 32, 1u);
        wait_free(sync, NK);
        LAS unsigned char* reg = (LAS unsigned char*)smem + wid * 32768 + r * 256;
#pragma unroll
        for (int mi = 0; mi < 4; ++mi) {
#pragma unroll
            for (int ni = 0; ni < 2; ++ni)
#pragma unroll
                for (int g = 0; g < 4; ++g) {
                    const f32x16& a = acc[2 * mi + ni];
                    *(LAS f32x4*)(reg + mi * 8192 + (((8 * ni + 2 * g + h) ^ (r & 7)) << 4)) = (f32x4){a[4 * g], a[4 * g + 1], a[4 * g + 2], a[4 * g + 3]};
                }
            if (mi < 2) lds_st(sync + 48 + 4 * wid, mi + 1);
        }
    } else {
        __builtin_amdgcn_s_setprio(HPRIO);
        Helper d;
        d.hq = wid - 4; d.lane = lane; d.call = call; d.sync = sync; d.ring = (LAS unsigned char*)smem; d.cb = cb; d.dead = false;
        d.tmo = (GAS unsigned*)(myws + WS_TMO);
        d.rw = __builtin_amdgcn_make_buffer_rsrc((void*)p.w, 0, (unsigned)(K_ * NL), 0x00020000);
        d.we = (lane >> 4) * NL + (((lane & 15) ^ (2 * (lane >> 4))) << 4);
        d.wo = (lane >> 4) * NL + (((lane & 15) ^ (2 * (4 + (lane >> 4)))) << 4);
        d.w0 = d.hq * 32 * NL + cb * BN;
        d.st0 = 0; d.st1 = 0;
        if (src == rank) {
            const unsigned ve = (lane >> 3) * K_ + (((lane & 7) ^ (lane >> 4)) << 4), vo = (lane >> 3) * K_ + (((lane & 7) ^ (4 + (lane >> 4))) << 4);
            unsigned slot = 0;
#pragma nounroll
            for (unsigned k = 0; k < NK; ++k) {
                if (k >= NSTAGE) wait_free(sync, k - NSTAGE + 1);
                dma4(rx, ve, vo, (rb * BM + d.hq * 32) * K_ + k * BK, 8 * K_, lds0 + slot * STAGE + d.hq * 4096);
                d.fill_w(k, slot);
                asm volatile("s_waitcnt vmcnt(12)" ::: "memory");
                lds_st(sync + 4 * d.hq, k);
                slot = slot == NSTAGE - 1 ? 0 : slot + 1;
            }
            asm volatile("s_waitcnt vmcnt(0)" ::: "memory");
        } else {
            d.rs = __builtin_amdgcn_make_buffer_rsrc((void*)(p.ws[src] + WS_DATA), 0, DATA_BYTES, 0x00020000);
            d.flagp = myws + WS_FLAG + (((call & 1) * NR + src) * 256 + (rb * 2 + (d.hq >> 1)) * 32) * 8;
            d.s0 = (call & 1) * PACK_PAR + (rb * 4 + d.hq) * 32 * QU; d.raw0 = RAW_OFF + (call & 1) * (unsigned)SHARD + (rb * 4 + d.hq) * 32 * 4096;
            d.vq = lane * 16; d.ve = lane * 2; d.vf = (lane & 31) * 8; d.csh = 8 + 8 * (d.hq & 1);
            d.m60 = 0x60606060u; asm volatile("" : "+s"(d.m60));
            d.fill_w(0, 0);
            d.poll(FIRSTMASK, invp);
            if (!d.dead) {
                d.ip = 0;
#pragma unroll
                for (int i = 0; i < 3; ++i) asm volatile("" : "=v"(d.P[i][0]), "=v"(d.P[i][1]), "=v"(d.P[i][2]), "=v"(d.E[i][0]), "=v"(d.E[i][1]));
                asm volatile("" : "=v"(d.F));
                STAMP(d.st0, 0);
                d.ahead<2>(~0u);
                d.fill_w(1, 1);
#pragma nounroll
                for (unsigned k = 0;; k += 3) {
                    d.stage<0>(k); if (d.dead) break;
                    d.stage<1>(k + 1); if (d.dead || k + 2 >= NK) break;
                    d.stage<2>(k + 2); if (d.dead) break;
                }
                d.drain();
            }
        }
        if (d.dead) lds_st(sync + 32, 1u);
        lds_st(sync + 4 * d.hq, NK);
        if (rfl(__hip_atomic_load(d.tmo, __ATOMIC_RELAXED, __HIP_MEMORY_SCOPE_AGENT)) == call) lds_st(sync + 32, 1u);
        __builtin_amdgcn_s_setprio(0);
    }
    {
        if (wid >= 4) scale = p.sx[0] * p.sw[0];
        const unsigned cw = wid & 3, b0 = wid < 4 ? 2 : 0;
        const __amdgpu_buffer_rsrc_t ro = __builtin_amdgcn_make_buffer_rsrc((void*)p.out, 0, (unsigned)(M_ * NL * 4), 0x00020000);
        const unsigned voff = (lane >> 4) * NL * 4 + (lane & 15) * 16;
        const unsigned soff = ((src * MS + rb * BM + b0 * 32) * NL + cb * BN + cw * 64) * 4;
        const LAS unsigned char* blk = (const LAS unsigned char*)smem + cw * 32768 + b0 * 8192;
        if (wid >= 4) wait_word(sync + 48 + 4 * cw, ~0u, 1);
        if (lds_ld1(sync + 32) != 0u) scale = __builtin_nanf("");
        const float nsl = scale * -1.44269504088896340736f;
        epi_block(blk, lane, ro, voff, soff, scale, nsl);
        if (wid >= 4) wait_word(sync + 48 + 4 * cw, ~0u, 2);
        epi_block(blk + 8192, lane, ro, voff, soff + 32 * NL * 4, scale, nsl);
    }
}

}

extern "C" void kernel_launch(int rank, int nranks, ncclComm_t comm, hipStream_t stream,
                              void* const* d_in, const int* in_sizes, int n_in,
                              void* d_out, int out_size, const int* in_dtypes, int out_dtype,
                              void* const* d_out_all, void* const* d_ws_all, size_t ws_bytes) {
    (void)comm; (void)in_dtypes; (void)out_dtype; (void)d_out_all;
    if (nranks != NR || n_in != 4 || rank < 0 || rank >= NR || in_sizes[0] != MS * K_ || in_sizes[1] != K_ * NL || out_size != M_ * NL || ws_bytes < WS_NEED) {
        fprintf(stderr, "ag_gemm: unexpected shapes\n");
        return;
    }
    Params p;
    p.x = (const unsigned char*)d_in[0]; p.w = (const unsigned char*)d_in[1];
    p.sx = (const float*)d_in[2]; p.sw = (const float*)d_in[3]; p.out = (float*)d_out;
    for (int i = 0; i < NR; ++i) p.ws[i] = (unsigned char*)d_ws_all[i];
    p.rank = (unsigned)rank; p.pad = 0;
    hipLaunchKernelGGL(ag_gemm_kernel, dim3(NWG), dim3(512), 0, stream, p);
    hipError_t e = hipGetLastError();
    if (e != hipSuccess) fprintf(stderr, "ag_gemm launch: %s\n", hipGetErrorString(e));
}
```

```cpp
#include <hip/hip_runtime.h>
#include <rccl/rccl.h>
#include <cstdio>
#include <cstdint>
#include <utility>

namespace {

constexpr int NR = 8, M_ = 4096, K_ = 4096, NL = 2048, MS = M_ / NR;
constexpr int BM = 128, BN = 256, BK = 128, NK = K_ / BK;
constexpr unsigned A_BYTES = BM * BK, B_BYTES = BK * BN, STAGE = A_BYTES + B_BYTES, NSTAGE = 3;
constexpr unsigned ENC_OFF = NSTAGE * STAGE;
constexpr unsigned LIST_OFF = ENC_OFF + 6144;
constexpr unsigned SYNC_OFF = LIST_OFF + 512;
constexpr unsigned LDS_BYTES = SYNC_OFF + 64;
constexpr int NWG = 256;
constexpr unsigned EXC_CAP = 128;
constexpr size_t SHARD = (size_t)MS * K_;
constexpr size_t WS_CNT = 0;
constexpr size_t WS_TMO = 64 << 10;
constexpr size_t WS_FLAG = 128 << 10;
constexpr size_t WS_DATA = 1 << 20;
constexpr unsigned QU = 3072 + 256;
constexpr unsigned PACK_PAR = 512 * QU;
constexpr unsigned RAW_OFF = 4 << 20;
constexpr unsigned DATA_BYTES = RAW_OFF + 2 * (unsigned)SHARD;
static_assert(2 * PACK_PAR <= RAW_OFF, "workspace map");
constexpr size_t WS_STAMP = WS_DATA + DATA_BYTES;
constexpr size_t WS_NEED = WS_STAMP + (size_t)NWG * 256 * 8;
constexpr unsigned long long DEADLINE_TICKS = 30000000ull;

typedef unsigned u32x2 __attribute__((ext_vector_type(2)));
typedef unsigned u32x4 __attribute__((ext_vector_type(4)));
typedef float f32x4 __attribute__((ext_vector_type(4)));
typedef float f32x16 __attribute__((ext_vector_type(16)));
#define LAS __attribute__((address_space(3)))
#define GAS __attribute__((address_space(1)))

struct Params {
    const unsigned char* x; const unsigned char* w; const float* sx; const float* sw; float* out;
    unsigned char* ws[NR];
    unsigned rank, pad;
};

__device__ __forceinline__ unsigned rfl(unsigned v) { return __builtin_amdgcn_readfirstlane(v); }
__device__ __forceinline__ void lds_st(unsigned addr, unsigned v) { asm volatile("ds_write_b32 %0, %1" :: "v"(addr), "v"(v) : "memory"); }
__device__ __forceinline__ u32x4 lds_ld4(unsigned addr) { u32x4 r; asm volatile("ds_read_b128 %0, %1\n\ts_waitcnt lgkmcnt(0)" : "=&v"(r) : "v"(addr) : "memory"); return r; }
__device__ __forceinline__ unsigned min4(u32x4 v) { return rfl(min(min(v.x, v.y), min(v.z, v.w))); }

#define DMA_PIECE(V, SC) "buffer_load_dwordx4 " V ", %[rs], %[so] offen" SC " lds\n\ts_add_u32 m0, m0, 0x400\n\ts_add_u32 %[so], %[so], %[st]\n\t"
#define DMA_PAIR(SC) DMA_PIECE("%[ve]", SC) DMA_PIECE("%[vo]", SC)
#define DMA16(SC) "s_mov_b32 %[kp], m0\n\ts_mov_b32 m0, %[ld]\n\ts_nop 0\n\t" DMA_PAIR(SC) DMA_PAIR(SC) DMA_PAIR(SC) DMA_PAIR(SC) DMA_PAIR(SC) DMA_PAIR(SC) DMA_PAIR(SC) DMA_PAIR(SC) "s_mov_b32 m0, %[kp]"
__device__ __forceinline__ void dma16(__amdgpu_buffer_rsrc_t rs, unsigned ve, unsigned vo, unsigned soff, unsigned step, unsigned lds) {
    unsigned keep;
    asm volatile(DMA16("") : [kp] "=&s"(keep), [so] "+s"(soff) : [ve] "v"(ve), [vo] "v"(vo), [rs] "s"(rs), [st] "s"(step), [ld] "s"(lds) : "memory", "scc");
}
#define DMA8 "s_mov_b32 %[kp], m0\n\ts_mov_b32 m0, %[ld]\n\ts_nop 0\n\t" DMA_PAIR("") DMA_PAIR("") DMA_PAIR("") DMA_PAIR("") "s_mov_b32 m0, %[kp]"
__device__ __forceinline__ void dma8(__amdgpu_buffer_rsrc_t rs, unsigned ve, unsigned vo, unsigned soff, unsigned step, unsigned lds) {
    unsigned keep;
    asm volatile(DMA8 : [kp] "=&s"(keep), [so] "+s"(soff) : [ve] "v"(ve), [vo] "v"(vo), [rs] "s"(rs), [st] "s"(step), [ld] "s"(lds) : "memory", "scc");
}
#define DMA4 "s_mov_b32 %[kp], m0\n\ts_mov_b32 m0, %[ld]\n\ts_nop 0\n\t" DMA_PAIR("") DMA_PAIR("") "s_mov_b32 m0, %[kp]"
__device__ __forceinline__ void dma4(__amdgpu_buffer_rsrc_t rs, unsigned ve, unsigned vo, unsigned soff, unsigned step, unsigned lds) {
    unsigned keep;
    asm volatile(DMA4 : [kp] "=&s"(keep), [so] "+s"(soff) : [ve] "v"(ve), [vo] "v"(vo), [rs] "s"(rs), [st] "s"(step), [ld] "s"(lds) : "memory", "scc");
}
__device__ __forceinline__ unsigned wave_sum(unsigned v) {
    v += __builtin_amdgcn_update_dpp(0u, v, 0x111, 0xf, 0xf, true);
    v += __builtin_amdgcn_update_dpp(0u, v, 0x112, 0xf, 0xf, true);
    v += __builtin_amdgcn_update_dpp(0u, v, 0x114, 0xf, 0xf, true);
    v += __builtin_amdgcn_update_dpp(0u, v, 0x118, 0xf, 0xf, true);
    v += __builtin_amdgcn_update_dpp(0u, v, 0x142, 0xa, 0xf, false);
    v += __builtin_amdgcn_update_dpp(0u, v, 0x143, 0xc, 0xf, false);
    return __builtin_amdgcn_readlane(v, 63);
}
__device__ __forceinline__ unsigned bfi(unsigned m, unsigned a, unsigned b) { return (a & m) | (b & ~m); }
__device__ __forceinline__ unsigned lds_ld1(unsigned addr) { unsigned r; asm volatile("ds_read_b32 %0, %1\n\ts_waitcnt lgkmcnt(0)" : "=&v"(r) : "v"(addr) : "memory"); return rfl(r); }
__device__ __forceinline__ unsigned wait_word(unsigned addr, unsigned mask, unsigned need) {
    unsigned v = 0;
#pragma nounroll
    for (unsigned sp = 0; sp < (1u << 25); ++sp) {
        v = lds_ld1(addr);
        if ((v & mask) >= need) break;
        __builtin_amdgcn_s_sleep(1);
    }
    return v;
}
__device__ __forceinline__ void wait_free(unsigned sync, unsigned need) {
#pragma nounroll
    for (unsigned sp = 0; sp < (1u << 25); ++sp) {
        if (min4(lds_ld4(sync + 16)) >= need) break;
        __builtin_amdgcn_s_sleep(1);
    }
}

#define STAMPH(var, idx) do {} while (0)
#define STAMP(var, idx) do {} while (0)

#define MF(ACC, W, X) "v_mfma_f32_32x32x64_f8f6f4 " ACC ", " W ", " X ", " ACC " cbsz:1 blgp:1\n\t"
#define RX(R, A, OFF) "ds_read_b128 " R ", " A " offset:" OFF "\n\t"
#define RW(R, A, OFF) "ds_read_b64_tr_b8 " R ", " A " offset:" OFF "\n\t"
#define R0_W0a RW("v[64:65]", "%[b0]", "16384") RW("v[66:67]", "%[b0]", "18432") RW("v[68:69]", "%[b0]", "20480")
#define R0_W0b RW("v[70:71]", "%[b0]", "22528") RX("v[32:35]", "%[a0]", "0") RX("v[36:39]", "%[a1]", "0")
#define R0_W1a RW("v[72:73]", "%[b1]", "16384") RW("v[74:75]", "%[b1]", "18432") RW("v[76:77]", "%[b1]", "20480")
#define R0_W1b RW("v[78:79]", "%[b1]", "22528") RX("v[40:43]", "%[a0]", "4096") RX("v[44:47]", "%[a1]", "4096")
#define R0_X2  RX("v[48:51]", "%[a0]", "8192") RX("v[52:55]", "%[a1]", "8192")
#define R0_X3  RX("v[56:59]", "%[a0]", "12288") RX("v[60:63]", "%[a1]", "12288")
#define R1_W0a RW("v[112:113]", "%[b0]", "32768") RW("v[114:115]", "%[b0]", "34816") RW("v[116:117]", "%[b0]", "36864")
#define R1_W0b RW("v[118:119]", "%[b0]", "38912") RX("v[80:83]", "%[a2]", "0") RX("v[84:87]", "%[a3]", "0")
#define R1_W1a RW("v[120:121]", "%[b1]", "32768") RW("v[122:123]", "%[b1]", "34816") RW("v[124:125]", "%[b1]", "36864")
#define R1_W1b RW("v[126:127]", "%[b1]", "38912") RX("v[88:91]", "%[a2]", "4096") RX("v[92:95]", "%[a3]", "4096")
#define R1_X2  RX("v[96:99]", "%[a2]", "8192") RX("v[100:103]", "%[a3]", "8192")
#define R1_X3  RX("v[104:107]", "%[a2]", "12288") RX("v[108:111]", "%[a3]", "12288")
#define CLOB8(a,b,c,d,e,f,g,h) "v" #a, "v" #b, "v" #c, "v" #d, "v" #e, "v" #f, "v" #g, "v" #h
#define FRAG_CLOBBERS CLOB8(28,29,30,31,32,33,34,35), CLOB8(36,37,38,39,40,41,42,43), CLOB8(44,45,46,47,48,49,50,51), CLOB8(52,53,54,55,56,57,58,59), \
    CLOB8(60,61,62,63,64,65,66,67), CLOB8(68,69,70,71,72,73,74,75), CLOB8(76,77,78,79,80,81,82,83), CLOB8(84,85,86,87,88,89,90,91), \
    CLOB8(92,93,94,95,96,97,98,99), CLOB8(100,101,102,103,104,105,106,107), CLOB8(108,109,110,111,112,113,114,115), CLOB8(116,117,118,119,120,121,122,123), \
    "v124", "v125", "v126", "v127"

#define OUT_BITS " sc1"
#define DSRC src
#define LEADP 1
#define NH 4
#define SLEEP0 10
#define LMX 0u
#define LM0 0u
#define LM1 0u
#define LM2 0u
#define LM3 LM2
#define HPRIO 0
#define CSLEEP
#define FIRSTMASK 0x7u
#define STR_(x) #x
#define STR(x) STR_(x)
__device__ __forceinline__ f32x4 act4(f32x4 a, float nsl, float is) {
    const f32x4 t = a * nsl;
    f32x4 d;
#pragma unroll
    for (int j = 0; j < 4; ++j) d[j] = __builtin_amdgcn_exp2f(__builtin_amdgcn_fmed3f(t[j], -86.5617024533378f, 86.5617024533378f));
    d = d * is + is;
#pragma unroll
    for (int j = 0; j < 4; ++j) d[j] = __builtin_amdgcn_rcpf(d[j]);
    return a * d;
}
__device__ __forceinline__ void st_out(f32x4 o, __amdgpu_buffer_rsrc_t ro, unsigned voff, unsigned soff) {
    asm volatile("buffer_store_dwordx4 %0, %1, %2, %3 offen" OUT_BITS "\n\ts_nop 1" :: "v"(o), "v"(voff), "s"(ro), "s"(soff) : "memory");
}

__device__ __forceinline__ bool poll_flags(const unsigned char* flagp, unsigned vf, unsigned mask, const unsigned char* inv, bool& inv_ok, GAS unsigned* tmo, unsigned call, unsigned& fmeta) {
    const unsigned long long t0 = __builtin_amdgcn_s_memrealtime();
#pragma nounroll
    for (unsigned sp = 0;; ++sp) {
        u32x2 f; unsigned iv;
        asm volatile("global_load_dwordx2 %0, %2, %3 sc0 sc1\n\tglobal_load_dword %1, %4, %5 sc1\n\ts_waitcnt vmcnt(0)" : "=&v"(f), "=&v"(iv) : "v"(vf), "s"(flagp), "v"(0u), "s"(inv) : "memory");
        fmeta = f.y;
        if (rfl(iv) >= call) inv_ok = true;
        if (inv_ok && ((unsigned)__ballot(f.x >= call) & mask) == mask) return true;
        if ((sp & 15) == 15) {
            const unsigned long long dt = __builtin_amdgcn_s_memrealtime() - t0;
            if (!inv_ok && dt > 2000) { asm volatile("buffer_inv sc0 sc1\n\ts_waitcnt vmcnt(0)" ::: "memory"); inv_ok = true; }
            if (rfl(__hip_atomic_load(tmo, __ATOMIC_RELAXED, __HIP_MEMORY_SCOPE_AGENT)) == call || dt > DEADLINE_TICKS) {
                __hip_atomic_store(tmo, call, __ATOMIC_RELAXED, __HIP_MEMORY_SCOPE_AGENT);
                return false;
            }
        }
    }
}

#define HELPER_ASM \
    "s_mov_b32 %[kp], m0\n\t" \
    "v_lshrrev_b32 v61, 3, %[vq]\n\t" \
    "v_mov_b32 v62, 0xffc\n\t" \
    "v_add_u32 v63, %[hq4], %[vs]\n\t" \
    "s_add_u32 %[l1], %[l0], 0xc000\n\t" \
    "s_add_u32 %[l2], %[l0], 0x18000\n\t" \
    "v_add_u32 v56, %[l0], %[vq]\n\t" \
    "v_add_u32 v57, %[l1], %[vq]\n\t" \
    "v_add_u32 v58, %[l2], %[vq]\n\t" \
    "v_add_u32 v48, 0x0, %[we]\n\t" \
    "v_add_u32 v49, 0x1c00, %[wo]\n\t" \
    "v_add_u32 v50, 0x3800, %[we]\n\t" \
    "v_add_u32 v51, 0x5400, %[wo]\n\t" \
    "v_add_u32 v52, 0x8000, %[we]\n\t" \
    "v_add_u32 v53, 0x9c00, %[wo]\n\t" \
    "v_add_u32 v54, 0xb800, %[we]\n\t" \
    "v_add_u32 v55, 0xd400, %[wo]\n\t" \
    "v_lshrrev_b32 v44, %[csh], %[fm]\n\t" \
    "v_and_b32 v60, 0x7f, %[fm]\n\t" \
    "v_and_b32 v44, 0xff, v44\n\t" \
    "v_lshl_or_b32 v60, v44, 8, v60\n\t" \
    "s_mov_b32 %[k1], 0\n\t" \
    "s_nop 0\n\t" \
    "v_readlane_b32 %[m0], v60, %[k1]\n\t" \
    "s_nop 0\n\t" \
    "s_cmp_gt_u32 %[m0], 0x40ff\n\t" \
    "s_cselect_b32 %[t1], 0x80, 0\n\t" \
    "s_add_u32 %[t1], %[t1], %[sa]\n\t" \
    "buffer_load_dwordx4 v[64:67], %[vq], %[ra], %[sa] offen\n\t" \
    "buffer_load_dwordx4 v[68:71], %[vq], %[ra], %[sa] offen offset:1024\n\t" \
    "buffer_load_dwordx4 v[72:75], %[vq], %[ra], %[sa] offen offset:2048\n\t" \
    "buffer_load_ushort v76, v61, %[ra], %[sa] offen offset:3072\n\t" \
    "buffer_load_ushort v77, v61, %[ra], %[t1] offen offset:3072\n\t" \
    "s_add_u32 %[sa], %[sa], 0xd00\n\t" \
    "s_mov_b32 %[k1], 1\n\t" \
    "v_and_b32 v45, 0x1f0, %[vq]\n\t" \
    "v_lshrrev_b32 v45, 1, v45\n\t" \
    "s_sleep " STR(SLEEP0) "\n\t" \
    "global_load_dwordx2 v[42:43], v45, %[fp] sc0 sc1\n\t" \
    "s_waitcnt vmcnt(0)\n\t" \
    "v_cmp_le_u32 vcc, %[call], v42\n\t" \
    "s_cmp_eq_u64 vcc, -1\n\t" \
    "s_cbranch_scc0 .Lend%=\n\t" \
    "v_lshrrev_b32 v44, %[csh], v43\n\t" \
    "v_and_b32 v60, 0x7f, v43\n\t" \
    "v_and_b32 v44, 0xff, v44\n\t" \
    "v_lshl_or_b32 v60, v44, 8, v60\n\t" \
    "s_nop 0\n\t" \
    ".Lloop%=:\n\t" \
    "s_waitcnt vmcnt(8) lgkmcnt(0)\n\t" \
    "v_readlane_b32 %[m1], v60, %[k1]\n\t" \
    "s_nop 0\n\t" \
    "s_cmp_gt_u32 %[m1], 0x40ff\n\t" \
    "s_cselect_b32 %[t1], 0x80, 0\n\t" \
    "s_add_u32 %[t1], %[t1], %[sa]\n\t" \
    "buffer_load_dwordx4 v[80:83], %[vq], %[ra], %[sa] offen\n\t" \
    "buffer_load_dwordx4 v[84:87], %[vq], %[ra], %[sa] offen offset:1024\n\t" \
    "buffer_load_dwordx4 v[88:91], %[vq], %[ra], %[sa] offen offset:2048\n\t" \
    "buffer_load_ushort v92, v61, %[ra], %[sa] offen offset:3072\n\t" \
    "buffer_load_ushort v93, v61, %[ra], %[t1] offen offset:3072\n\t" \
    "s_add_u32 %[sa], %[sa], 0xd00\n\t" \
    "ds_read_b128 v[40:43], %[vs] offset:16\n\t" \
    "s_cmp_gt_u32 %[m0], 0x80ff\n\t" \
    "s_cbranch_scc1 .LRa%=\n\t" \
    "s_and_b32 %[t0], %[m0], 0x7f\n\t" \
    "s_mul_i32 %[t0], %[t0], 0x1010101\n\t" \
    "s_add_u32 %[t1], %[t0], 0x90801000\n\t" \
    "v_and_b32 v112, %[m60], v64\n\t" \
    "v_and_b32 v116, %[m60], v68\n\t" \
    "v_and_b32 v120, %[m60], v72\n\t" \
    "v_and_b32 v113, %[m60], v65\n\t" \
    "v_and_b32 v117, %[m60], v69\n\t" \
    "v_and_b32 v121, %[m60], v73\n\t" \
    "v_and_b32 v114, %[m60], v66\n\t" \
    "v_and_b32 v118, %[m60], v70\n\t" \
    "v_and_b32 v122, %[m60], v74\n\t" \
    "v_and_b32 v115, %[m60], v67\n\t" \
    "v_and_b32 v119, %[m60], v71\n\t" \
    "v_and_b32 v123, %[m60], v75\n\t" \
    "v_xad_u32 v64, v64, v112, %[t0]\n\t" \
    "v_xad_u32 v65, v65, v113, %[t0]\n\t" \
    "v_xad_u32 v66, v66, v114, %[t0]\n\t" \
    "v_xad_u32 v67, v67, v115, %[t0]\n\t" \
    "ds_write_b128 v56, v[64:67]\n\t" \
    "v_xad_u32 v68, v68, v116, %[t0]\n\t" \
    "v_xad_u32 v69, v69, v117, %[t0]\n\t" \
    "v_xad_u32 v70, v70, v118, %[t0]\n\t" \
    "v_xad_u32 v71, v71, v119, %[t0]\n\t" \
    "ds_write_b128 v56, v[68:71] offset:1024\n\t" \
    "v_xad_u32 v72, v72, v120, %[t0]\n\t" \
    "v_xad_u32 v73, v73, v121, %[t0]\n\t" \
    "v_xad_u32 v74, v74, v122, %[t0]\n\t" \
    "v_xad_u32 v75, v75, v123, %[t0]\n\t" \
    "ds_write_b128 v56, v[72:75] offset:2048\n\t" \
    "v_lshrrev_b32 v120, 5, v120\n\t" \
    "v_lshrrev_b32 v121, 5, v121\n\t" \
    "v_lshrrev_b32 v122, 5, v122\n\t" \
    "v_lshrrev_b32 v123, 5, v123\n\t" \
    "v_lshrrev_b32 v112, 5, v112\n\t" \
    "v_lshrrev_b32 v113, 5, v113\n\t" \
    "v_lshrrev_b32 v114, 5, v114\n\t" \
    "v_lshrrev_b32 v115, 5, v115\n\t" \
    "v_perm_b32 v120, %[t1], %[t1], v120\n\t" \
    "v_perm_b32 v121, %[t1], %[t1], v121\n\t" \
    "v_perm_b32 v122, %[t1], %[t1], v122\n\t" \
    "v_perm_b32 v123, %[t1], %[t1], v123\n\t" \
    "v_lshrrev_b32 v116, 3, v116\n\t" \
    "v_lshrrev_b32 v117, 3, v117\n\t" \
    "v_lshrrev_b32 v118, 3, v118\n\t" \
    "v_lshrrev_b32 v119, 3, v119\n\t" \
    "v_add3_u32 v124, v112, v116, v120\n\t" \
    "v_add3_u32 v125, v113, v117, v121\n\t" \
    "v_add3_u32 v126, v114, v118, v122\n\t" \
    "v_add3_u32 v127, v115, v119, v123\n\t" \
    "ds_write_b128 v56, v[124:127] offset:3072\n\t" \
    "v_bfe_i32 v44, v76, 13, 3\n\t" \
    "v_lshl_add_u32 v45, v76, 3, 5\n\t" \
    "v_and_or_b32 v46, v76, v62, %[l0]\n\t" \
    "v_lshlrev_b32 v44, v45, v44\n\t" \
    "v_mov_b32 v47, %[k1]\n\t" \
    "ds_add_u32 v46, v44\n\t" \
    "s_cmp_gt_u32 %[m0], 0x40ff\n\t" \
    "s_cbranch_scc1 .LXa%=\n\t" \
    ".LXBa%=:\n\t" \
    "s_waitcnt lgkmcnt(5)\n\t" \
    ".LFCa%=:\n\t" \
    "v_min_u32 v40, v40, v41\n\t" \
    "v_min3_u32 v40, v40, v42, v43\n\t" \
    "s_nop 0\n\t" \
    "v_readfirstlane_b32 %[t0], v40\n\t" \
    "s_add_u32 %[t0], %[t0], 2\n\t" \
    "s_cmp_ge_u32 %[t0], %[k1]\n\t" \
    "s_cbranch_scc0 .LFa%=\n\t" \
    "s_add_u32 m0, %[lb], 49152\n\t" \
    "s_nop 0\n\t" \
    "buffer_load_dwordx4 v48, %[rw], %[sw] offen offset:0 lds\n\t" \
    "buffer_load_dwordx4 v49, %[rw], %[sw] offen offset:1024 lds\n\t" \
    "buffer_load_dwordx4 v50, %[rw], %[sw] offen offset:2048 lds\n\t" \
    "buffer_load_dwordx4 v51, %[rw], %[sw] offen offset:3072 lds\n\t" \
    "s_add_u32 m0, %[lb], 53248\n\t" \
    "s_nop 0\n\t" \
    "buffer_load_dwordx4 v52, %[rw], %[sw] offen offset:0 lds\n\t" \
    "buffer_load_dwordx4 v53, %[rw], %[sw] offen offset:1024 lds\n\t" \
    "buffer_load_dwordx4 v54, %[rw], %[sw] offen offset:2048 lds\n\t" \
    "buffer_load_dwordx4 v55, %[rw], %[sw] offen offset:3072 lds\n\t" \
    "s_add_u32 %[sw], %[sw], 0x40000\n\t" \
    "s_waitcnt vmcnt(13)\n\t" \
    "ds_write_b32 v63, v47\n\t" \
    "s_add_u32 %[k1], %[k1], 1\n\t" \
    "s_waitcnt vmcnt(8) lgkmcnt(0)\n\t" \
    "v_readlane_b32 %[m2], v60, %[k1]\n\t" \
    "s_nop 0\n\t" \
    "s_cmp_gt_u32 %[m2], 0x40ff\n\t" \
    "s_cselect_b32 %[t1], 0x80, 0\n\t" \
    "s_add_u32 %[t1], %[t1], %[sa]\n\t" \
    "buffer_load_dwordx4 v[96:99], %[vq], %[ra], %[sa] offen\n\t" \
    "buffer_load_dwordx4 v[100:103], %[vq], %[ra], %[sa] offen offset:1024\n\t" \
    "buffer_load_dwordx4 v[104:107], %[vq], %[ra], %[sa] offen offset:2048\n\t" \
    "buffer_load_ushort v108, v61, %[ra], %[sa] offen offset:3072\n\t" \
    "buffer_load_ushort v109, v61, %[ra], %[t1] offen offset:3072\n\t" \
    "s_add_u32 %[sa], %[sa], 0xd00\n\t" \
    "ds_read_b128 v[40:43], %[vs] offset:16\n\t" \
    "s_cmp_gt_u32 %[m1], 0x80ff\n\t" \
    "s_cbranch_scc1 .LRb%=\n\t" \
    "s_and_b32 %[t0], %[m1], 0x7f\n\t" \
    "s_mul_i32 %[t0], %[t0], 0x1010101\n\t" \
    "s_add_u32 %[t1], %[t0], 0x90801000\n\t" \
    "v_and_b32 v112, %[m60], v80\n\t" \
    "v_and_b32 v116, %[m60], v84\n\t" \
    "v_and_b32 v120, %[m60], v88\n\t" \
    "v_and_b32 v113, %[m60], v81\n\t" \
    "v_and_b32 v117, %[m60], v85\n\t" \
    "v_and_b32 v121, %[m60], v89\n\t" \
    "v_and_b32 v114, %[m60], v82\n\t" \
    "v_and_b32 v118, %[m60], v86\n\t" \
    "v_and_b32 v122, %[m60], v90\n\t" \
    "v_and_b32 v115, %[m60], v83\n\t" \
    "v_and_b32 v119, %[m60], v87\n\t" \
    "v_and_b32 v123, %[m60], v91\n\t" \
    "v_xad_u32 v80, v80, v112, %[t0]\n\t" \
    "v_xad_u32 v81, v81, v113, %[t0]\n\t" \
    "v_xad_u32 v82, v82, v114, %[t0]\n\t" \
    "v_xad_u32 v83, v83, v115, %[t0]\n\t" \
    "ds_write_b128 v57, v[80:83]\n\t" \
    "v_xad_u32 v84, v84, v116, %[t0]\n\t" \
    "v_xad_u32 v85, v85, v117, %[t0]\n\t" \
    "v_xad_u32 v86, v86, v118, %[t0]\n\t" \
    "v_xad_u32 v87, v87, v119, %[t0]\n\t" \
    "ds_write_b128 v57, v[84:87] offset:1024\n\t" \
    "v_xad_u32 v88, v88, v120, %[t0]\n\t" \
    "v_xad_u32 v89, v89, v121, %[t0]\n\t" \
    "v_xad_u32 v90, v90, v122, %[t0]\n\t" \
    "v_xad_u32 v91, v91, v123, %[t0]\n\t" \
    "ds_write_b128 v57, v[88:91] offset:2048\n\t" \
    "v_lshrrev_b32 v120, 5, v120\n\t" \
    "v_lshrrev_b32 v121, 5, v121\n\t" \
    "v_lshrrev_b32 v122, 5, v122\n\t" \
    "v_lshrrev_b32 v123, 5, v123\n\t" \
    "v_lshrrev_b32 v112, 5, v112\n\t" \
    "v_lshrrev_b32 v113, 5, v113\n\t" \
    "v_lshrrev_b32 v114, 5, v114\n\t" \
    "v_lshrrev_b32 v115, 5, v115\n\t" \
    "v_perm_b32 v120, %[t1], %[t1], v120\n\t" \
    "v_perm_b32 v121, %[t1], %[t1], v121\n\t" \
    "v_perm_b32 v122, %[t1], %[t1], v122\n\t" \
    "v_perm_b32 v123, %[t1], %[t1], v123\n\t" \
    "v_lshrrev_b32 v116, 3, v116\n\t" \
    "v_lshrrev_b32 v117, 3, v117\n\t" \
    "v_lshrrev_b32 v118, 3, v118\n\t" \
    "v_lshrrev_b32 v119, 3, v119\n\t" \
    "v_add3_u32 v124, v112, v116, v120\n\t" \
    "v_add3_u32 v125, v113, v117, v121\n\t" \
    "v_add3_u32 v126, v114, v118, v122\n\t" \
    "v_add3_u32 v127, v115, v119, v123\n\t" \
    "ds_write_b128 v57, v[124:127] offset:3072\n\t" \
    "v_bfe_i32 v44, v92, 13, 3\n\t" \
    "v_lshl_add_u32 v45, v92, 3, 5\n\t" \
    "v_and_or_b32 v46, v92, v62, %[l1]\n\t" \
    "v_lshlrev_b32 v44, v45, v44\n\t" \
    "v_mov_b32 v47, %[k1]\n\t" \
    "ds_add_u32 v46, v44\n\t" \
    "s_cmp_gt_u32 %[m1], 0x40ff\n\t" \
    "s_cbranch_scc1 .LXb%=\n\t" \
    ".LXBb%=:\n\t" \
    "s_waitcnt lgkmcnt(5)\n\t" \
    ".LFCb%=:\n\t" \
    "v_min_u32 v40, v40, v41\n\t" \
    "v_min3_u32 v40, v40, v42, v43\n\t" \
    "s_nop 0\n\t" \
    "v_readfirstlane_b32 %[t0], v40\n\t" \
    "s_add_u32 %[t0], %[t0], 2\n\t" \
    "s_cmp_ge_u32 %[t0], %[k1]\n\t" \
    "s_cbranch_scc0 .LFb%=\n\t" \
    "s_add_u32 m0, %[lb], 98304\n\t" \
    "s_nop 0\n\t" \
    "buffer_load_dwordx4 v48, %[rw], %[sw] offen offset:0 lds\n\t" \
    "buffer_load_dwordx4 v49, %[rw], %[sw] offen offset:1024 lds\n\t" \
    "buffer_load_dwordx4 v50, %[rw], %[sw] offen offset:2048 lds\n\t" \
    "buffer_load_dwordx4 v51, %[rw], %[sw] offen offset:3072 lds\n\t" \
    "s_add_u32 m0, %[lb], 102400\n\t" \
    "s_nop 0\n\t" \
    "buffer_load_dwordx4 v52, %[rw], %[sw] offen offset:0 lds\n\t" \
    "buffer_load_dwordx4 v53, %[rw], %[sw] offen offset:1024 lds\n\t" \
    "buffer_load_dwordx4 v54, %[rw], %[sw] offen offset:2048 lds\n\t" \
    "buffer_load_dwordx4 v55, %[rw], %[sw] offen offset:3072 lds\n\t" \
    "s_add_u32 %[sw], %[sw], 0x40000\n\t" \
    "s_waitcnt vmcnt(13)\n\t" \
    "ds_write_b32 v63, v47\n\t" \
    "s_add_u32 %[k1], %[k1], 1\n\t" \
    "s_waitcnt vmcnt(8) lgkmcnt(0)\n\t" \
    "v_readlane_b32 %[m0], v60, %[k1]\n\t" \
    "s_nop 0\n\t" \
    "s_cmp_gt_u32 %[m0], 0x40ff\n\t" \
    "s_cselect_b32 %[t1], 0x80, 0\n\t" \
    "s_add_u32 %[t1], %[t1], %[sa]\n\t" \
    "buffer_load_dwordx4 v[64:67], %[vq], %[ra], %[sa] offen\n\t" \
    "buffer_load_dwordx4 v[68:71], %[vq], %[ra], %[sa] offen offset:1024\n\t" \
    "buffer_load_dwordx4 v[72:75], %[vq], %[ra], %[sa] offen offset:2048\n\t" \
    "buffer_load_ushort v76, v61, %[ra], %[sa] offen offset:3072\n\t" \
    "buffer_load_ushort v77, v61, %[ra], %[t1] offen offset:3072\n\t" \
    "s_add_u32 %[sa], %[sa], 0xd00\n\t" \
    "ds_read_b128 v[40:43], %[vs] offset:16\n\t" \
    "s_cmp_gt_u32 %[m2], 0x80ff\n\t" \
    "s_cbranch_scc1 .LRc%=\n\t" \
    "s_and_b32 %[t0], %[m2], 0x7f\n\t" \
    "s_mul_i32 %[t0], %[t0], 0x1010101\n\t" \
    "s_add_u32 %[t1], %[t0], 0x90801000\n\t" \
    "v_and_b32 v112, %[m60], v96\n\t" \
    "v_and_b32 v116, %[m60], v100\n\t" \
    "v_and_b32 v120, %[m60], v104\n\t" \
    "v_and_b32 v113, %[m60], v97\n\t" \
    "v_and_b32 v117, %[m60], v101\n\t" \
    "v_and_b32 v121, %[m60], v105\n\t" \
    "v_and_b32 v114, %[m60], v98\n\t" \
    "v_and_b32 v118, %[m60], v102\n\t" \
    "v_and_b32 v122, %[m60], v106\n\t" \
    "v_and_b32 v115, %[m60], v99\n\t" \
    "v_and_b32 v119, %[m60], v103\n\t" \
    "v_and_b32 v123, %[m60], v107\n\t" \
    "v_xad_u32 v96, v96, v112, %[t0]\n\t" \
    "v_xad_u32 v97, v97, v113, %[t0]\n\t" \
    "v_xad_u32 v98, v98, v114, %[t0]\n\t" \
    "v_xad_u32 v99, v99, v115, %[t0]\n\t" \
    "ds_write_b128 v58, v[96:99]\n\t" \
    "v_xad_u32 v100, v100, v116, %[t0]\n\t" \
    "v_xad_u32 v101, v101, v117, %[t0]\n\t" \
    "v_xad_u32 v102, v102, v118, %[t0]\n\t" \
    "v_xad_u32 v103, v103, v119, %[t0]\n\t" \
    "ds_write_b128 v58, v[100:103] offset:1024\n\t" \
    "v_xad_u32 v104, v104, v120, %[t0]\n\t" \
    "v_xad_u32 v105, v105, v121, %[t0]\n\t" \
    "v_xad_u32 v106, v106, v122, %[t0]\n\t" \
    "v_xad_u32 v107, v107, v123, %[t0]\n\t" \
    "ds_write_b128 v58, v[104:107] offset:2048\n\t" \
    "v_lshrrev_b32 v120, 5, v120\n\t" \
    "v_lshrrev_b32 v121, 5, v121\n\t" \
    "v_lshrrev_b32 v122, 5, v122\n\t" \
    "v_lshrrev_b32 v123, 5, v123\n\t" \
    "v_lshrrev_b32 v112, 5, v112\n\t" \
    "v_lshrrev_b32 v113, 5, v113\n\t" \
    "v_lshrrev_b32 v114, 5, v114\n\t" \
    "v_lshrrev_b32 v115, 5, v115\n\t" \
    "v_perm_b32 v120, %[t1], %[t1], v120\n\t" \
    "v_perm_b32 v121, %[t1], %[t1], v121\n\t" \
    "v_perm_b32 v122, %[t1], %[t1], v122\n\t" \
    "v_perm_b32 v123, %[t1], %[t1], v123\n\t" \
    "v_lshrrev_b32 v116, 3, v116\n\t" \
    "v_lshrrev_b32 v117, 3, v117\n\t" \
    "v_lshrrev_b32 v118, 3, v118\n\t" \
    "v_lshrrev_b32 v119, 3, v119\n\t" \
    "v_add3_u32 v124, v112, v116, v120\n\t" \
    "v_add3_u32 v125, v113, v117, v121\n\t" \
    "v_add3_u32 v126, v114, v118, v122\n\t" \
    "v_add3_u32 v127, v115, v119, v123\n\t" \
    "ds_write_b128 v58, v[124:127] offset:3072\n\t" \
    "v_bfe_i32 v44, v108, 13, 3\n\t" \
    "v_lshl_add_u32 v45, v108, 3, 5\n\t" \
    "v_and_or_b32 v46, v108, v62, %[l2]\n\t" \
    "v_lshlrev_b32 v44, v45, v44\n\t" \
    "v_mov_b32 v47, %[k1]\n\t" \
    "ds_add_u32 v46, v44\n\t" \
    "s_cmp_gt_u32 %[m2], 0x40ff\n\t" \
    "s_cbranch_scc1 .LXc%=\n\t" \
    ".LXBc%=:\n\t" \
    "s_waitcnt lgkmcnt(5)\n\t" \
    ".LFCc%=:\n\t" \
    "v_min_u32 v40, v40, v41\n\t" \
    "v_min3_u32 v40, v40, v42, v43\n\t" \
    "s_nop 0\n\t" \
    "v_readfirstlane_b32 %[t0], v40\n\t" \
    "s_add_u32 %[t0], %[t0], 2\n\t" \
    "s_cmp_ge_u32 %[t0], %[k1]\n\t" \
    "s_cbranch_scc0 .LFc%=\n\t" \
    "s_add_u32 m0, %[lb], 0\n\t" \
    "s_nop 0\n\t" \
    "buffer_load_dwordx4 v48, %[rw], %[sw] offen offset:0 lds\n\t" \
    "buffer_load_dwordx4 v49, %[rw], %[sw] offen offset:1024 lds\n\t" \
    "buffer_load_dwordx4 v50, %[rw], %[sw] offen offset:2048 lds\n\t" \
    "buffer_load_dwordx4 v51, %[rw], %[sw] offen offset:3072 lds\n\t" \
    "s_add_u32 m0, %[lb], 4096\n\t" \
    "s_nop 0\n\t" \
    "buffer_load_dwordx4 v52, %[rw], %[sw] offen offset:0 lds\n\t" \
    "buffer_load_dwordx4 v53, %[rw], %[sw] offen offset:1024 lds\n\t" \
    "buffer_load_dwordx4 v54, %[rw], %[sw] offen offset:2048 lds\n\t" \
    "buffer_load_dwordx4 v55, %[rw], %[sw] offen offset:3072 lds\n\t" \
    "s_add_u32 %[sw], %[sw], 0x40000\n\t" \
    "s_waitcnt vmcnt(13)\n\t" \
    "ds_write_b32 v63, v47\n\t" \
    "s_add_u32 %[k1], %[k1], 1\n\t" \
    "s_cmp_lt_u32 %[k1], 31\n\t" \
    "s_cbranch_scc1 .Lloop%=\n\t" \
    "s_waitcnt vmcnt(8) lgkmcnt(0)\n\t" \
    "v_readlane_b32 %[m1], v60, %[k1]\n\t" \
    "s_nop 0\n\t" \
    "s_cmp_gt_u32 %[m1], 0x40ff\n\t" \
    "s_cselect_b32 %[t1], 0x80, 0\n\t" \
    "s_add_u32 %[t1], %[t1], %[sa]\n\t" \
    "buffer_load_dwordx4 v[80:83], %[vq], %[ra], %[sa] offen\n\t" \
    "buffer_load_dwordx4 v[84:87], %[vq], %[ra], %[sa] offen offset:1024\n\t" \
    "buffer_load_dwordx4 v[88:91], %[vq], %[ra], %[sa] offen offset:2048\n\t" \
    "buffer_load_ushort v92, v61, %[ra], %[sa] offen offset:3072\n\t" \
    "buffer_load_ushort v93, v61, %[ra], %[t1] offen offset:3072\n\t" \
    "s_add_u32 %[sa], %[sa], 0xd00\n\t" \
    "ds_read_b128 v[40:43], %[vs] offset:16\n\t" \
    "s_cmp_gt_u32 %[m0], 0x80ff\n\t" \
    "s_cbranch_scc1 .LRd%=\n\t" \
    "s_and_b32 %[t0], %[m0], 0x7f\n\t" \
    "s_mul_i32 %[t0], %[t0], 0x1010101\n\t" \
    "s_add_u32 %[t1], %[t0], 0x90801000\n\t" \
    "v_and_b32 v112, %[m60], v64\n\t" \
    "v_and_b32 v116, %[m60], v68\n\t" \
    "v_and_b32 v120, %[m60], v72\n\t" \
    "v_and_b32 v113, %[m60], v65\n\t" \
    "v_and_b32 v117, %[m60], v69\n\t" \
    "v_and_b32 v121, %[m60], v73\n\t" \
    "v_and_b32 v114, %[m60], v66\n\t" \
    "v_and_b32 v118, %[m60], v70\n\t" \
    "v_and_b32 v122, %[m60], v74\n\t" \
    "v_and_b32 v115, %[m60], v67\n\t" \
    "v_and_b32 v119, %[m60], v71\n\t" \
    "v_and_b32 v123, %[m60], v75\n\t" \
    "v_xad_u32 v64, v64, v112, %[t0]\n\t" \
    "v_xad_u32 v65, v65, v113, %[t0]\n\t" \
    "v_xad_u32 v66, v66, v114, %[t0]\n\t" \
    "v_xad_u32 v67, v67, v115, %[t0]\n\t" \
    "ds_write_b128 v56, v[64:67]\n\t" \
    "v_xad_u32 v68, v68, v116, %[t0]\n\t" \
    "v_xad_u32 v69, v69, v117, %[t0]\n\t" \
    "v_xad_u32 v70, v70, v118, %[t0]\n\t" \
    "v_xad_u32 v71, v71, v119, %[t0]\n\t" \
    "ds_write_b128 v56, v[68:71] offset:1024\n\t" \
    "v_xad_u32 v72, v72, v120, %[t0]\n\t" \
    "v_xad_u32 v73, v73, v121, %[t0]\n\t" \
    "v_xad_u32 v74, v74, v122, %[t0]\n\t" \
    "v_xad_u32 v75, v75, v123, %[t0]\n\t" \
    "ds_write_b128 v56, v[72:75] offset:2048\n\t" \
    "v_lshrrev_b32 v120, 5, v120\n\t" \
    "v_lshrrev_b32 v121, 5, v121\n\t" \
    "v_lshrrev_b32 v122, 5, v122\n\t" \
    "v_lshrrev_b32 v123, 5, v123\n\t" \
    "v_lshrrev_b32 v112, 5, v112\n\t" \
    "v_lshrrev_b32 v113, 5, v113\n\t" \
    "v_lshrrev_b32 v114, 5, v114\n\t" \
    "v_lshrrev_b32 v115, 5, v115\n\t" \
    "v_perm_b32 v120, %[t1], %[t1], v120\n\t" \
    "v_perm_b32 v121, %[t1], %[t1], v121\n\t" \
    "v_perm_b32 v122, %[t1], %[t1], v122\n\t" \
    "v_perm_b32 v123, %[t1], %[t1], v123\n\t" \
    "v_lshrrev_b32 v116, 3, v116\n\t" \
    "v_lshrrev_b32 v117, 3, v117\n\t" \
    "v_lshrrev_b32 v118, 3, v118\n\t" \
    "v_lshrrev_b32 v119, 3, v119\n\t" \
    "v_add3_u32 v124, v112, v116, v120\n\t" \
    "v_add3_u32 v125, v113, v117, v121\n\t" \
    "v_add3_u32 v126, v114, v118, v122\n\t" \
    "v_add3_u32 v127, v115, v119, v123\n\t" \
    "ds_write_b128 v56, v[124:127] offset:3072\n\t" \
    "v_bfe_i32 v44, v76, 13, 3\n\t" \
    "v_lshl_add_u32 v45, v76, 3, 5\n\t" \
    "v_and_or_b32 v46, v76, v62, %[l0]\n\t" \
    "v_lshlrev_b32 v44, v45, v44\n\t" \
    "v_mov_b32 v47, %[k1]\n\t" \
    "ds_add_u32 v46, v44\n\t" \
    "s_cmp_gt_u32 %[m0], 0x40ff\n\t" \
    "s_cbranch_scc1 .LXd%=\n\t" \
    ".LXBd%=:\n\t" \
    "s_waitcnt lgkmcnt(5)\n\t" \
    ".LFCd%=:\n\t" \
    "v_min_u32 v40, v40, v41\n\t" \
    "v_min3_u32 v40, v40, v42, v43\n\t" \
    "s_nop 0\n\t" \
    "v_readfirstlane_b32 %[t0], v40\n\t" \
    "s_add_u32 %[t0], %[t0], 2\n\t" \
    "s_cmp_ge_u32 %[t0], %[k1]\n\t" \
    "s_cbranch_scc0 .LFd%=\n\t" \
    "s_add_u32 m0, %[lb], 49152\n\t" \
    "s_nop 0\n\t" \
    "buffer_load_dwordx4 v48, %[rw], %[sw] offen offset:0 lds\n\t" \
    "buffer_load_dwordx4 v49, %[rw], %[sw] offen offset:1024 lds\n\t" \
    "buffer_load_dwordx4 v50, %[rw], %[sw] offen offset:2048 lds\n\t" \
    "buffer_load_dwordx4 v51, %[rw], %[sw] offen offset:3072 lds\n\t" \
    "s_add_u32 m0, %[lb], 53248\n\t" \
    "s_nop 0\n\t" \
    "buffer_load_dwordx4 v52, %[rw], %[sw] offen offset:0 lds\n\t" \
    "buffer_load_dwordx4 v53, %[rw], %[sw] offen offset:1024 lds\n\t" \
    "buffer_load_dwordx4 v54, %[rw], %[sw] offen offset:2048 lds\n\t" \
    "buffer_load_dwordx4 v55, %[rw], %[sw] offen offset:3072 lds\n\t" \
    "s_add_u32 %[sw], %[sw], 0x40000\n\t" \
    "s_waitcnt vmcnt(13)\n\t" \
    "ds_write_b32 v63, v47\n\t" \
    "s_add_u32 %[k1], %[k1], 1\n\t" \
    "s_waitcnt vmcnt(8) lgkmcnt(0)\n\t" \
    "s_cmp_gt_u32 %[m1], 0x80ff\n\t" \
    "s_cbranch_scc1 .LRe%=\n\t" \
    "s_and_b32 %[t0], %[m1], 0x7f\n\t" \
    "s_mul_i32 %[t0], %[t0], 0x1010101\n\t" \
    "s_add_u32 %[t1], %[t0], 0x90801000\n\t" \
    "v_and_b32 v112, %[m60], v80\n\t" \
    "v_and_b32 v116, %[m60], v84\n\t" \
    "v_and_b32 v120, %[m60], v88\n\t" \
    "v_and_b32 v113, %[m60], v81\n\t" \
    "v_and_b32 v117, %[m60], v85\n\t" \
    "v_and_b32 v121, %[m60], v89\n\t" \
    "v_and_b32 v114, %[m60], v82\n\t" \
    "v_and_b32 v118, %[m60], v86\n\t" \
    "v_and_b32 v122, %[m60], v90\n\t" \
    "v_and_b32 v115, %[m60], v83\n\t" \
    "v_and_b32 v119, %[m60], v87\n\t" \
    "v_and_b32 v123, %[m60], v91\n\t" \
    "v_xad_u32 v80, v80, v112, %[t0]\n\t" \
    "v_xad_u32 v81, v81, v113, %[t0]\n\t" \
    "v_xad_u32 v82, v82, v114, %[t0]\n\t" \
    "v_xad_u32 v83, v83, v115, %[t0]\n\t" \
    "ds_write_b128 v57, v[80:83]\n\t" \
    "v_xad_u32 v84, v84, v116, %[t0]\n\t" \
    "v_xad_u32 v85, v85, v117, %[t0]\n\t" \
    "v_xad_u32 v86, v86, v118, %[t0]\n\t" \
    "v_xad_u32 v87, v87, v119, %[t0]\n\t" \
    "ds_write_b128 v57, v[84:87] offset:1024\n\t" \
    "v_xad_u32 v88, v88, v120, %[t0]\n\t" \
    "v_xad_u32 v89, v89, v121, %[t0]\n\t" \
    "v_xad_u32 v90, v90, v122, %[t0]\n\t" \
    "v_xad_u32 v91, v91, v123, %[t0]\n\t" \
    "ds_write_b128 v57, v[88:91] offset:2048\n\t" \
    "v_lshrrev_b32 v120, 5, v120\n\t" \
    "v_lshrrev_b32 v121, 5, v121\n\t" \
    "v_lshrrev_b32 v122, 5, v122\n\t" \
    "v_lshrrev_b32 v123, 5, v123\n\t" \
    "v_lshrrev_b32 v112, 5, v112\n\t" \
    "v_lshrrev_b32 v113, 5, v113\n\t" \
    "v_lshrrev_b32 v114, 5, v114\n\t" \
    "v_lshrrev_b32 v115, 5, v115\n\t" \
    "v_perm_b32 v120, %[t1], %[t1], v120\n\t" \
    "v_perm_b32 v121, %[t1], %[t1], v121\n\t" \
    "v_perm_b32 v122, %[t1], %[t1], v122\n\t" \
    "v_perm_b32 v123, %[t1], %[t1], v123\n\t" \
    "v_lshrrev_b32 v116, 3, v116\n\t" \
    "v_lshrrev_b32 v117, 3, v117\n\t" \
    "v_lshrrev_b32 v118, 3, v118\n\t" \
    "v_lshrrev_b32 v119, 3, v119\n\t" \
    "v_add3_u32 v124, v112, v116, v120\n\t" \
    "v_add3_u32 v125, v113, v117, v121\n\t" \
    "v_add3_u32 v126, v114, v118, v122\n\t" \
    "v_add3_u32 v127, v115, v119, v123\n\t" \
    "ds_write_b128 v57, v[124:127] offset:3072\n\t" \
    "v_bfe_i32 v44, v92, 13, 3\n\t" \
    "v_lshl_add_u32 v45, v92, 3, 5\n\t" \
    "v_and_or_b32 v46, v92, v62, %[l1]\n\t" \
    "v_lshlrev_b32 v44, v45, v44\n\t" \
    "v_mov_b32 v47, %[k1]\n\t" \
    "ds_add_u32 v46, v44\n\t" \
    "s_cmp_gt_u32 %[m1], 0x40ff\n\t" \
    "s_cbranch_scc1 .LXe%=\n\t" \
    ".LXBe%=:\n\t" \
    "s_waitcnt vmcnt(0)\n\t" \
    "ds_write_b32 v63, v47\n\t" \
    "s_add_u32 %[k1], %[k1], 1\n\t" \
    "s_branch .Lend%=\n\t" \
    ".LXa%=:\n\t" \
    "v_bfe_i32 v44, v77, 13, 3\n\t" \
    "v_lshl_add_u32 v45, v77, 3, 5\n\t" \
    "v_and_or_b32 v46, v77, v62, %[l0]\n\t" \
    "v_lshlrev_b32 v44, v45, v44\n\t" \
    "ds_add_u32 v46, v44\n\t" \
    "s_branch .LXBa%=\n\t" \
    ".LRa%=:\n\t" \
    "s_lshl_b32 %[t0], %[k1], 12\n\t" \
    "s_add_u32 %[t0], %[t0], %[sr]\n\t" \
    "s_mov_b32 m0, %[l0]\n\t" \
    "s_nop 0\n\t" \
    "buffer_load_dwordx4 %[vq], %[ra], %[t0] offen offset:0 lds\n\t" \
    "buffer_load_dwordx4 %[vq], %[ra], %[t0] offen offset:1024 lds\n\t" \
    "buffer_load_dwordx4 %[vq], %[ra], %[t0] offen offset:2048 lds\n\t" \
    "buffer_load_dwordx4 %[vq], %[ra], %[t0] offen offset:3072 lds\n\t" \
    "v_mov_b32 v47, %[k1]\n\t" \
    "s_waitcnt vmcnt(0) lgkmcnt(0)\n\t" \
    "s_branch .LXBa%=\n\t" \
    ".LFa%=:\n\t" \
    "ds_read_b128 v[40:43], %[vs] offset:16\n\t" \
    "s_sub_u32 %[sp], %[sp], 1\n\t" \
    "s_cbranch_scc1 .Lend%=\n\t" \
    "s_waitcnt lgkmcnt(0)\n\t" \
    "s_branch .LFCa%=\n\t" \
    ".LXb%=:\n\t" \
    "v_bfe_i32 v44, v93, 13, 3\n\t" \
    "v_lshl_add_u32 v45, v93, 3, 5\n\t" \
    "v_and_or_b32 v46, v93, v62, %[l1]\n\t" \
    "v_lshlrev_b32 v44, v45, v44\n\t" \
    "ds_add_u32 v46, v44\n\t" \
    "s_branch .LXBb%=\n\t" \
    ".LRb%=:\n\t" \
    "s_lshl_b32 %[t0], %[k1], 12\n\t" \
    "s_add_u32 %[t0], %[t0], %[sr]\n\t" \
    "s_mov_b32 m0, %[l1]\n\t" \
    "s_nop 0\n\t" \
    "buffer_load_dwordx4 %[vq], %[ra], %[t0] offen offset:0 lds\n\t" \
    "buffer_load_dwordx4 %[vq], %[ra], %[t0] offen offset:1024 lds\n\t" \
    "buffer_load_dwordx4 %[vq], %[ra], %[t0] offen offset:2048 lds\n\t" \
    "buffer_load_dwordx4 %[vq], %[ra], %[t0] offen offset:3072 lds\n\t" \
    "v_mov_b32 v47, %[k1]\n\t" \
    "s_waitcnt vmcnt(0) lgkmcnt(0)\n\t" \
    "s_branch .LXBb%=\n\t" \
    ".LFb%=:\n\t" \
    "ds_read_b128 v[40:43], %[vs] offset:16\n\t" \
    "s_sub_u32 %[sp], %[sp], 1\n\t" \
    "s_cbranch_scc1 .Lend%=\n\t" \
    "s_waitcnt lgkmcnt(0)\n\t" \
    "s_branch .LFCb%=\n\t" \
    ".LXc%=:\n\t" \
    "v_bfe_i32 v44, v109, 13, 3\n\t" \
    "v_lshl_add_u32 v45, v109, 3, 5\n\t" \
    "v_and_or_b32 v46, v109, v62, %[l2]\n\t" \
    "v_lshlrev_b32 v44, v45, v44\n\t" \
    "ds_add_u32 v46, v44\n\t" \
    "s_branch .LXBc%=\n\t" \
    ".LRc%=:\n\t" \
    "s_lshl_b32 %[t0], %[k1], 12\n\t" \
    "s_add_u32 %[t0], %[t0], %[sr]\n\t" \
    "s_mov_b32 m0, %[l2]\n\t" \
    "s_nop 0\n\t" \
    "buffer_load_dwordx4 %[vq], %[ra], %[t0] offen offset:0 lds\n\t" \
    "buffer_load_dwordx4 %[vq], %[ra], %[t0] offen offset:1024 lds\n\t" \
    "buffer_load_dwordx4 %[vq], %[ra], %[t0] offen offset:2048 lds\n\t" \
    "buffer_load_dwordx4 %[vq], %[ra], %[t0] offen offset:3072 lds\n\t" \
    "v_mov_b32 v47, %[k1]\n\t" \
    "s_waitcnt vmcnt(0) lgkmcnt(0)\n\t" \
    "s_branch .LXBc%=\n\t" \
    ".LFc%=:\n\t" \
    "ds_read_b128 v[40:43], %[vs] offset:16\n\t" \
    "s_sub_u32 %[sp], %[sp], 1\n\t" \
    "s_cbranch_scc1 .Lend%=\n\t" \
    "s_waitcnt lgkmcnt(0)\n\t" \
    "s_branch .LFCc%=\n\t" \
    ".LXd%=:\n\t" \
    "v_bfe_i32 v44, v77, 13, 3\n\t" \
    "v_lshl_add_u32 v45, v77, 3, 5\n\t" \
    "v_and_or_b32 v46, v77, v62, %[l0]\n\t" \
    "v_lshlrev_b32 v44, v45, v44\n\t" \
    "ds_add_u32 v46, v44\n\t" \
    "s_branch .LXBd%=\n\t" \
    ".LRd%=:\n\t" \
    "s_lshl_b32 %[t0], %[k1], 12\n\t" \
    "s_add_u32 %[t0], %[t0], %[sr]\n\t" \
    "s_mov_b32 m0, %[l0]\n\t" \
    "s_nop 0\n\t" \
    "buffer_load_dwordx4 %[vq], %[ra], %[t0] offen offset:0 lds\n\t" \
    "buffer_load_dwordx4 %[vq], %[ra], %[t0] offen offset:1024 lds\n\t" \
    "buffer_load_dwordx4 %[vq], %[ra], %[t0] offen offset:2048 lds\n\t" \
    "buffer_load_dwordx4 %[vq], %[ra], %[t0] offen offset:3072 lds\n\t" \
    "v_mov_b32 v47, %[k1]\n\t" \
    "s_waitcnt vmcnt(0) lgkmcnt(0)\n\t" \
    "s_branch .LXBd%=\n\t" \
    ".LFd%=:\n\t" \
    "ds_read_b128 v[40:43], %[vs] offset:16\n\t" \
    "s_sub_u32 %[sp], %[sp], 1\n\t" \
    "s_cbranch_scc1 .Lend%=\n\t" \
    "s_waitcnt lgkmcnt(0)\n\t" \
    "s_branch .LFCd%=\n\t" \
    ".LXe%=:\n\t" \
    "v_bfe_i32 v44, v93, 13, 3\n\t" \
    "v_lshl_add_u32 v45, v93, 3, 5\n\t" \
    "v_and_or_b32 v46, v93, v62, %[l1]\n\t" \
    "v_lshlrev_b32 v44, v45, v44\n\t" \
    "ds_add_u32 v46, v44\n\t" \
    "s_branch .LXBe%=\n\t" \
    ".LRe%=:\n\t" \
    "s_lshl_b32 %[t0], %[k1], 12\n\t" \
    "s_add_u32 %[t0], %[t0], %[sr]\n\t" \
    "s_mov_b32 m0, %[l1]\n\t" \
    "s_nop 0\n\t" \
    "buffer_load_dwordx4 %[vq], %[ra], %[t0] offen offset:0 lds\n\t" \
    "buffer_load_dwordx4 %[vq], %[ra], %[t0] offen offset:1024 lds\n\t" \
    "buffer_load_dwordx4 %[vq], %[ra], %[t0] offen offset:2048 lds\n\t" \
    "buffer_load_dwordx4 %[vq], %[ra], %[t0] offen offset:3072 lds\n\t" \
    "v_mov_b32 v47, %[k1]\n\t" \
    "s_waitcnt vmcnt(0) lgkmcnt(0)\n\t" \
    "s_branch .LXBe%=\n\t" \
    ".Lend%=:\n\t" \
    "s_waitcnt vmcnt(0) lgkmcnt(0)\n\t" \
    "s_mov_b32 m0, %[kp]\n\t" \
    ""
#define HELPER_CLOBBERS "v40", "v41", "v42", "v43", "v44", "v45", "v46", "v47", "v48", "v49", "v50", "v51", "v52", "v53", "v54", "v55", "v56", "v57", "v58", "v59", "v60", "v61", "v62", "v63", "v64", "v65", "v66", "v67", "v68", "v69", "v70", "v71", "v72", "v73", "v74", "v75", "v76", "v77", "v78", "v79", "v80", "v81", "v82", "v83", "v84", "v85", "v86", "v87", "v88", "v89", "v90", "v91", "v92", "v93", "v94", "v95", "v96", "v97", "v98", "v99", "v100", "v101", "v102", "v103", "v104", "v105", "v106", "v107", "v108", "v109", "v110", "v111", "v112", "v113", "v114", "v115", "v116", "v117", "v118", "v119", "v120", "v121", "v122", "v123", "v124", "v125", "v126", "v127"

__global__ void __launch_bounds__(512, 2) ag_gemm_kernel(Params p) {
    __shared__ __attribute__((aligned(1024))) unsigned char smem[LDS_BYTES];
    const unsigned lds0 = (unsigned)(size_t)(LAS unsigned char*)smem;
    const unsigned sync = lds0 + SYNC_OFF;
    const unsigned lane = threadIdx.x & 63, wid = rfl(threadIdx.x >> 6), bid = blockIdx.x;
    const unsigned rank = p.rank & 7;
    const unsigned src = ((((0xb2u >> rank) & 1) ? 0x63207541u : 0x75416320u) >> (4 * (bid & 7))) & 7, rb = (bid >> 3) & 3, cb = bid >> 5;
    unsigned char* const myws = p.ws[rank];
    const unsigned cks = ((bid >> 3) + 31) & 31, crb = (bid >> 1) & 3, chf = bid & 1;
    GAS unsigned* const cntp = (GAS unsigned*)(myws + WS_CNT + bid * 64);
    unsigned char* const invp = myws + WS_TMO + 128 + (__builtin_amdgcn_s_getreg((3 << 11) | 20) & 15u) * 128;
    const __amdgpu_buffer_rsrc_t rx = __builtin_amdgcn_make_buffer_rsrc((void*)p.x, 0, (unsigned)SHARD, 0x00020000);
    const unsigned xso = (crb * BM + chf * 64) * K_ + cks * BK;
    unsigned call, Kb;
    {
        if (bid < 8 && wid == 1) asm volatile("buffer_inv sc0 sc1" ::: "memory");
        const unsigned q2 = wid >> 2, sl = wid & 3;
        const unsigned cv = __hip_atomic_load(cntp, __ATOMIC_RELAXED, __HIP_MEMORY_SCOPE_AGENT);
        const u32x4 smp = __builtin_amdgcn_raw_buffer_load_b128(rx, lane * K_ + (lane & 7) * 16, xso, 0);
        unsigned x[4];
#pragma unroll
        for (int i = 0; i < 4; ++i)
            x[i] = __builtin_amdgcn_raw_buffer_load_b32(rx, (2 * sl + (lane >> 5)) * K_ + ((((lane >> 2) & 7) ^ (4 * (i & 1) + sl)) << 4) + 4 * (lane & 3), xso + (32 * q2 + 8 * i) * K_, 0);
        if (threadIdx.x < 144) lds_st(lds0 + LIST_OFF + 4 * threadIdx.x, threadIdx.x < 128 ? lane * 0x80008u + 0x40000u : 0u);
        asm volatile("s_waitcnt lgkmcnt(0)\n\ts_barrier" ::: "memory");
        call = rfl(cv) + 1u;
        unsigned sum = 0;
#pragma unroll
        for (int i = 0; i < 4; ++i) sum = __builtin_amdgcn_sad_u8(smp[i] & 0x7f7f7f7fu, 0u, sum);
        const int kq = ((int)wave_sum(sum) - 20582) >> 10;
        Kb = kq < 0 ? 0u : kq > 96 ? 96u : (unsigned)kq;
        const unsigned KK = Kb * 0x01010101u;
        unsigned e[4], c[4], M = 0;
#pragma unroll
        for (int i = 0; i < 4; ++i) {
            const unsigned t = (x[i] | 0x80808080u) - KK;
            e[i] = (t ^ 0x80808080u) & 0xe0e0e0e0u;
            c[i] = bfi(0x1f1f1f1fu, t, x[i]);
            M |= (((e[i] >> 1) + 0x70707070u) & 0x80808080u) >> (7 - i);
        }
        LAS unsigned* enc = (LAS unsigned*)((LAS unsigned char*)smem + ENC_OFF + q2 * 3072 + sl * 256) + lane;
        enc[0] = bfi(0x60606060u, c[3] << 5, c[0]);
        enc[256] = bfi(0x60606060u, c[3] << 3, c[1]);
        enc[512] = bfi(0x20202020u, c[3] << 1, bfi(0x40404040u, c[3] >> 1, c[2]));
        const unsigned tot = wave_sum(__builtin_popcount(M));
        unsigned idx;
        asm volatile("s_mov_b64 exec, 1\n\tds_add_rtn_u32 %0, %1, %2\n\ts_mov_b64 exec, -1\n\ts_waitcnt lgkmcnt(0)" : "=&v"(idx) : "v"(sync + 36 + 4 * q2), "v"(tot) : "memory");
        idx = rfl(idx);
        LAS unsigned short* list = (LAS unsigned short*)((LAS unsigned char*)smem + LIST_OFF + 256 * q2);
#pragma nounroll
        for (;;) {
            const unsigned long long act = __ballot(M != 0);
            if (act == 0ull) break;
            if (M != 0) {
                const unsigned bit = __builtin_ctz(M), i = bit & 3, by = bit >> 3;
                const unsigned ei = i == 0 ? e[0] : i == 1 ? e[1] : i == 2 ? e[2] : e[3];
                const unsigned my = idx + __builtin_amdgcn_mbcnt_hi((unsigned)(act >> 32), __builtin_amdgcn_mbcnt_lo((unsigned)act, 0u));
                if (my < EXC_CAP) list[my] = (unsigned short)((i * 1024 + sl * 256 + lane * 4 + by) | (((ei >> (8 * by + 5)) & 7) << 13));
                M &= M - 1;
            }
            idx += __builtin_popcountll(act);
        }
        if (bid < 8 && wid == 1) {
            asm volatile("s_waitcnt vmcnt(0)" ::: "memory");
            if (lane == 0) __hip_atomic_store((GAS unsigned*)invp, call, __ATOMIC_RELAXED, __HIP_MEMORY_SCOPE_AGENT);
        }
        asm volatile("s_waitcnt lgkmcnt(0)\n\ts_barrier" ::: "memory");
    }
    if (wid == 0) {
        STAMP(st0, 10);
        const u32x4 cw = lds_ld4(sync + 32);
        const unsigned c0 = min(rfl(cw.y), 255u), c1 = min(rfl(cw.z), 255u), hu = (crb * 2 + chf) * 32 + cks, qu = (crb * 4 + chf * 2) * 32 + cks;
        const unsigned meta = Kb | (c0 << 8) | (c1 << 16);
        {
            const LAS u32x4* enc = (const LAS u32x4*)((LAS unsigned char*)smem + ENC_OFF) + lane;
            const u32x4 d0 = enc[0], d1 = enc[64], d2 = enc[128], d3 = enc[192], d4 = enc[256], d5 = enc[320];
            const u32x4 d6 = *((const LAS u32x4*)((LAS unsigned char*)smem + LIST_OFF) + (lane & 31));
            unsigned char* dst = myws + WS_DATA + (call & 1) * PACK_PAR + qu * QU;
            asm volatile("global_store_dwordx4 %7, %0, %10 sc0 sc1\n\tglobal_store_dwordx4 %7, %1, %10 offset:1024 sc0 sc1\n\tglobal_store_dwordx4 %7, %2, %10 offset:2048 sc0 sc1\n\t"
                         "global_store_dwordx4 %8, %3, %10 sc0 sc1\n\tglobal_store_dwordx4 %8, %4, %10 offset:1024 sc0 sc1\n\tglobal_store_dwordx4 %8, %5, %10 offset:2048 sc0 sc1\n\t"
                         "s_mov_b64 exec, 0xffffffff\n\tglobal_store_dwordx4 %9, %6, %10 offset:3072 sc0 sc1\n\ts_mov_b64 exec, -1\n\ts_nop 1"
                         :: "v"(d0), "v"(d1), "v"(d2), "v"(d3), "v"(d4), "v"(d5), "v"(d6), "v"(lane * 16), "v"(lane * 16 + 32 * QU), "v"((lane & 15) * 16 + ((lane >> 4) & 1) * 32 * QU), "s"(dst) : "memory");
        }
        if (c0 > EXC_CAP || c1 > EXC_CAP) {
            unsigned char* dst = myws + WS_DATA + RAW_OFF + (call & 1) * (unsigned)SHARD + qu * 4096;
#pragma nounroll
            for (unsigned i = 0; i < 8; ++i) {
                const u32x4 d = __builtin_amdgcn_raw_buffer_load_b128(rx, (lane >> 3) * K_ + (((lane & 7) ^ (4 * (i & 1) + (lane >> 4))) << 4), xso + 8 * i * K_, 0);
                asm volatile("global_store_dwordx4 %1, %0, %2 sc0 sc1\n\ts_nop 1" :: "v"(d), "v"(lane * 16 + (i & 3) * 1024 + (i >> 2) * 32 * 4096), "s"(dst) : "memory");
            }
        }
        asm volatile("s_waitcnt vmcnt(0)" ::: "memory");
        const u32x2 fv = {call, meta};
#pragma unroll
        for (int i = 1; i <= NR; ++i) {
            unsigned char* fp = p.ws[(rank + i) & 7] + WS_FLAG + (((call & 1) * NR + rank) * 256 + hu) * 8;
            asm volatile("s_mov_b64 exec, 1\n\tglobal_store_dwordx2 %0, %1, %2 sc0 sc1\n\ts_mov_b64 exec, -1" :: "v"(0u), "v"(fv), "s"(fp) : "memory");
        }
        if (lane == 0) __hip_atomic_store(cntp, call, __ATOMIC_RELAXED, __HIP_MEMORY_SCOPE_AGENT);
        STAMP(st0, 11);
    }

    float scale;
    const __amdgpu_buffer_rsrc_t ro = __builtin_amdgcn_make_buffer_rsrc((void*)p.out, 0, (unsigned)(M_ * NL * 4), 0x00020000);
    const unsigned osoff = ((src * MS + rb * BM) * NL + cb * BN + (wid & 3) * 64) * 4;
    if (wid < 4) {
        scale = p.sx[0] * p.sw[0];
        asm volatile("" : "+v"(scale));
        const unsigned r = lane & 31, h = lane >> 5, key = (r >> 1) & 7;
        unsigned a0 = lds0 + r * 128 + (((2 * h + 0) ^ key) << 4), a1 = lds0 + r * 128 + (((2 * h + 1) ^ key) << 4);
        unsigned a2 = lds0 + r * 128 + (((4 + 2 * h + 0) ^ key) << 4), a3 = lds0 + r * 128 + (((4 + 2 * h + 1) ^ key) << 4);
        const unsigned ch = (lane >> 4) & 1, q = (lane & 15) >> 1, pp = lane & 1;
        unsigned b0 = lds0 + (32 * h + q) * 256 + (((4 * wid + 0 + ch) ^ (2 * q)) << 4) + 8 * pp;
        unsigned b1 = lds0 + (32 * h + q) * 256 + (((4 * wid + 2 + ch) ^ (2 * q)) << 4) + 8 * pp;
        f32x16 acc[8];
#pragma unroll
        for (int i = 0; i < 8; ++i)
#pragma unroll
            for (int j = 0; j < 16; ++j) acc[i][j] = 0.f;
        unsigned kk = 0, slot = 0, dl, tmp, spin = 0;
        const unsigned vsync = sync, vcons = sync + 16 + 4 * wid;
        asm volatile(
            "s_nop 1\n\t"
            "s_branch .Lspin%=\n"
            ".Lloop%=:\n\t"
            "s_waitcnt lgkmcnt(0)\n\t"
            MF("%0", "v[64:71]", "v[32:39]") R1_W0a
            MF("%1", "v[72:79]", "v[32:39]") R1_W0b
            MF("%2", "v[64:71]", "v[40:47]") R1_W1a
            MF("%3", "v[72:79]", "v[40:47]") R1_W1b
            MF("%4", "v[64:71]", "v[48:55]") R1_X2
            "s_add_u32 %[kk], %[kk], 1\n\t"
            "v_mov_b32 v28, %[kk]\n\t"
            MF("%5", "v[72:79]", "v[48:55]") R1_X3
            "ds_write_b32 %[vc], v28\n\t"
            "s_add_u32 %[sl], %[sl], 1\n\t"
            "s_cmp_eq_u32 %[sl], 3\n\t"
            "s_cselect_b32 %[dl], %[dn], %[dp]\n\t"
            "s_cselect_b32 %[sl], 0, %[sl]\n\t"
            MF("%6", "v[64:71]", "v[56:63]")
            "ds_read_b128 v[28:31], %[vs]\n\t"
            "v_add_u32 %[a0], %[dl], %[a0]\n\tv_add_u32 %[a1], %[dl], %[a1]\n\tv_add_u32 %[a2], %[dl], %[a2]\n\t"
            "v_add_u32 %[a3], %[dl], %[a3]\n\tv_add_u32 %[b0], %[dl], %[b0]\n\tv_add_u32 %[b1], %[dl], %[b1]\n\t"
            MF("%7", "v[72:79]", "v[56:63]")
            "s_waitcnt lgkmcnt(0)\n\t"
            MF("%0", "v[112:119]", "v[80:87]")
            "v_min3_u32 v28, v28, v29, v30\n\t"
            "v_min_u32 v28, v28, v31\n\t"
            "s_nop 0\n\t"
            "v_readfirstlane_b32 %[tm], v28\n\t"
            "s_nop 0\n\t"
            "s_cmp_gt_u32 %[tm], %[kk]\n\t"
            "s_cbranch_scc0 .Lslow%=\n\t"
            R0_W0a
            MF("%1", "v[120:127]", "v[80:87]") R0_W0b
            MF("%2", "v[112:119]", "v[88:95]") R0_W1a
            MF("%3", "v[120:127]", "v[88:95]") R0_W1b
            MF("%4", "v[112:119]", "v[96:103]") R0_X2
            MF("%5", "v[120:127]", "v[96:103]") R0_X3
            MF("%6", "v[112:119]", "v[104:111]")
            MF("%7", "v[120:127]", "v[104:111]")
            "s_branch .Lloop%=\n"
            ".Lslow%=:\n\t"
            MF("%1", "v[120:127]", "v[80:87]")
            MF("%2", "v[112:119]", "v[88:95]")
            MF("%3", "v[120:127]", "v[88:95]")
            MF("%4", "v[112:119]", "v[96:103]")
            MF("%5", "v[120:127]", "v[96:103]")
            MF("%6", "v[112:119]", "v[104:111]")
            MF("%7", "v[120:127]", "v[104:111]")
            "s_cmp_ge_u32 %[kk], 32\n\t"
            "s_cbranch_scc1 .Ldone%=\n"
            ".Lspin%=:\n\t"
            CSLEEP
            "ds_read_b128 v[28:31], %[vs]\n\t"
            "s_add_u32 %[sp], %[sp], 1\n\t"
            "s_waitcnt lgkmcnt(0)\n\t"
            "v_min3_u32 v28, v28, v29, v30\n\t"
            "v_min_u32 v28, v28, v31\n\t"
            "s_nop 0\n\t"
            "v_readfirstlane_b32 %[tm], v28\n\t"
            "s_cmp_gt_u32 %[sp], 0x2000000\n\t"
            "s_cbranch_scc1 .Ldone%=\n\t"
            "s_cmp_gt_u32 %[tm], %[kk]\n\t"
            "s_cbranch_scc0 .Lspin%=\n\t"
            R0_W0a R0_W0b R0_W1a R0_W1b R0_X2 R0_X3
            "s_branch .Lloop%=\n"
            ".Ldone%=:\n\t"
            "ds_read_b128 v[28:31], %[vs] offset:16\n\t"
            "ds_read_b32 v32, %[vs] offset:32\n\t"
            "s_nop 15\n\t"
            "s_waitcnt lgkmcnt(0)\n\t"
            "v_min3_u32 v28, v28, v29, v30\n\t"
            "v_min_u32 v28, v28, v31\n\t"
            "v_readfirstlane_b32 %[dl], v32\n\t"
            "v_readfirstlane_b32 %[tm], v28\n\t"
            "s_nop 3"
            : "+a"(acc[0]), "+a"(acc[1]), "+a"(acc[2]), "+a"(acc[3]), "+a"(acc[4]), "+a"(acc[5]), "+a"(acc[6]), "+a"(acc[7]),
              [a0] "+v"(a0), [a1] "+v"(a1), [a2] "+v"(a2), [a3] "+v"(a3), [b0] "+v"(b0), [b1] "+v"(b1),
              [kk] "+s"(kk), [sl] "+s"(slot), [dl] "=&s"(dl), [tm] "=&s"(tmp), [sp] "+s"(spin)
            : [vs] "v"(vsync), [vc] "v"(vcons), [dp] "s"(STAGE), [dn] "s"(0u - (NSTAGE - 1) * STAGE)
            : "memory", "scc", FRAG_CLOBBERS);
        STAMP(st0, 0);
        if (kk != (unsigned)NK) { lds_st(sync + 32, 1u); dl = 1u; }
        if (tmp < (unsigned)NK) wait_free(sync, NK);
        float is = __builtin_amdgcn_rcpf(scale);
        if (dl != 0u) is = __builtin_nanf("");
        const float nsl = scale * -1.44269504088896340736f;
        unsigned ln = lane; asm volatile("" : "+v"(ln));
        const unsigned ovoff = (ln >> 3) * (NL * 4) + (ln & 7) * 16;
        LAS unsigned char* wr = (LAS unsigned char*)smem + wid * 32768 + r * 128;
        const LAS unsigned char* rd = (const LAS unsigned char*)smem + wid * 32768 + (lane >> 3) * 128 + (((lane & 7) ^ (lane >> 3)) << 4);
        auto put = [&](int sb, int g, f32x4 v) { *(LAS f32x4*)(wr + sb * 4096 + (((2 * g + h) ^ (r & 7)) << 4)) = v; };
        auto quad = [&](int sb, int g) { const f32x16& a = acc[sb]; return (f32x4){a[4 * g], a[4 * g + 1], a[4 * g + 2], a[4 * g + 3]}; };
        auto dump = [&](int sb) {
#pragma unroll
            for (int g = 0; g < 4; ++g) put(sb, g, quad(sb, g));
            lds_st(sync + 48 + 4 * wid, sb + 1);
        };
        auto flush = [&](int sb) {
            f32x4 v[4];
#pragma unroll
            for (int i = 0; i < 4; ++i) v[i] = *(const LAS f32x4*)(rd + sb * 4096 + i * 1024);
#pragma unroll
            for (int i = 0; i < 4; ++i) st_out(v[i], ro, ovoff, osoff + ((sb >> 1) * 32 + 8 * i) * (NL * 4) + (sb & 1) * 128);
        };
        dump(0);
        asm volatile("" : "+a"(acc[4]), "+a"(acc[5]), "+a"(acc[6]), "+a"(acc[7]));
        if (2 * NH > 8) dump(1);
#pragma unroll
        for (int sb = NH; sb < 8; ++sb) {
            const int nx = sb - NH + (2 * NH > 8 ? 2 : 1);
            if (nx < NH) dump(nx);
#pragma unroll
            for (int g = 0; g < 4; ++g) put(sb, g, act4(quad(sb, g), nsl, is));
            if (sb > NH) flush(sb - 1);
        }
        flush(7);
    } else {
        __builtin_amdgcn_s_setprio(HPRIO);
        const unsigned hq = wid - 4;
        GAS unsigned* tmo = (GAS unsigned*)(myws + WS_TMO);
        const __amdgpu_buffer_rsrc_t rw = __builtin_amdgcn_make_buffer_rsrc((void*)p.w, 0, (unsigned)(K_ * NL), 0x00020000);
        const unsigned we = (lane >> 4) * NL + (((lane & 15) ^ (2 * (lane >> 4))) << 4);
        const unsigned wo = (lane >> 4) * NL + (((lane & 15) ^ (2 * (4 + (lane >> 4)))) << 4);
        const unsigned w0 = hq * 32 * NL + cb * BN, lb = lds0 + A_BYTES + hq * 8192;
        bool dead = false;
        {
            const __amdgpu_buffer_rsrc_t ra = __builtin_amdgcn_make_buffer_rsrc((void*)(p.ws[DSRC] + WS_DATA), 0, DATA_BYTES, 0x00020000);
            const unsigned char* flagp = myws + WS_FLAG + (((call & 1) * NR + DSRC) * 256 + (rb * 2 + (hq >> 1)) * 32) * 8;
            const unsigned rot = LEADP == 1 ? ~0u : (LEADP == 2 ? 0x55555555u : LEADP == 4 ? 0x11111111u : 0x01010101u) << (cb & (LEADP - 1));
            const unsigned lead = (rot & (rb == 0 ? LM0 : rb == 1 ? LM1 : rb == 2 ? LM2 : LM3)) | LMX;
            dma8(rw, we, wo, w0, 4 * NL, lb);
            bool inv_ok = false;
            unsigned mask = FIRSTMASK;
#pragma nounroll
            for (;;) {
                unsigned fmeta;
                if (!poll_flags(flagp, (lane & 31) * 8, mask, invp, inv_ok, tmo, call, fmeta)) { dead = true; break; }
                STAMP(st1, mask == FIRSTMASK ? 0 : 1);
                unsigned sa = (call & 1) * PACK_PAR + (rb * 4 + hq) * 32 * QU, sw = w0 + BK * NL, spin = 1u << 25;
                unsigned k1, m0_, m1_, m2_, t0, t1, l1, l2, keep;
                asm volatile(HELPER_ASM
                    : [k1] "=&s"(k1), [m0] "=&s"(m0_), [m1] "=&s"(m1_), [m2] "=&s"(m2_), [t0] "=&s"(t0), [t1] "=&s"(t1), [l1] "=&s"(l1), [l2] "=&s"(l2), [kp] "=&s"(keep),
                      [sa] "+s"(sa), [sw] "+s"(sw), [sp] "+s"(spin)
                    : [ra] "s"(ra), [rw] "s"(rw), [sr] "s"(RAW_OFF + (call & 1) * (unsigned)SHARD + (rb * 4 + hq) * 32 * 4096 - 4096), [call] "s"(call), [fp] "s"(flagp),
                      [l0] "s"(lds0 + hq * 4096), [lb] "s"(lb), [csh] "s"(8 + 8 * (hq & 1)), [ld] "s"(lead), [m60] "s"(0x60606060u), [hq4] "s"(4 * hq),
                      [vq] "v"(lane * 16), [we] "v"(we), [wo] "v"(wo), [vs] "v"(sync), [fm] "v"(fmeta)
                    : "memory", "scc", "vcc", HELPER_CLOBBERS);
                if (k1 != 1) { dead = k1 != NK + 1; break; }
                if (mask == ~0u) { dead = true; break; }
                mask = ~0u;
            }
            STAMP(st1, 31);
        }
        if (dead) lds_st(sync + 32, 1u);
        lds_st(sync + 4 * hq, NK);
        scale = p.sx[0] * p.sw[0];
        if (rfl(__hip_atomic_load(tmo, __ATOMIC_RELAXED, __HIP_MEMORY_SCOPE_AGENT)) == call) lds_st(sync + 32, 1u);
        __builtin_amdgcn_s_setprio(0);
        float is = __builtin_amdgcn_rcpf(scale);
        const float nsl = scale * -1.44269504088896340736f;
        const unsigned ovoff = (lane >> 3) * (NL * 4) + (lane & 7) * 16;
        const LAS unsigned char* rd = (const LAS unsigned char*)smem + hq * 32768 + (lane >> 3) * 128 + (((lane & 7) ^ (lane >> 3)) << 4);
        unsigned have = wait_word(sync + 48 + 4 * hq, ~0u, 1);
        if (lds_ld1(sync + 32) != 0u) is = __builtin_nanf("");
#pragma unroll
        for (int sb = 0; sb < NH; ++sb) {
            if (have <= (unsigned)sb) have = wait_word(sync + 48 + 4 * hq, ~0u, sb + 1);
            f32x4 v[4];
#pragma unroll
            for (int i = 0; i < 4; ++i) v[i] = *(const LAS f32x4*)(rd + sb * 4096 + i * 1024);
#pragma unroll
            for (int i = 0; i < 4; ++i) st_out(act4(v[i], nsl, is), ro, ovoff, osoff + ((sb >> 1) * 32 + 8 * i) * (NL * 4) + (sb & 1) * 128);
        }
    }
}

}

extern "C" void kernel_launch(int rank, int nranks, ncclComm_t comm, hipStream_t stream,
                              void* const* d_in, const int* in_sizes, int n_in,
                              void* d_out, int out_size, const int* in_dtypes, int out_dtype,
                              void* const* d_out_all, void* const* d_ws_all, size_t ws_bytes) {
    (void)comm; (void)in_dtypes; (void)out_dtype; (void)d_out_all;
    if (nranks != NR || n_in != 4 || rank < 0 || rank >= NR || in_sizes[0] != MS * K_ || in_sizes[1] != K_ * NL || out_size != M_ * NL || ws_bytes < WS_NEED) {
        fprintf(stderr, "ag_gemm: unexpected shapes\n");
        return;
    }
    Params p;
    p.x = (const unsigned char*)d_in[0]; p.w = (const unsigned char*)d_in[1];
    p.sx = (const float*)d_in[2]; p.sw = (const float*)d_in[3]; p.out = (float*)d_out;
    for (int i = 0; i < NR; ++i) p.ws[i] = (unsigned char*)d_ws_all[i];
    p.rank = (unsigned)rank; p.pad = 0;
    hipLaunchKernelGGL(ag_gemm_kernel, dim3(NWG), dim3(512), 0, stream, p);
    hipError_t e = hipGetLastError();
    if (e != hipSuccess) fprintf(stderr, "ag_gemm launch: %s\n", hipGetErrorString(e));
}
```

```cpp
#include <hip/hip_runtime.h>
#include <rccl/rccl.h>
#include <cstdio>

namespace {

constexpr int NR = 8, M_ = 4096, K_ = 4096, NL = 2048, MS = M_ / NR;
constexpr int BM = 128, BN = 256, BK = 128, NK = K_ / BK, NWG = 256;
constexpr unsigned STAGE = BM * BK, NSTAGE = 7;
constexpr unsigned WT_OFF = NSTAGE * STAGE;
constexpr unsigned ENC_OFF = WT_OFF + BK * BN;
constexpr unsigned LIST_OFF = ENC_OFF + 6144;
constexpr unsigned SYNC_OFF = LIST_OFF + 512;
constexpr unsigned LDS_BYTES = SYNC_OFF + 112;
constexpr unsigned EXC_CAP = 128;
constexpr size_t SHARD = (size_t)MS * K_;
constexpr size_t WS_CNT = 0;
constexpr size_t WS_TMO = 64 << 10;
constexpr size_t WS_WFLAG = 128 << 10;
constexpr size_t WS_FLAG = 512 << 10;
constexpr size_t WS_DATA = 1 << 20;
constexpr unsigned QU = 3072 + 256;
constexpr unsigned PACK_PAR = 512 * QU;
constexpr unsigned RAW_OFF = 4 << 20;
constexpr unsigned DATA_BYTES = RAW_OFF + 2 * (unsigned)SHARD;
constexpr unsigned NWT = 8;
constexpr size_t WS_WT = WS_DATA + DATA_BYTES;
constexpr size_t WS_NEED = WS_WT + (size_t)(NWT + 1) * K_ * NL;
static_assert(2 * PACK_PAR <= RAW_OFF && QU == 0xd00 && NSTAGE == 7 && SYNC_OFF % 16 == 0, "constants of the asm statements");
constexpr unsigned long long DEADLINE_TICKS = 30000000ull;
constexpr unsigned LOOK = 12;
constexpr unsigned WEARLY = 16;

typedef unsigned u32x2 __attribute__((ext_vector_type(2)));
typedef unsigned u32x4 __attribute__((ext_vector_type(4)));
typedef float f32x4 __attribute__((ext_vector_type(4)));
#define LAS __attribute__((address_space(3)))
#define GAS __attribute__((address_space(1)))

struct Params {
    unsigned char* ws[NR];
    const unsigned char* x; unsigned char* myws; const unsigned char* w; float* out; const float* sx; const float* sw;
    unsigned rank, srcmap;
};

__device__ __forceinline__ unsigned rfl(unsigned v) { return __builtin_amdgcn_readfirstlane(v); }
__device__ __forceinline__ void lds_st(unsigned addr, unsigned v) { asm volatile("ds_write_b32 %0, %1" :: "v"(addr), "v"(v) : "memory"); }
__device__ __forceinline__ unsigned lds_ld(unsigned addr) { unsigned r; asm volatile("ds_read_b32 %0, %1\n\ts_waitcnt lgkmcnt(0)" : "=&v"(r) : "v"(addr) : "memory"); return r; }
__device__ __forceinline__ unsigned wait_lds(unsigned addr, unsigned need) {
    unsigned v = 0;
#pragma nounroll
    for (unsigned sp = 0; sp < (1u << 25); ++sp) {
        v = lds_ld(addr);
        if (__ballot(v >= need) == ~0ull) break;
        __builtin_amdgcn_s_sleep(1);
    }
    return rfl(v);
}
__device__ __forceinline__ bool give_up(unsigned long long t0, GAS unsigned* tmo, unsigned call) {
    if (rfl(__hip_atomic_load(tmo, __ATOMIC_RELAXED, __HIP_MEMORY_SCOPE_AGENT)) != call && __builtin_amdgcn_s_memrealtime() - t0 <= DEADLINE_TICKS) return false;
    __hip_atomic_store(tmo, call, __ATOMIC_RELAXED, __HIP_MEMORY_SCOPE_AGENT);
    return true;
}
__device__ __forceinline__ bool poll_flags(const unsigned char* base, unsigned off, GAS unsigned* tmo, unsigned call, unsigned& meta) {
    const unsigned long long t0 = __builtin_amdgcn_s_memrealtime();
#pragma nounroll
    for (unsigned sp = 0;; ++sp) {
        u32x2 f;
        asm volatile("global_load_dwordx2 %0, %1, %2 sc0 sc1\n\ts_waitcnt vmcnt(0)" : "=&v"(f) : "v"(off), "s"(base) : "memory");
        meta = f.y;
        if (__ballot(f.x >= call) == ~0ull) return true;
        if ((sp & 15) == 15 && give_up(t0, tmo, call)) return false;
        __builtin_amdgcn_s_sleep(4);
    }
}
__device__ __forceinline__ f32x4 act4(f32x4 a, float nsl, float is) {
    const f32x4 t = a * nsl;
    f32x4 d;
#pragma unroll
    for (int j = 0; j < 4; ++j) d[j] = __builtin_amdgcn_exp2f(__builtin_amdgcn_fmed3f(t[j], -86.5617024533378f, 86.5617024533378f));
    d = d * is + is;
#pragma unroll
    for (int j = 0; j < 4; ++j) d[j] = __builtin_amdgcn_rcpf(d[j]);
    return a * d;
}
__device__ __forceinline__ void st_out(f32x4 o, __amdgpu_buffer_rsrc_t ro, unsigned voff, unsigned soff) {
    asm volatile("buffer_store_dwordx4 %0, %1, %2, %3 offen sc1\n\ts_nop 1" :: "v"(o), "v"(voff), "s"(ro), "s"(soff) : "memory");
}

#define S40_59 "s40", "s41", "s42", "s43", "s44", "s45", "s46", "s47", "s48", "s49", "s50", "s51", "s52", "s53", "s54", "s55", "s56", "s57", "s58", "s59"
#define S60_69 "s60", "s61", "s62", "s63", "s64", "s65", "s66", "s67", "s68", "s69"
#define V6_27 "v6", "v7", "v8", "v9", "v10", "v11", "v12", "v13", "v14", "v15", "v16", "v17", "v18", "v19", "v20", "v21", "v22", "v23", "v24", "v25", "v26", "v27"
#define V28_127 "v28", "v29", "v30", "v31", "v32", "v33", "v34", "v35", "v36", "v37", "v38", "v39", "v40", "v41", "v42", "v43", "v44", "v45", "v46", "v47", "v48", "v49", "v50", "v51", "v52", "v53", "v54", "v55", "v56", "v57", "v58", "v59", "v60", "v61", "v62", "v63", "v64", "v65", "v66", "v67", "v68", "v69", "v70", "v71", "v72", "v73", "v74", "v75", "v76", "v77", "v78", "v79", "v80", "v81", "v82", "v83", "v84", "v85", "v86", "v87", "v88", "v89", "v90", "v91", "v92", "v93", "v94", "v95", "v96", "v97", "v98", "v99", "v100", "v101", "v102", "v103", "v104", "v105", "v106", "v107", "v108", "v109", "v110", "v111", "v112", "v113", "v114", "v115", "v116", "v117", "v118", "v119", "v120", "v121", "v122", "v123", "v124", "v125", "v126", "v127"
#define V40_127 "v40", "v41", "v42", "v43", "v44", "v45", "v46", "v47", "v48", "v49", "v50", "v51", "v52", "v53", "v54", "v55", "v56", "v57", "v58", "v59", "v60", "v61", "v62", "v63", "v64", "v65", "v66", "v67", "v68", "v69", "v70", "v71", "v72", "v73", "v74", "v75", "v76", "v77", "v78", "v79", "v80", "v81", "v82", "v83", "v84", "v85", "v86", "v87", "v88", "v89", "v90", "v91", "v92", "v93", "v94", "v95", "v96", "v97", "v98", "v99", "v100", "v101", "v102", "v103", "v104", "v105", "v106", "v107", "v108", "v109", "v110", "v111", "v112", "v113", "v114", "v115", "v116", "v117", "v118", "v119", "v120", "v121", "v122", "v123", "v124", "v125", "v126", "v127"
#define V44_107 "v44", "v45", "v46", "v47", "v48", "v49", "v50", "v51", "v52", "v53", "v54", "v55", "v56", "v57", "v58", "v59", "v60", "v61", "v62", "v63", "v64", "v65", "v66", "v67", "v68", "v69", "v70", "v71", "v72", "v73", "v74", "v75", "v76", "v77", "v78", "v79", "v80", "v81", "v82", "v83", "v84", "v85", "v86", "v87", "v88", "v89", "v90", "v91", "v92", "v93", "v94", "v95", "v96", "v97", "v98", "v99", "v100", "v101", "v102", "v103", "v104", "v105", "v106", "v107"
#define A0_127 "a0", "a1", "a2", "a3", "a4", "a5", "a6", "a7", "a8", "a9", "a10", "a11", "a12", "a13", "a14", "a15", "a16", "a17", "a18", "a19", "a20", "a21", "a22", "a23", "a24", "a25", "a26", "a27", "a28", "a29", "a30", "a31", "a32", "a33", "a34", "a35", "a36", "a37", "a38", "a39", "a40", "a41", "a42", "a43", "a44", "a45", "a46", "a47", "a48", "a49", "a50", "a51", "a52", "a53", "a54", "a55", "a56", "a57", "a58", "a59", "a60", "a61", "a62", "a63", "a64", "a65", "a66", "a67", "a68", "a69", "a70", "a71", "a72", "a73", "a74", "a75", "a76", "a77", "a78", "a79", "a80", "a81", "a82", "a83", "a84", "a85", "a86", "a87", "a88", "a89", "a90", "a91", "a92", "a93", "a94", "a95", "a96", "a97", "a98", "a99", "a100", "a101", "a102", "a103", "a104", "a105", "a106", "a107", "a108", "a109", "a110", "a111", "a112", "a113", "a114", "a115", "a116", "a117", "a118", "a119", "a120", "a121", "a122", "a123", "a124", "a125", "a126", "a127"

#define ENCODE_ASM \
    "ds_write_b32 %[via], %[viv]\n\ts_mov_b64 s[40:41], %[st]\n\ts_and_b32 s41, s41, 0xffff\n\ts_mov_b32 s42, 0x1bad00\n\ts_mov_b32 s43, 0x20000\n\t" \
    "s_mov_b32 s44, 0x80808080\n\ts_mov_b32 s46, 0x1f1f1f1f\n\ts_mov_b32 s47, 0x60606060\n\ts_mov_b32 s48, 0x40404040\n\t" \
    "s_mov_b32 s49, 0x20202020\n\ts_mov_b32 s55, 0x02020202\n\ts_mov_b32 s56, 0x04040404\n\ts_mov_b32 s57, 0x08080808\n\t" \
    "v_mov_b32 v45, 0xe0e0e0e0\n\tv_add_u32 v46, 0x1a000, %[vq]\n\tv_and_b32 v47, 0xf0, %[vq]\n\tv_bfe_u32 v44, %[vq], 8, 1\n\t" \
    "v_mul_u32_u24 v44, 0x1a000, v44\n\tv_add_u32 v47, v47, v44\n\tv_and_b32 v72, 0x1f0, %[vq]\n\tv_sub_u32 v72, v72, %[vq]\n\t" \
    "v_add_u32 v72, v72, %[vrd]\n\ts_waitcnt lgkmcnt(0)\n\ts_barrier\n\t" \
      \
    "s_cmp_eq_u32 %[role], 1\n\ts_cbranch_scc0 .Lnw%=\n\ts_mov_b32 s59, m0\n\ts_mov_b32 m0, %[wld]\n\tv_add_u32 v64, 0x3c00, %[vw]\n\t" \
    "v_add_u32 v65, 0x7800, %[vw]\n\tv_add_u32 v66, 0xb400, %[vw]\n\ts_add_u32 s52, %[wso], 0x10000\n\t" \
    "buffer_load_dwordx4 %[vw], %[rw], %[wso] offen offset:0 lds\n\tbuffer_load_dwordx4 v64, %[rw], %[wso] offen offset:1024 lds\n\t" \
    "buffer_load_dwordx4 v65, %[rw], %[wso] offen offset:2048 lds\n\tbuffer_load_dwordx4 v66, %[rw], %[wso] offen offset:3072 lds\n\t" \
    "s_add_u32 m0, m0, 0x1000\n\ts_nop 0\n\tbuffer_load_dwordx4 %[vw], %[rw], s52 offen offset:0 lds\n\t" \
    "buffer_load_dwordx4 v64, %[rw], s52 offen offset:1024 lds\n\tbuffer_load_dwordx4 v65, %[rw], s52 offen offset:2048 lds\n\t" \
    "buffer_load_dwordx4 v66, %[rw], s52 offen offset:3072 lds\n\ts_mov_b32 m0, s59\n\ts_waitcnt vmcnt(8)\n\ts_branch .Lw8%=\n\t" \
    ".Lnw%=:\n\ts_waitcnt vmcnt(0)\n\t" \
    ".Lw8%=:\n\t" \
      \
    "v_and_b32 v48, 0x7f7f7f7f, v36\n\tv_and_b32 v49, 0x7f7f7f7f, v37\n\tv_and_b32 v50, 0x7f7f7f7f, v38\n\tv_and_b32 v51, 0x7f7f7f7f, v39\n\t" \
    "v_sad_u8 v48, v48, 0, 0\n\tv_sad_u8 v50, v50, 0, 0\n\tv_sad_u8 v48, v49, 0, v48\n\tv_sad_u8 v50, v51, 0, v50\n\t" \
    "v_readfirstlane_b32 %[call], v32\n\tv_add_u32 v48, v48, v50\n\tv_or_b32 v52, 0x80808080, v40\n\tv_or_b32 v53, 0x80808080, v41\n\t" \
    "v_add_u32_dpp v48, v48, v48 row_shr:1 row_mask:0xf bank_mask:0xf bound_ctrl:1\n\tv_or_b32 v54, 0x80808080, v42\n\t" \
    "v_or_b32 v55, 0x80808080, v43\n\tv_add_u32_dpp v48, v48, v48 row_shr:2 row_mask:0xf bank_mask:0xf bound_ctrl:1\n\t" \
    "s_add_u32 %[call], %[call], 1\n\ts_and_b32 s53, %[call], 1\n\tv_add_u32_dpp v48, v48, v48 row_shr:4 row_mask:0xf bank_mask:0xf bound_ctrl:1\n\t" \
    "s_mul_i32 s54, s53, 0x1a0000\n\ts_lshl_b32 s53, s53, 18\n\tv_add_u32_dpp v48, v48, v48 row_shr:8 row_mask:0xf bank_mask:0xf bound_ctrl:1\n\t" \
    "s_add_u32 s53, s53, %[fo]\n\ts_nop 0\n\tv_add_u32_dpp v48, v48, v48 row_bcast:15 row_mask:0xa bank_mask:0xf\n\t" \
    "v_add_co_u32 v70, vcc, s53, v34\n\ts_nop 0\n\tv_add_u32_dpp v48, v48, v48 row_bcast:31 row_mask:0xc bank_mask:0xf\n\t" \
    "v_addc_co_u32 v71, vcc, 0, v35, vcc\n\tv_readlane_b32 %[meta], v48, 63\n\ts_sub_i32 %[meta], %[meta], 20582\n\t" \
    "s_ashr_i32 %[meta], %[meta], 10\n\ts_max_i32 %[meta], %[meta], 0\n\ts_min_i32 %[meta], %[meta], 96\n\ts_mul_i32 s52, %[meta], 0x1010101\n\t" \
      \
    "s_cmp_eq_u32 %[role], 4\n\ts_cbranch_scc0 .Lnr%=\n\ts_lshl_b32 s59, %[call], 3\n\ts_or_b32 s59, s59, %[xc]\n\tv_mov_b32 v62, 0\n\t" \
    "v_mov_b32 v63, s59\n\ts_mov_b64 exec, 1\n\tglobal_store_dword v62, v63, %[rp] sc1\n\ts_mov_b64 exec, -1\n\t" \
    ".Lnr%=:\n\t" \
      \
    "v_subrev_u32 v52, s52, v52\n\tv_subrev_u32 v53, s52, v53\n\tv_subrev_u32 v54, s52, v54\n\tv_subrev_u32 v55, s52, v55\n\t" \
    "v_bitop3_b32 v56, v52, s44, v45 bitop3:0x28\n\tv_bitop3_b32 v57, v53, s44, v45 bitop3:0x28\n\tv_bitop3_b32 v58, v54, s44, v45 bitop3:0x28\n\t" \
    "v_bitop3_b32 v59, v55, s44, v45 bitop3:0x28\n\tv_lshrrev_b32 v48, 1, v56\n\tv_lshrrev_b32 v49, 1, v57\n\tv_lshrrev_b32 v50, 1, v58\n\t" \
    "v_lshrrev_b32 v51, 1, v59\n\tv_add_u32 v48, 0x70707070, v48\n\tv_add_u32 v49, 0x70707070, v49\n\tv_add_u32 v50, 0x70707070, v50\n\t" \
    "v_add_u32 v51, 0x70707070, v51\n\tv_lshrrev_b32 v48, 7, v48\n\tv_lshrrev_b32 v49, 6, v49\n\tv_lshrrev_b32 v50, 5, v50\n\t" \
    "v_lshrrev_b32 v51, 4, v51\n\tv_and_b32 v60, 0x01010101, v48\n\tv_and_or_b32 v60, v49, s55, v60\n\tv_and_or_b32 v60, v50, s56, v60\n\t" \
    "v_and_or_b32 v60, v51, s57, v60\n\tv_bcnt_u32_b32 v61, v60, 0\n\tv_cmp_ne_u32 vcc, 0, v60\n\ts_mov_b64 exec, vcc\n\t" \
    "ds_add_rtn_u32 v61, %[vca], v61\n\ts_mov_b64 exec, -1\n\t" \
      \
    "v_bfi_b32 v52, s46, v52, v40\n\tv_bfi_b32 v53, s46, v53, v41\n\tv_bfi_b32 v54, s46, v54, v42\n\tv_bfi_b32 v55, s46, v55, v43\n\t" \
    "v_lshlrev_b32 v48, 5, v55\n\tv_lshlrev_b32 v49, 3, v55\n\tv_lshrrev_b32 v50, 1, v55\n\tv_lshlrev_b32 v51, 1, v55\n\t" \
    "v_bfi_b32 v52, s47, v48, v52\n\tv_bfi_b32 v53, s47, v49, v53\n\tv_bfi_b32 v54, s48, v50, v54\n\tds_write_b32 %[ven], v52\n\t" \
    "v_bfi_b32 v54, s49, v51, v54\n\tds_write_b32 %[ven], v53 offset:1024\n\tds_write_b32 %[ven], v54 offset:2048\n\ts_waitcnt lgkmcnt(3)\n\t" \
      \
    ".Lx%=:\n\tv_cmp_ne_u32 vcc, 0, v60\n\ts_and_b64 exec, exec, vcc\n\ts_cbranch_execz .Lxe%=\n\tv_ffbl_b32 v48, v60\n\tv_and_b32 v49, 1, v48\n\t" \
    "v_and_b32 v50, 2, v48\n\tv_cmp_ne_u32 vcc, 0, v49\n\tv_cmp_ne_u32 s[50:51], 0, v50\n\tv_and_or_b32 v51, v48, 24, 5\n\tv_and_b32 v62, 3, v48\n\t" \
    "v_cndmask_b32 v49, v56, v57, vcc\n\tv_cndmask_b32 v50, v58, v59, vcc\n\tv_lshrrev_b32 v48, 3, v48\n\tv_cndmask_b32 v49, v49, v50, s[50:51]\n\t" \
    "v_lshl_or_b32 v48, v62, 10, v48\n\tv_lshrrev_b32 v49, v51, v49\n\tv_min_u32 v50, 0x7f, v61\n\tv_lshlrev_b32 v49, 13, v49\n\t" \
    "v_lshl_add_u32 v50, v50, 1, %[lst]\n\tv_or3_b32 v48, v48, v49, %[vps]\n\tv_add_u32 v62, -1, v60\n\tds_write_b16 v50, v48\n\t" \
    "v_add_u32 v61, 1, v61\n\tv_and_b32 v60, v60, v62\n\ts_branch .Lx%=\n\t" \
    ".Lxe%=:\n\ts_mov_b64 exec, -1\n\ts_waitcnt lgkmcnt(0)\n\ts_barrier\n\t" \
      \
    "s_cmp_eq_u32 %[role], 2\n\ts_cbranch_scc0 .Lns%=\n\tds_read_b128 v[76:79], %[vrd] offset:0\n\tds_read_b128 v[80:83], %[vrd] offset:1024\n\t" \
    "ds_read_b128 v[84:87], %[vrd] offset:2048\n\tds_read_b128 v[88:91], %[vrd] offset:3072\n\tds_read_b128 v[92:95], %[vrd] offset:4096\n\t" \
    "ds_read_b128 v[96:99], %[vrd] offset:5120\n\tds_read_b128 v[100:103], v72 offset:6144\n\tv_and_b32 v48, -16, %[vca]\n\t" \
    "ds_read_b128 v[104:107], v48 offset:0\n\ts_waitcnt lgkmcnt(7)\n\tbuffer_store_dwordx4 v[76:79], %[vq], s[40:43], s54 offen offset:0 sc0 sc1\n\t" \
    "s_waitcnt lgkmcnt(6)\n\tbuffer_store_dwordx4 v[80:83], %[vq], s[40:43], s54 offen offset:1024 sc0 sc1\n\ts_waitcnt lgkmcnt(5)\n\t" \
    "buffer_store_dwordx4 v[84:87], %[vq], s[40:43], s54 offen offset:2048 sc0 sc1\n\ts_waitcnt lgkmcnt(4)\n\t" \
    "buffer_store_dwordx4 v[88:91], v46, s[40:43], s54 offen offset:0 sc0 sc1\n\ts_waitcnt lgkmcnt(3)\n\t" \
    "buffer_store_dwordx4 v[92:95], v46, s[40:43], s54 offen offset:1024 sc0 sc1\n\ts_waitcnt lgkmcnt(2)\n\t" \
    "buffer_store_dwordx4 v[96:99], v46, s[40:43], s54 offen offset:2048 sc0 sc1\n\ts_waitcnt lgkmcnt(0)\n\ts_mov_b32 exec_hi, 0\n\t" \
    "buffer_store_dwordx4 v[100:103], v47, s[40:43], s54 offen offset:3072 sc0 sc1\n\ts_mov_b64 exec, -1\n\tv_readfirstlane_b32 s52, v105\n\t" \
    "v_readfirstlane_b32 s53, v106\n\ts_min_u32 s52, s52, 0xff\n\ts_min_u32 s53, s53, 0xff\n\ts_max_u32 s59, s52, s53\n\ts_lshl_b32 s52, s52, 8\n\t" \
    "s_lshl_b32 s53, s53, 16\n\ts_or_b32 %[meta], %[meta], s52\n\ts_or_b32 %[meta], %[meta], s53\n\tv_mov_b32 v48, %[call]\n\t" \
    "v_mov_b32 v49, %[meta]\n\ts_cmp_gt_u32 s59, 0x80\n\ts_waitcnt vmcnt(0)\n\ts_cbranch_scc1 .Lns%=\n\ts_mov_b64 exec, 0xff\n\t" \
    "global_store_dwordx2 v[70:71], v[48:49], off sc0 sc1\n\ts_mov_b64 exec, -1\n\t" \
    ".Lns%=:\n\t" \
    ""

#define WDMA_ASM \
    "s_mov_b32 s59, m0\n\ts_mov_b32 m0, %[wld]\n\tv_add_u32 v64, 0x3c00, %[vw]\n\tv_add_u32 v65, 0x7800, %[vw]\n\tv_add_u32 v66, 0xb400, %[vw]\n\t" \
    "s_add_u32 s52, %[wso], 0x10000\n\tbuffer_load_dwordx4 %[vw], %[rw], %[wso] offen offset:0 lds\n\t" \
    "buffer_load_dwordx4 v64, %[rw], %[wso] offen offset:1024 lds\n\tbuffer_load_dwordx4 v65, %[rw], %[wso] offen offset:2048 lds\n\t" \
    "buffer_load_dwordx4 v66, %[rw], %[wso] offen offset:3072 lds\n\ts_add_u32 m0, m0, 0x1000\n\ts_nop 0\n\t" \
    "buffer_load_dwordx4 %[vw], %[rw], s52 offen offset:0 lds\n\tbuffer_load_dwordx4 v64, %[rw], s52 offen offset:1024 lds\n\t" \
    "buffer_load_dwordx4 v65, %[rw], s52 offen offset:2048 lds\n\tbuffer_load_dwordx4 v66, %[rw], s52 offen offset:3072 lds\n\ts_mov_b32 m0, s59\n\t" \
    ""

#define RETILE_ASM \
    "s_waitcnt vmcnt(0)\n\tds_read_b64_tr_b8 v[32:33], %[ta] offset:0\n\tds_read_b64_tr_b8 v[34:35], %[ta] offset:1024\n\t" \
    "ds_read_b64_tr_b8 v[36:37], %[ta] offset:16\n\tds_read_b64_tr_b8 v[38:39], %[ta] offset:1040\n\tds_read_b64_tr_b8 v[40:41], %[ta] offset:32\n\t" \
    "ds_read_b64_tr_b8 v[42:43], %[ta] offset:1056\n\tds_read_b64_tr_b8 v[44:45], %[ta] offset:48\n\t" \
    "ds_read_b64_tr_b8 v[46:47], %[ta] offset:1072\n\tds_read_b64_tr_b8 v[48:49], %[ta] offset:64\n\t" \
    "ds_read_b64_tr_b8 v[50:51], %[ta] offset:1088\n\tds_read_b64_tr_b8 v[52:53], %[ta] offset:80\n\t" \
    "ds_read_b64_tr_b8 v[54:55], %[ta] offset:1104\n\tds_read_b64_tr_b8 v[56:57], %[ta] offset:96\n\t" \
    "ds_read_b64_tr_b8 v[58:59], %[ta] offset:1120\n\tds_read_b64_tr_b8 v[60:61], %[ta] offset:112\n\t" \
    "ds_read_b64_tr_b8 v[62:63], %[ta] offset:1136\n\ts_waitcnt lgkmcnt(0)\n\tglobal_store_dwordx4 %[v0], v[32:35], %[dst] offset:0 sc1\n\t" \
    "global_store_dwordx4 %[v0], v[36:39], %[dst] offset:1024 sc1\n\tglobal_store_dwordx4 %[v0], v[40:43], %[dst] offset:2048 sc1\n\t" \
    "global_store_dwordx4 %[v0], v[44:47], %[dst] offset:3072 sc1\n\tglobal_store_dwordx4 %[v1], v[48:51], %[dst] offset:0 sc1\n\t" \
    "global_store_dwordx4 %[v1], v[52:55], %[dst] offset:1024 sc1\n\tglobal_store_dwordx4 %[v1], v[56:59], %[dst] offset:2048 sc1\n\t" \
    "global_store_dwordx4 %[v1], v[60:63], %[dst] offset:3072 sc1\n\t" \
    ""
#define RETILE_FLAG_ASM \
    "v_mov_b32 v29, 1\n\ts_waitcnt vmcnt(0)\n\ts_mov_b64 exec, 1\n\tds_add_rtn_u32 v28, %[ca], v29\n\ts_waitcnt lgkmcnt(0)\n\t" \
    "v_readfirstlane_b32 %[t0], v28\n\ts_cmp_eq_u32 %[t0], 3\n\ts_cbranch_scc0 .LN%=\n\tglobal_store_dword %[vf], %[call], %[fp] sc1\n\t" \
    ".LN%=:\n\ts_mov_b64 exec, -1\n\t" \
    ""

#define ZERO_ASM \
    "v_accvgpr_write_b32 a0, 0\n\tv_accvgpr_write_b32 a1, 0\n\tv_accvgpr_write_b32 a2, 0\n\tv_accvgpr_write_b32 a3, 0\n\t" \
    "v_accvgpr_write_b32 a4, 0\n\tv_accvgpr_write_b32 a5, 0\n\tv_accvgpr_write_b32 a6, 0\n\tv_accvgpr_write_b32 a7, 0\n\t" \
    "v_accvgpr_write_b32 a8, 0\n\tv_accvgpr_write_b32 a9, 0\n\tv_accvgpr_write_b32 a10, 0\n\tv_accvgpr_write_b32 a11, 0\n\t" \
    "v_accvgpr_write_b32 a12, 0\n\tv_accvgpr_write_b32 a13, 0\n\tv_accvgpr_write_b32 a14, 0\n\tv_accvgpr_write_b32 a15, 0\n\t" \
    "v_accvgpr_write_b32 a16, 0\n\tv_accvgpr_write_b32 a17, 0\n\tv_accvgpr_write_b32 a18, 0\n\tv_accvgpr_write_b32 a19, 0\n\t" \
    "v_accvgpr_write_b32 a20, 0\n\tv_accvgpr_write_b32 a21, 0\n\tv_accvgpr_write_b32 a22, 0\n\tv_accvgpr_write_b32 a23, 0\n\t" \
    "v_accvgpr_write_b32 a24, 0\n\tv_accvgpr_write_b32 a25, 0\n\tv_accvgpr_write_b32 a26, 0\n\tv_accvgpr_write_b32 a27, 0\n\t" \
    "v_accvgpr_write_b32 a28, 0\n\tv_accvgpr_write_b32 a29, 0\n\tv_accvgpr_write_b32 a30, 0\n\tv_accvgpr_write_b32 a31, 0\n\t" \
    "v_accvgpr_write_b32 a32, 0\n\tv_accvgpr_write_b32 a33, 0\n\tv_accvgpr_write_b32 a34, 0\n\tv_accvgpr_write_b32 a35, 0\n\t" \
    "v_accvgpr_write_b32 a36, 0\n\tv_accvgpr_write_b32 a37, 0\n\tv_accvgpr_write_b32 a38, 0\n\tv_accvgpr_write_b32 a39, 0\n\t" \
    "v_accvgpr_write_b32 a40, 0\n\tv_accvgpr_write_b32 a41, 0\n\tv_accvgpr_write_b32 a42, 0\n\tv_accvgpr_write_b32 a43, 0\n\t" \
    "v_accvgpr_write_b32 a44, 0\n\tv_accvgpr_write_b32 a45, 0\n\tv_accvgpr_write_b32 a46, 0\n\tv_accvgpr_write_b32 a47, 0\n\t" \
    "v_accvgpr_write_b32 a48, 0\n\tv_accvgpr_write_b32 a49, 0\n\tv_accvgpr_write_b32 a50, 0\n\tv_accvgpr_write_b32 a51, 0\n\t" \
    "v_accvgpr_write_b32 a52, 0\n\tv_accvgpr_write_b32 a53, 0\n\tv_accvgpr_write_b32 a54, 0\n\tv_accvgpr_write_b32 a55, 0\n\t" \
    "v_accvgpr_write_b32 a56, 0\n\tv_accvgpr_write_b32 a57, 0\n\tv_accvgpr_write_b32 a58, 0\n\tv_accvgpr_write_b32 a59, 0\n\t" \
    "v_accvgpr_write_b32 a60, 0\n\tv_accvgpr_write_b32 a61, 0\n\tv_accvgpr_write_b32 a62, 0\n\tv_accvgpr_write_b32 a63, 0\n\t" \
    "v_accvgpr_write_b32 a64, 0\n\tv_accvgpr_write_b32 a65, 0\n\tv_accvgpr_write_b32 a66, 0\n\tv_accvgpr_write_b32 a67, 0\n\t" \
    "v_accvgpr_write_b32 a68, 0\n\tv_accvgpr_write_b32 a69, 0\n\tv_accvgpr_write_b32 a70, 0\n\tv_accvgpr_write_b32 a71, 0\n\t" \
    "v_accvgpr_write_b32 a72, 0\n\tv_accvgpr_write_b32 a73, 0\n\tv_accvgpr_write_b32 a74, 0\n\tv_accvgpr_write_b32 a75, 0\n\t" \
    "v_accvgpr_write_b32 a76, 0\n\tv_accvgpr_write_b32 a77, 0\n\tv_accvgpr_write_b32 a78, 0\n\tv_accvgpr_write_b32 a79, 0\n\t" \
    "v_accvgpr_write_b32 a80, 0\n\tv_accvgpr_write_b32 a81, 0\n\tv_accvgpr_write_b32 a82, 0\n\tv_accvgpr_write_b32 a83, 0\n\t" \
    "v_accvgpr_write_b32 a84, 0\n\tv_accvgpr_write_b32 a85, 0\n\tv_accvgpr_write_b32 a86, 0\n\tv_accvgpr_write_b32 a87, 0\n\t" \
    "v_accvgpr_write_b32 a88, 0\n\tv_accvgpr_write_b32 a89, 0\n\tv_accvgpr_write_b32 a90, 0\n\tv_accvgpr_write_b32 a91, 0\n\t" \
    "v_accvgpr_write_b32 a92, 0\n\tv_accvgpr_write_b32 a93, 0\n\tv_accvgpr_write_b32 a94, 0\n\tv_accvgpr_write_b32 a95, 0\n\t" \
    "v_accvgpr_write_b32 a96, 0\n\tv_accvgpr_write_b32 a97, 0\n\tv_accvgpr_write_b32 a98, 0\n\tv_accvgpr_write_b32 a99, 0\n\t" \
    "v_accvgpr_write_b32 a100, 0\n\tv_accvgpr_write_b32 a101, 0\n\tv_accvgpr_write_b32 a102, 0\n\tv_accvgpr_write_b32 a103, 0\n\t" \
    "v_accvgpr_write_b32 a104, 0\n\tv_accvgpr_write_b32 a105, 0\n\tv_accvgpr_write_b32 a106, 0\n\tv_accvgpr_write_b32 a107, 0\n\t" \
    "v_accvgpr_write_b32 a108, 0\n\tv_accvgpr_write_b32 a109, 0\n\tv_accvgpr_write_b32 a110, 0\n\tv_accvgpr_write_b32 a111, 0\n\t" \
    "v_accvgpr_write_b32 a112, 0\n\tv_accvgpr_write_b32 a113, 0\n\tv_accvgpr_write_b32 a114, 0\n\tv_accvgpr_write_b32 a115, 0\n\t" \
    "v_accvgpr_write_b32 a116, 0\n\tv_accvgpr_write_b32 a117, 0\n\tv_accvgpr_write_b32 a118, 0\n\tv_accvgpr_write_b32 a119, 0\n\t" \
    "v_accvgpr_write_b32 a120, 0\n\tv_accvgpr_write_b32 a121, 0\n\tv_accvgpr_write_b32 a122, 0\n\tv_accvgpr_write_b32 a123, 0\n\t" \
    "v_accvgpr_write_b32 a124, 0\n\tv_accvgpr_write_b32 a125, 0\n\tv_accvgpr_write_b32 a126, 0\n\tv_accvgpr_write_b32 a127, 0\n\t" \
    ""

#define CONSUMER_ASM \
      \
    "buffer_load_dwordx4 v[32:35], %[vq], %[rw], %[sw0] offen offset:0 sc1\n\t" \
    "buffer_load_dwordx4 v[36:39], %[vq], %[rw], %[sw1] offen offset:0 sc1\n\t" \
    "buffer_load_dwordx4 v[40:43], %[vq], %[rw], %[sw0] offen offset:1024 sc1\n\t" \
    "buffer_load_dwordx4 v[44:47], %[vq], %[rw], %[sw1] offen offset:1024 sc1\n\t" \
    "buffer_load_dwordx4 v[48:51], %[vq], %[rw], %[sw0] offen offset:2048 sc1\n\t" \
    "buffer_load_dwordx4 v[52:55], %[vq], %[rw], %[sw1] offen offset:2048 sc1\n\t" \
    "buffer_load_dwordx4 v[56:59], %[vq], %[rw], %[sw0] offen offset:3072 sc1\n\t" \
    "buffer_load_dwordx4 v[60:63], %[vq], %[rw], %[sw1] offen offset:3072 sc1\n\ts_add_u32 %[sw0], %[sw0], 0x40000\n\t" \
    "s_add_u32 %[sw1], %[sw1], 0x40000\n\tbuffer_load_dwordx4 v[64:67], %[vq], %[rw], %[sw0] offen offset:0 sc1\n\t" \
    "buffer_load_dwordx4 v[68:71], %[vq], %[rw], %[sw1] offen offset:0 sc1\n\t" \
    "buffer_load_dwordx4 v[72:75], %[vq], %[rw], %[sw0] offen offset:1024 sc1\n\t" \
    "buffer_load_dwordx4 v[76:79], %[vq], %[rw], %[sw1] offen offset:1024 sc1\n\t" \
    "buffer_load_dwordx4 v[80:83], %[vq], %[rw], %[sw0] offen offset:2048 sc1\n\t" \
    "buffer_load_dwordx4 v[84:87], %[vq], %[rw], %[sw1] offen offset:2048 sc1\n\t" \
    "buffer_load_dwordx4 v[88:91], %[vq], %[rw], %[sw0] offen offset:3072 sc1\n\t" \
    "buffer_load_dwordx4 v[92:95], %[vq], %[rw], %[sw1] offen offset:3072 sc1\n\ts_add_u32 %[sw0], %[sw0], 0x40000\n\t" \
    "s_add_u32 %[sw1], %[sw1], 0x40000\n\t" \
      \
    "s_cmp_eq_u32 %[wf], 0\n\ts_cbranch_scc1 .Lspinc%=\n\tv_mov_b32 v6, 1\n\tv_mov_b32 v7, %[wf]\n\ts_mov_b64 exec, 1\n\ts_waitcnt vmcnt(16)\n\t" \
    "ds_add_rtn_u32 v6, %[vs], v6 offset:64\n\ts_waitcnt lgkmcnt(0)\n\tv_readfirstlane_b32 %[dl], v6\n\tv_mov_b32 v6, %[call]\n\t" \
    "s_cmp_eq_u32 %[dl], 3\n\ts_cbranch_scc0 .Lnf%=\n\tglobal_store_dword v7, v6, %[my] sc1\n\t" \
    ".Lnf%=:\n\ts_mov_b64 exec, -1\n\ts_branch .Lspinc%=\n\t" \
      \
    ".Lsa%=:\n\ts_waitcnt vmcnt(8)\n\ts_waitcnt lgkmcnt(2)\n\t" \
    "v_mfma_f32_16x16x128_f8f6f4 a[0:3], v[32:39], v[8:15], a[0:3] cbsz:1 blgp:1\n\tds_read_b128 v[24:27], %[a0] offset:4096\n\t" \
    "ds_read_b128 v[28:31], %[a1] offset:4096\n\t" \
    "v_mfma_f32_16x16x128_f8f6f4 a[4:7], v[40:47], v[8:15], a[4:7] cbsz:1 blgp:1\n\t" \
    "buffer_load_dwordx4 v[96:99], %[vq], %[rw], %[sw0] offen offset:0 sc1\n\t" \
    "v_mfma_f32_16x16x128_f8f6f4 a[16:19], v[48:55], v[8:15], a[16:19] cbsz:1 blgp:1\n\t" \
    "buffer_load_dwordx4 v[100:103], %[vq], %[rw], %[sw1] offen offset:0 sc1\n\t" \
    "v_mfma_f32_16x16x128_f8f6f4 a[20:23], v[56:63], v[8:15], a[20:23] cbsz:1 blgp:1\n\ts_add_u32 %[sl], %[sl], 1\n\ts_cmp_eq_u32 %[sl], 7\n\t" \
    "s_cselect_b32 %[dl], %[dn], %[dp]\n\ts_cselect_b32 %[sl], 0, %[sl]\n\ts_waitcnt lgkmcnt(2)\n\t" \
    "v_mfma_f32_16x16x128_f8f6f4 a[8:11], v[32:39], v[16:23], a[8:11] cbsz:1 blgp:1\n\tds_read_b128 v[8:11], %[a0] offset:6144\n\t" \
    "ds_read_b128 v[12:15], %[a1] offset:6144\n\t" \
    "v_mfma_f32_16x16x128_f8f6f4 a[12:15], v[40:47], v[16:23], a[12:15] cbsz:1 blgp:1\n\t" \
    "buffer_load_dwordx4 v[104:107], %[vq], %[rw], %[sw0] offen offset:1024 sc1\n\t" \
    "v_mfma_f32_16x16x128_f8f6f4 a[24:27], v[48:55], v[16:23], a[24:27] cbsz:1 blgp:1\n\t" \
    "buffer_load_dwordx4 v[108:111], %[vq], %[rw], %[sw1] offen offset:1024 sc1\n\t" \
    "v_mfma_f32_16x16x128_f8f6f4 a[28:31], v[56:63], v[16:23], a[28:31] cbsz:1 blgp:1\n\ts_add_u32 %[kk], %[kk], 1\n\tv_mov_b32 v7, %[kk]\n\t" \
    "s_waitcnt lgkmcnt(2)\n\t" \
    "v_mfma_f32_16x16x128_f8f6f4 a[32:35], v[32:39], v[24:31], a[32:35] cbsz:1 blgp:1\n\tds_read_b128 v[16:19], %[a0] offset:8192\n\t" \
    "ds_read_b128 v[20:23], %[a1] offset:8192\n\t" \
    "v_mfma_f32_16x16x128_f8f6f4 a[36:39], v[40:47], v[24:31], a[36:39] cbsz:1 blgp:1\n\t" \
    "buffer_load_dwordx4 v[112:115], %[vq], %[rw], %[sw0] offen offset:2048 sc1\n\t" \
    "v_mfma_f32_16x16x128_f8f6f4 a[48:51], v[48:55], v[24:31], a[48:51] cbsz:1 blgp:1\n\t" \
    "buffer_load_dwordx4 v[116:119], %[vq], %[rw], %[sw1] offen offset:2048 sc1\n\t" \
    "v_mfma_f32_16x16x128_f8f6f4 a[52:55], v[56:63], v[24:31], a[52:55] cbsz:1 blgp:1\n\ts_waitcnt lgkmcnt(2)\n\t" \
    "v_mfma_f32_16x16x128_f8f6f4 a[40:43], v[32:39], v[8:15], a[40:43] cbsz:1 blgp:1\n\tds_read_b128 v[24:27], %[a0] offset:10240\n\t" \
    "ds_read_b128 v[28:31], %[a1] offset:10240\n\t" \
    "v_mfma_f32_16x16x128_f8f6f4 a[44:47], v[40:47], v[8:15], a[44:47] cbsz:1 blgp:1\n\t" \
    "buffer_load_dwordx4 v[120:123], %[vq], %[rw], %[sw0] offen offset:3072 sc1\n\t" \
    "v_mfma_f32_16x16x128_f8f6f4 a[56:59], v[48:55], v[8:15], a[56:59] cbsz:1 blgp:1\n\t" \
    "buffer_load_dwordx4 v[124:127], %[vq], %[rw], %[sw1] offen offset:3072 sc1\n\t" \
    "v_mfma_f32_16x16x128_f8f6f4 a[60:63], v[56:63], v[8:15], a[60:63] cbsz:1 blgp:1\n\ts_add_u32 %[sw0], %[sw0], 0x40000\n\t" \
    "s_add_u32 %[sw1], %[sw1], 0x40000\n\ts_waitcnt lgkmcnt(2)\n\t" \
    "v_mfma_f32_16x16x128_f8f6f4 a[64:67], v[32:39], v[16:23], a[64:67] cbsz:1 blgp:1\n\tds_read_b128 v[8:11], %[a0] offset:12288\n\t" \
    "ds_read_b128 v[12:15], %[a1] offset:12288\n\tds_read_b32 v6, %[vs]\n\t" \
    "v_mfma_f32_16x16x128_f8f6f4 a[68:71], v[40:47], v[16:23], a[68:71] cbsz:1 blgp:1\n\t" \
    "v_mfma_f32_16x16x128_f8f6f4 a[80:83], v[48:55], v[16:23], a[80:83] cbsz:1 blgp:1\n\t" \
    "v_mfma_f32_16x16x128_f8f6f4 a[84:87], v[56:63], v[16:23], a[84:87] cbsz:1 blgp:1\n\ts_waitcnt lgkmcnt(3)\n\t" \
    "v_mfma_f32_16x16x128_f8f6f4 a[72:75], v[32:39], v[24:31], a[72:75] cbsz:1 blgp:1\n\tds_read_b128 v[16:19], %[a0] offset:14336\n\t" \
    "ds_read_b128 v[20:23], %[a1] offset:14336\n\ts_mov_b64 exec, %[em]\n\tds_write_b32 %[vs], v7 offset:16\n\ts_mov_b64 exec, -1\n\t" \
    "v_mfma_f32_16x16x128_f8f6f4 a[76:79], v[40:47], v[24:31], a[76:79] cbsz:1 blgp:1\n\tv_add_u32 %[a0], %[dl], %[a0]\n\t" \
    "v_add_u32 %[a1], %[dl], %[a1]\n\t" \
    "v_mfma_f32_16x16x128_f8f6f4 a[88:91], v[48:55], v[24:31], a[88:91] cbsz:1 blgp:1\n\t" \
    "v_mfma_f32_16x16x128_f8f6f4 a[92:95], v[56:63], v[24:31], a[92:95] cbsz:1 blgp:1\n\ts_waitcnt lgkmcnt(3)\n\tv_cmp_lt_u32 vcc, %[kk], v6\n\t" \
    "v_mfma_f32_16x16x128_f8f6f4 a[96:99], v[32:39], v[8:15], a[96:99] cbsz:1 blgp:1\n\ts_cmp_eq_u64 vcc, -1\n\ts_cbranch_scc0 .Lslowa%=\n\t" \
    "ds_read_b128 v[24:27], %[a0] offset:0\n\tds_read_b128 v[28:31], %[a1] offset:0\n\t" \
    "v_mfma_f32_16x16x128_f8f6f4 a[100:103], v[40:47], v[8:15], a[100:103] cbsz:1 blgp:1\n\t" \
    "v_mfma_f32_16x16x128_f8f6f4 a[112:115], v[48:55], v[8:15], a[112:115] cbsz:1 blgp:1\n\t" \
    "v_mfma_f32_16x16x128_f8f6f4 a[116:119], v[56:63], v[8:15], a[116:119] cbsz:1 blgp:1\n\ts_waitcnt lgkmcnt(2)\n\t" \
    "v_mfma_f32_16x16x128_f8f6f4 a[104:107], v[32:39], v[16:23], a[104:107] cbsz:1 blgp:1\n\tds_read_b128 v[8:11], %[a0] offset:2048\n\t" \
    "ds_read_b128 v[12:15], %[a1] offset:2048\n\t" \
    "v_mfma_f32_16x16x128_f8f6f4 a[108:111], v[40:47], v[16:23], a[108:111] cbsz:1 blgp:1\n\t" \
    "v_mfma_f32_16x16x128_f8f6f4 a[120:123], v[48:55], v[16:23], a[120:123] cbsz:1 blgp:1\n\t" \
    "v_mfma_f32_16x16x128_f8f6f4 a[124:127], v[56:63], v[16:23], a[124:127] cbsz:1 blgp:1\n\t" \
      \
    ".Lsb%=:\n\ts_waitcnt vmcnt(8)\n\ts_waitcnt lgkmcnt(2)\n\t" \
    "v_mfma_f32_16x16x128_f8f6f4 a[0:3], v[64:71], v[24:31], a[0:3] cbsz:1 blgp:1\n\tds_read_b128 v[16:19], %[a0] offset:4096\n\t" \
    "ds_read_b128 v[20:23], %[a1] offset:4096\n\t" \
    "v_mfma_f32_16x16x128_f8f6f4 a[4:7], v[72:79], v[24:31], a[4:7] cbsz:1 blgp:1\n\t" \
    "buffer_load_dwordx4 v[32:35], %[vq], %[rw], %[sw0] offen offset:0 sc1\n\t" \
    "v_mfma_f32_16x16x128_f8f6f4 a[16:19], v[80:87], v[24:31], a[16:19] cbsz:1 blgp:1\n\t" \
    "buffer_load_dwordx4 v[36:39], %[vq], %[rw], %[sw1] offen offset:0 sc1\n\t" \
    "v_mfma_f32_16x16x128_f8f6f4 a[20:23], v[88:95], v[24:31], a[20:23] cbsz:1 blgp:1\n\ts_add_u32 %[sl], %[sl], 1\n\ts_cmp_eq_u32 %[sl], 7\n\t" \
    "s_cselect_b32 %[dl], %[dn], %[dp]\n\ts_cselect_b32 %[sl], 0, %[sl]\n\ts_waitcnt lgkmcnt(2)\n\t" \
    "v_mfma_f32_16x16x128_f8f6f4 a[8:11], v[64:71], v[8:15], a[8:11] cbsz:1 blgp:1\n\tds_read_b128 v[24:27], %[a0] offset:6144\n\t" \
    "ds_read_b128 v[28:31], %[a1] offset:6144\n\t" \
    "v_mfma_f32_16x16x128_f8f6f4 a[12:15], v[72:79], v[8:15], a[12:15] cbsz:1 blgp:1\n\t" \
    "buffer_load_dwordx4 v[40:43], %[vq], %[rw], %[sw0] offen offset:1024 sc1\n\t" \
    "v_mfma_f32_16x16x128_f8f6f4 a[24:27], v[80:87], v[8:15], a[24:27] cbsz:1 blgp:1\n\t" \
    "buffer_load_dwordx4 v[44:47], %[vq], %[rw], %[sw1] offen offset:1024 sc1\n\t" \
    "v_mfma_f32_16x16x128_f8f6f4 a[28:31], v[88:95], v[8:15], a[28:31] cbsz:1 blgp:1\n\ts_add_u32 %[kk], %[kk], 1\n\tv_mov_b32 v7, %[kk]\n\t" \
    "s_waitcnt lgkmcnt(2)\n\t" \
    "v_mfma_f32_16x16x128_f8f6f4 a[32:35], v[64:71], v[16:23], a[32:35] cbsz:1 blgp:1\n\tds_read_b128 v[8:11], %[a0] offset:8192\n\t" \
    "ds_read_b128 v[12:15], %[a1] offset:8192\n\t" \
    "v_mfma_f32_16x16x128_f8f6f4 a[36:39], v[72:79], v[16:23], a[36:39] cbsz:1 blgp:1\n\t" \
    "buffer_load_dwordx4 v[48:51], %[vq], %[rw], %[sw0] offen offset:2048 sc1\n\t" \
    "v_mfma_f32_16x16x128_f8f6f4 a[48:51], v[80:87], v[16:23], a[48:51] cbsz:1 blgp:1\n\t" \
    "buffer_load_dwordx4 v[52:55], %[vq], %[rw], %[sw1] offen offset:2048 sc1\n\t" \
    "v_mfma_f32_16x16x128_f8f6f4 a[52:55], v[88:95], v[16:23], a[52:55] cbsz:1 blgp:1\n\ts_waitcnt lgkmcnt(2)\n\t" \
    "v_mfma_f32_16x16x128_f8f6f4 a[40:43], v[64:71], v[24:31], a[40:43] cbsz:1 blgp:1\n\tds_read_b128 v[16:19], %[a0] offset:10240\n\t" \
    "ds_read_b128 v[20:23], %[a1] offset:10240\n\t" \
    "v_mfma_f32_16x16x128_f8f6f4 a[44:47], v[72:79], v[24:31], a[44:47] cbsz:1 blgp:1\n\t" \
    "buffer_load_dwordx4 v[56:59], %[vq], %[rw], %[sw0] offen offset:3072 sc1\n\t" \
    "v_mfma_f32_16x16x128_f8f6f4 a[56:59], v[80:87], v[24:31], a[56:59] cbsz:1 blgp:1\n\t" \
    "buffer_load_dwordx4 v[60:63], %[vq], %[rw], %[sw1] offen offset:3072 sc1\n\t" \
    "v_mfma_f32_16x16x128_f8f6f4 a[60:63], v[88:95], v[24:31], a[60:63] cbsz:1 blgp:1\n\ts_add_u32 %[sw0], %[sw0], 0x40000\n\t" \
    "s_add_u32 %[sw1], %[sw1], 0x40000\n\ts_waitcnt lgkmcnt(2)\n\t" \
    "v_mfma_f32_16x16x128_f8f6f4 a[64:67], v[64:71], v[8:15], a[64:67] cbsz:1 blgp:1\n\tds_read_b128 v[24:27], %[a0] offset:12288\n\t" \
    "ds_read_b128 v[28:31], %[a1] offset:12288\n\tds_read_b32 v6, %[vs]\n\t" \
    "v_mfma_f32_16x16x128_f8f6f4 a[68:71], v[72:79], v[8:15], a[68:71] cbsz:1 blgp:1\n\t" \
    "v_mfma_f32_16x16x128_f8f6f4 a[80:83], v[80:87], v[8:15], a[80:83] cbsz:1 blgp:1\n\t" \
    "v_mfma_f32_16x16x128_f8f6f4 a[84:87], v[88:95], v[8:15], a[84:87] cbsz:1 blgp:1\n\ts_waitcnt lgkmcnt(3)\n\t" \
    "v_mfma_f32_16x16x128_f8f6f4 a[72:75], v[64:71], v[16:23], a[72:75] cbsz:1 blgp:1\n\tds_read_b128 v[8:11], %[a0] offset:14336\n\t" \
    "ds_read_b128 v[12:15], %[a1] offset:14336\n\ts_mov_b64 exec, %[em]\n\tds_write_b32 %[vs], v7 offset:16\n\ts_mov_b64 exec, -1\n\t" \
    "v_mfma_f32_16x16x128_f8f6f4 a[76:79], v[72:79], v[16:23], a[76:79] cbsz:1 blgp:1\n\tv_add_u32 %[a0], %[dl], %[a0]\n\t" \
    "v_add_u32 %[a1], %[dl], %[a1]\n\t" \
    "v_mfma_f32_16x16x128_f8f6f4 a[88:91], v[80:87], v[16:23], a[88:91] cbsz:1 blgp:1\n\t" \
    "v_mfma_f32_16x16x128_f8f6f4 a[92:95], v[88:95], v[16:23], a[92:95] cbsz:1 blgp:1\n\ts_waitcnt lgkmcnt(3)\n\tv_cmp_lt_u32 vcc, %[kk], v6\n\t" \
    "v_mfma_f32_16x16x128_f8f6f4 a[96:99], v[64:71], v[24:31], a[96:99] cbsz:1 blgp:1\n\ts_cmp_eq_u64 vcc, -1\n\ts_cbranch_scc0 .Lslowb%=\n\t" \
    "ds_read_b128 v[16:19], %[a0] offset:0\n\tds_read_b128 v[20:23], %[a1] offset:0\n\t" \
    "v_mfma_f32_16x16x128_f8f6f4 a[100:103], v[72:79], v[24:31], a[100:103] cbsz:1 blgp:1\n\t" \
    "v_mfma_f32_16x16x128_f8f6f4 a[112:115], v[80:87], v[24:31], a[112:115] cbsz:1 blgp:1\n\t" \
    "v_mfma_f32_16x16x128_f8f6f4 a[116:119], v[88:95], v[24:31], a[116:119] cbsz:1 blgp:1\n\ts_waitcnt lgkmcnt(2)\n\t" \
    "v_mfma_f32_16x16x128_f8f6f4 a[104:107], v[64:71], v[8:15], a[104:107] cbsz:1 blgp:1\n\tds_read_b128 v[24:27], %[a0] offset:2048\n\t" \
    "ds_read_b128 v[28:31], %[a1] offset:2048\n\t" \
    "v_mfma_f32_16x16x128_f8f6f4 a[108:111], v[72:79], v[8:15], a[108:111] cbsz:1 blgp:1\n\t" \
    "v_mfma_f32_16x16x128_f8f6f4 a[120:123], v[80:87], v[8:15], a[120:123] cbsz:1 blgp:1\n\t" \
    "v_mfma_f32_16x16x128_f8f6f4 a[124:127], v[88:95], v[8:15], a[124:127] cbsz:1 blgp:1\n\t" \
      \
    ".Lsc%=:\n\ts_waitcnt vmcnt(8)\n\ts_waitcnt lgkmcnt(2)\n\t" \
    "v_mfma_f32_16x16x128_f8f6f4 a[0:3], v[96:103], v[16:23], a[0:3] cbsz:1 blgp:1\n\tds_read_b128 v[8:11], %[a0] offset:4096\n\t" \
    "ds_read_b128 v[12:15], %[a1] offset:4096\n\t" \
    "v_mfma_f32_16x16x128_f8f6f4 a[4:7], v[104:111], v[16:23], a[4:7] cbsz:1 blgp:1\n\t" \
    "buffer_load_dwordx4 v[64:67], %[vq], %[rw], %[sw0] offen offset:0 sc1\n\t" \
    "v_mfma_f32_16x16x128_f8f6f4 a[16:19], v[112:119], v[16:23], a[16:19] cbsz:1 blgp:1\n\t" \
    "buffer_load_dwordx4 v[68:71], %[vq], %[rw], %[sw1] offen offset:0 sc1\n\t" \
    "v_mfma_f32_16x16x128_f8f6f4 a[20:23], v[120:127], v[16:23], a[20:23] cbsz:1 blgp:1\n\ts_add_u32 %[sl], %[sl], 1\n\ts_cmp_eq_u32 %[sl], 7\n\t" \
    "s_cselect_b32 %[dl], %[dn], %[dp]\n\ts_cselect_b32 %[sl], 0, %[sl]\n\ts_waitcnt lgkmcnt(2)\n\t" \
    "v_mfma_f32_16x16x128_f8f6f4 a[8:11], v[96:103], v[24:31], a[8:11] cbsz:1 blgp:1\n\tds_read_b128 v[16:19], %[a0] offset:6144\n\t" \
    "ds_read_b128 v[20:23], %[a1] offset:6144\n\t" \
    "v_mfma_f32_16x16x128_f8f6f4 a[12:15], v[104:111], v[24:31], a[12:15] cbsz:1 blgp:1\n\t" \
    "buffer_load_dwordx4 v[72:75], %[vq], %[rw], %[sw0] offen offset:1024 sc1\n\t" \
    "v_mfma_f32_16x16x128_f8f6f4 a[24:27], v[112:119], v[24:31], a[24:27] cbsz:1 blgp:1\n\t" \
    "buffer_load_dwordx4 v[76:79], %[vq], %[rw], %[sw1] offen offset:1024 sc1\n\t" \
    "v_mfma_f32_16x16x128_f8f6f4 a[28:31], v[120:127], v[24:31], a[28:31] cbsz:1 blgp:1\n\ts_add_u32 %[kk], %[kk], 1\n\tv_mov_b32 v7, %[kk]\n\t" \
    "s_waitcnt lgkmcnt(2)\n\t" \
    "v_mfma_f32_16x16x128_f8f6f4 a[32:35], v[96:103], v[8:15], a[32:35] cbsz:1 blgp:1\n\tds_read_b128 v[24:27], %[a0] offset:8192\n\t" \
    "ds_read_b128 v[28:31], %[a1] offset:8192\n\t" \
    "v_mfma_f32_16x16x128_f8f6f4 a[36:39], v[104:111], v[8:15], a[36:39] cbsz:1 blgp:1\n\t" \
    "buffer_load_dwordx4 v[80:83], %[vq], %[rw], %[sw0] offen offset:2048 sc1\n\t" \
    "v_mfma_f32_16x16x128_f8f6f4 a[48:51], v[112:119], v[8:15], a[48:51] cbsz:1 blgp:1\n\t" \
    "buffer_load_dwordx4 v[84:87], %[vq], %[rw], %[sw1] offen offset:2048 sc1\n\t" \
    "v_mfma_f32_16x16x128_f8f6f4 a[52:55], v[120:127], v[8:15], a[52:55] cbsz:1 blgp:1\n\ts_waitcnt lgkmcnt(2)\n\t" \
    "v_mfma_f32_16x16x128_f8f6f4 a[40:43], v[96:103], v[16:23], a[40:43] cbsz:1 blgp:1\n\tds_read_b128 v[8:11], %[a0] offset:10240\n\t" \
    "ds_read_b128 v[12:15], %[a1] offset:10240\n\t" \
    "v_mfma_f32_16x16x128_f8f6f4 a[44:47], v[104:111], v[16:23], a[44:47] cbsz:1 blgp:1\n\t" \
    "buffer_load_dwordx4 v[88:91], %[vq], %[rw], %[sw0] offen offset:3072 sc1\n\t" \
    "v_mfma_f32_16x16x128_f8f6f4 a[56:59], v[112:119], v[16:23], a[56:59] cbsz:1 blgp:1\n\t" \
    "buffer_load_dwordx4 v[92:95], %[vq], %[rw], %[sw1] offen offset:3072 sc1\n\t" \
    "v_mfma_f32_16x16x128_f8f6f4 a[60:63], v[120:127], v[16:23], a[60:63] cbsz:1 blgp:1\n\ts_add_u32 %[sw0], %[sw0], 0x40000\n\t" \
    "s_add_u32 %[sw1], %[sw1], 0x40000\n\ts_waitcnt lgkmcnt(2)\n\t" \
    "v_mfma_f32_16x16x128_f8f6f4 a[64:67], v[96:103], v[24:31], a[64:67] cbsz:1 blgp:1\n\tds_read_b128 v[16:19], %[a0] offset:12288\n\t" \
    "ds_read_b128 v[20:23], %[a1] offset:12288\n\tds_read_b32 v6, %[vs]\n\t" \
    "v_mfma_f32_16x16x128_f8f6f4 a[68:71], v[104:111], v[24:31], a[68:71] cbsz:1 blgp:1\n\t" \
    "v_mfma_f32_16x16x128_f8f6f4 a[80:83], v[112:119], v[24:31], a[80:83] cbsz:1 blgp:1\n\t" \
    "v_mfma_f32_16x16x128_f8f6f4 a[84:87], v[120:127], v[24:31], a[84:87] cbsz:1 blgp:1\n\ts_waitcnt lgkmcnt(3)\n\t" \
    "v_mfma_f32_16x16x128_f8f6f4 a[72:75], v[96:103], v[8:15], a[72:75] cbsz:1 blgp:1\n\tds_read_b128 v[24:27], %[a0] offset:14336\n\t" \
    "ds_read_b128 v[28:31], %[a1] offset:14336\n\ts_mov_b64 exec, %[em]\n\tds_write_b32 %[vs], v7 offset:16\n\ts_mov_b64 exec, -1\n\t" \
    "v_mfma_f32_16x16x128_f8f6f4 a[76:79], v[104:111], v[8:15], a[76:79] cbsz:1 blgp:1\n\tv_add_u32 %[a0], %[dl], %[a0]\n\t" \
    "v_add_u32 %[a1], %[dl], %[a1]\n\t" \
    "v_mfma_f32_16x16x128_f8f6f4 a[88:91], v[112:119], v[8:15], a[88:91] cbsz:1 blgp:1\n\t" \
    "v_mfma_f32_16x16x128_f8f6f4 a[92:95], v[120:127], v[8:15], a[92:95] cbsz:1 blgp:1\n\ts_waitcnt lgkmcnt(3)\n\tv_cmp_lt_u32 vcc, %[kk], v6\n\t" \
    "v_mfma_f32_16x16x128_f8f6f4 a[96:99], v[96:103], v[16:23], a[96:99] cbsz:1 blgp:1\n\ts_cmp_eq_u64 vcc, -1\n\ts_cbranch_scc0 .Lslowc%=\n\t" \
    "ds_read_b128 v[8:11], %[a0] offset:0\n\tds_read_b128 v[12:15], %[a1] offset:0\n\t" \
    "v_mfma_f32_16x16x128_f8f6f4 a[100:103], v[104:111], v[16:23], a[100:103] cbsz:1 blgp:1\n\t" \
    "v_mfma_f32_16x16x128_f8f6f4 a[112:115], v[112:119], v[16:23], a[112:115] cbsz:1 blgp:1\n\t" \
    "v_mfma_f32_16x16x128_f8f6f4 a[116:119], v[120:127], v[16:23], a[116:119] cbsz:1 blgp:1\n\ts_waitcnt lgkmcnt(2)\n\t" \
    "v_mfma_f32_16x16x128_f8f6f4 a[104:107], v[96:103], v[24:31], a[104:107] cbsz:1 blgp:1\n\tds_read_b128 v[16:19], %[a0] offset:2048\n\t" \
    "ds_read_b128 v[20:23], %[a1] offset:2048\n\t" \
    "v_mfma_f32_16x16x128_f8f6f4 a[108:111], v[104:111], v[24:31], a[108:111] cbsz:1 blgp:1\n\t" \
    "v_mfma_f32_16x16x128_f8f6f4 a[120:123], v[112:119], v[24:31], a[120:123] cbsz:1 blgp:1\n\t" \
    "v_mfma_f32_16x16x128_f8f6f4 a[124:127], v[120:127], v[24:31], a[124:127] cbsz:1 blgp:1\n\ts_branch .Lsa%=\n\t" \
      \
    ".Lslowa%=:\n\t" \
    "v_mfma_f32_16x16x128_f8f6f4 a[100:103], v[40:47], v[8:15], a[100:103] cbsz:1 blgp:1\n\t" \
    "v_mfma_f32_16x16x128_f8f6f4 a[112:115], v[48:55], v[8:15], a[112:115] cbsz:1 blgp:1\n\t" \
    "v_mfma_f32_16x16x128_f8f6f4 a[116:119], v[56:63], v[8:15], a[116:119] cbsz:1 blgp:1\n\ts_waitcnt lgkmcnt(0)\n\t" \
    "v_mfma_f32_16x16x128_f8f6f4 a[104:107], v[32:39], v[16:23], a[104:107] cbsz:1 blgp:1\n\t" \
    "v_mfma_f32_16x16x128_f8f6f4 a[108:111], v[40:47], v[16:23], a[108:111] cbsz:1 blgp:1\n\t" \
    "v_mfma_f32_16x16x128_f8f6f4 a[120:123], v[48:55], v[16:23], a[120:123] cbsz:1 blgp:1\n\t" \
    "v_mfma_f32_16x16x128_f8f6f4 a[124:127], v[56:63], v[16:23], a[124:127] cbsz:1 blgp:1\n\ts_cmp_eq_u32 %[kk], 31\n\ts_cbranch_scc1 .Llast%=\n\t" \
    ".Lspina%=:\n\ts_sleep 1\n\tds_read_b32 v6, %[vs]\n\ts_add_u32 %[sp], %[sp], 1\n\ts_cmp_gt_u32 %[sp], 0x2000000\n\ts_cbranch_scc1 .Ldone%=\n\t" \
    "s_waitcnt lgkmcnt(0)\n\tv_cmp_lt_u32 vcc, %[kk], v6\n\ts_cmp_eq_u64 vcc, -1\n\ts_cbranch_scc0 .Lspina%=\n\t" \
    "ds_read_b128 v[24:27], %[a0] offset:0\n\tds_read_b128 v[28:31], %[a1] offset:0\n\tds_read_b128 v[8:11], %[a0] offset:2048\n\t" \
    "ds_read_b128 v[12:15], %[a1] offset:2048\n\ts_branch .Lsb%=\n\t" \
      \
    ".Llast%=:\n\ts_mov_b32 %[sw0], 15\n\tv_mov_b32 v7, 32\n\ts_waitcnt vmcnt(0)\n\t" \
    ".Llp%=:\n\ts_sleep 1\n\tds_read_b32 v6, %[vs]\n\ts_add_u32 %[sp], %[sp], 1\n\ts_cmp_gt_u32 %[sp], 0x2000000\n\ts_cbranch_scc1 .Ldone%=\n\t" \
    "s_waitcnt lgkmcnt(0)\n\tv_cmp_lt_u32 vcc, %[kk], v6\n\ts_and_b32 %[sw1], vcc_lo, %[sw0]\n\ts_cbranch_scc0 .Llp%=\n\ts_bitcmp1_b32 %[sw1], 0\n\t" \
    "s_cbranch_scc0 .Lra%=\n\tds_read_b128 v[32:35], %[a0] offset:0\n\tds_read_b128 v[36:39], %[a1] offset:0\n\t" \
    "ds_read_b128 v[40:43], %[a0] offset:2048\n\tds_read_b128 v[44:47], %[a1] offset:2048\n\t" \
    ".Lra%=:\n\ts_bitcmp1_b32 %[sw1], 1\n\ts_cbranch_scc0 .Lrb%=\n\tds_read_b128 v[48:51], %[a0] offset:4096\n\t" \
    "ds_read_b128 v[52:55], %[a1] offset:4096\n\tds_read_b128 v[56:59], %[a0] offset:6144\n\tds_read_b128 v[60:63], %[a1] offset:6144\n\t" \
    ".Lrb%=:\n\ts_bitcmp1_b32 %[sw1], 2\n\ts_cbranch_scc0 .Lrc%=\n\tds_read_b128 v[96:99], %[a0] offset:8192\n\t" \
    "ds_read_b128 v[100:103], %[a1] offset:8192\n\tds_read_b128 v[104:107], %[a0] offset:10240\n\tds_read_b128 v[108:111], %[a1] offset:10240\n\t" \
    ".Lrc%=:\n\ts_bitcmp1_b32 %[sw1], 3\n\ts_cbranch_scc0 .Lrd%=\n\tds_read_b128 v[112:115], %[a0] offset:12288\n\t" \
    "ds_read_b128 v[116:119], %[a1] offset:12288\n\tds_read_b128 v[120:123], %[a0] offset:14336\n\tds_read_b128 v[124:127], %[a1] offset:14336\n\t" \
    ".Lrd%=:\n\ts_andn2_b32 %[sw0], %[sw0], %[sw1]\n\ts_cbranch_scc1 .Lnr%=\n\ts_mov_b64 exec, %[em]\n\tds_write_b32 %[vs], v7 offset:16\n\t" \
    "s_mov_b64 exec, -1\n\t" \
    ".Lnr%=:\n\ts_waitcnt lgkmcnt(0)\n\ts_bitcmp1_b32 %[sw1], 0\n\ts_cbranch_scc0 .Lma%=\n\t" \
    "v_mfma_f32_16x16x128_f8f6f4 a[0:3], v[64:71], v[32:39], a[0:3] cbsz:1 blgp:1\n\t" \
    "v_mfma_f32_16x16x128_f8f6f4 a[4:7], v[72:79], v[32:39], a[4:7] cbsz:1 blgp:1\n\t" \
    "v_mfma_f32_16x16x128_f8f6f4 a[16:19], v[80:87], v[32:39], a[16:19] cbsz:1 blgp:1\n\t" \
    "v_mfma_f32_16x16x128_f8f6f4 a[20:23], v[88:95], v[32:39], a[20:23] cbsz:1 blgp:1\n\t" \
    "v_mfma_f32_16x16x128_f8f6f4 a[8:11], v[64:71], v[40:47], a[8:11] cbsz:1 blgp:1\n\t" \
    "v_mfma_f32_16x16x128_f8f6f4 a[12:15], v[72:79], v[40:47], a[12:15] cbsz:1 blgp:1\n\t" \
    "v_mfma_f32_16x16x128_f8f6f4 a[24:27], v[80:87], v[40:47], a[24:27] cbsz:1 blgp:1\n\t" \
    "v_mfma_f32_16x16x128_f8f6f4 a[28:31], v[88:95], v[40:47], a[28:31] cbsz:1 blgp:1\n\t" \
    ".Lma%=:\n\ts_bitcmp1_b32 %[sw1], 1\n\ts_cbranch_scc0 .Lmb%=\n\t" \
    "v_mfma_f32_16x16x128_f8f6f4 a[32:35], v[64:71], v[48:55], a[32:35] cbsz:1 blgp:1\n\t" \
    "v_mfma_f32_16x16x128_f8f6f4 a[36:39], v[72:79], v[48:55], a[36:39] cbsz:1 blgp:1\n\t" \
    "v_mfma_f32_16x16x128_f8f6f4 a[48:51], v[80:87], v[48:55], a[48:51] cbsz:1 blgp:1\n\t" \
    "v_mfma_f32_16x16x128_f8f6f4 a[52:55], v[88:95], v[48:55], a[52:55] cbsz:1 blgp:1\n\t" \
    "v_mfma_f32_16x16x128_f8f6f4 a[40:43], v[64:71], v[56:63], a[40:43] cbsz:1 blgp:1\n\t" \
    "v_mfma_f32_16x16x128_f8f6f4 a[44:47], v[72:79], v[56:63], a[44:47] cbsz:1 blgp:1\n\t" \
    "v_mfma_f32_16x16x128_f8f6f4 a[56:59], v[80:87], v[56:63], a[56:59] cbsz:1 blgp:1\n\t" \
    "v_mfma_f32_16x16x128_f8f6f4 a[60:63], v[88:95], v[56:63], a[60:63] cbsz:1 blgp:1\n\t" \
    ".Lmb%=:\n\ts_bitcmp1_b32 %[sw1], 2\n\ts_cbranch_scc0 .Lmc%=\n\t" \
    "v_mfma_f32_16x16x128_f8f6f4 a[64:67], v[64:71], v[96:103], a[64:67] cbsz:1 blgp:1\n\t" \
    "v_mfma_f32_16x16x128_f8f6f4 a[68:71], v[72:79], v[96:103], a[68:71] cbsz:1 blgp:1\n\t" \
    "v_mfma_f32_16x16x128_f8f6f4 a[80:83], v[80:87], v[96:103], a[80:83] cbsz:1 blgp:1\n\t" \
    "v_mfma_f32_16x16x128_f8f6f4 a[84:87], v[88:95], v[96:103], a[84:87] cbsz:1 blgp:1\n\t" \
    "v_mfma_f32_16x16x128_f8f6f4 a[72:75], v[64:71], v[104:111], a[72:75] cbsz:1 blgp:1\n\t" \
    "v_mfma_f32_16x16x128_f8f6f4 a[76:79], v[72:79], v[104:111], a[76:79] cbsz:1 blgp:1\n\t" \
    "v_mfma_f32_16x16x128_f8f6f4 a[88:91], v[80:87], v[104:111], a[88:91] cbsz:1 blgp:1\n\t" \
    "v_mfma_f32_16x16x128_f8f6f4 a[92:95], v[88:95], v[104:111], a[92:95] cbsz:1 blgp:1\n\t" \
    ".Lmc%=:\n\ts_bitcmp1_b32 %[sw1], 3\n\ts_cbranch_scc0 .Lmd%=\n\t" \
    "v_mfma_f32_16x16x128_f8f6f4 a[96:99], v[64:71], v[112:119], a[96:99] cbsz:1 blgp:1\n\t" \
    "v_mfma_f32_16x16x128_f8f6f4 a[100:103], v[72:79], v[112:119], a[100:103] cbsz:1 blgp:1\n\t" \
    "v_mfma_f32_16x16x128_f8f6f4 a[112:115], v[80:87], v[112:119], a[112:115] cbsz:1 blgp:1\n\t" \
    "v_mfma_f32_16x16x128_f8f6f4 a[116:119], v[88:95], v[112:119], a[116:119] cbsz:1 blgp:1\n\t" \
    "v_mfma_f32_16x16x128_f8f6f4 a[104:107], v[64:71], v[120:127], a[104:107] cbsz:1 blgp:1\n\t" \
    "v_mfma_f32_16x16x128_f8f6f4 a[108:111], v[72:79], v[120:127], a[108:111] cbsz:1 blgp:1\n\t" \
    "v_mfma_f32_16x16x128_f8f6f4 a[120:123], v[80:87], v[120:127], a[120:123] cbsz:1 blgp:1\n\t" \
    "v_mfma_f32_16x16x128_f8f6f4 a[124:127], v[88:95], v[120:127], a[124:127] cbsz:1 blgp:1\n\t" \
    ".Lmd%=:\n\ts_cmp_lg_u32 %[sw0], 0\n\ts_cbranch_scc1 .Llp%=\n\ts_mov_b32 %[kk], 32\n\ts_branch .Ldone%=\n\t" \
    ".Lslowb%=:\n\t" \
    "v_mfma_f32_16x16x128_f8f6f4 a[100:103], v[72:79], v[24:31], a[100:103] cbsz:1 blgp:1\n\t" \
    "v_mfma_f32_16x16x128_f8f6f4 a[112:115], v[80:87], v[24:31], a[112:115] cbsz:1 blgp:1\n\t" \
    "v_mfma_f32_16x16x128_f8f6f4 a[116:119], v[88:95], v[24:31], a[116:119] cbsz:1 blgp:1\n\ts_waitcnt lgkmcnt(0)\n\t" \
    "v_mfma_f32_16x16x128_f8f6f4 a[104:107], v[64:71], v[8:15], a[104:107] cbsz:1 blgp:1\n\t" \
    "v_mfma_f32_16x16x128_f8f6f4 a[108:111], v[72:79], v[8:15], a[108:111] cbsz:1 blgp:1\n\t" \
    "v_mfma_f32_16x16x128_f8f6f4 a[120:123], v[80:87], v[8:15], a[120:123] cbsz:1 blgp:1\n\t" \
    "v_mfma_f32_16x16x128_f8f6f4 a[124:127], v[88:95], v[8:15], a[124:127] cbsz:1 blgp:1\n\ts_cmp_ge_u32 %[kk], 32\n\ts_cbranch_scc1 .Ldone%=\n\t" \
    ".Lspinb%=:\n\ts_sleep 1\n\tds_read_b32 v6, %[vs]\n\ts_add_u32 %[sp], %[sp], 1\n\ts_cmp_gt_u32 %[sp], 0x2000000\n\ts_cbranch_scc1 .Ldone%=\n\t" \
    "s_waitcnt lgkmcnt(0)\n\tv_cmp_lt_u32 vcc, %[kk], v6\n\ts_cmp_eq_u64 vcc, -1\n\ts_cbranch_scc0 .Lspinb%=\n\t" \
    "ds_read_b128 v[16:19], %[a0] offset:0\n\tds_read_b128 v[20:23], %[a1] offset:0\n\tds_read_b128 v[24:27], %[a0] offset:2048\n\t" \
    "ds_read_b128 v[28:31], %[a1] offset:2048\n\ts_branch .Lsc%=\n\t" \
    ".Lslowc%=:\n\t" \
    "v_mfma_f32_16x16x128_f8f6f4 a[100:103], v[104:111], v[16:23], a[100:103] cbsz:1 blgp:1\n\t" \
    "v_mfma_f32_16x16x128_f8f6f4 a[112:115], v[112:119], v[16:23], a[112:115] cbsz:1 blgp:1\n\t" \
    "v_mfma_f32_16x16x128_f8f6f4 a[116:119], v[120:127], v[16:23], a[116:119] cbsz:1 blgp:1\n\ts_waitcnt lgkmcnt(0)\n\t" \
    "v_mfma_f32_16x16x128_f8f6f4 a[104:107], v[96:103], v[24:31], a[104:107] cbsz:1 blgp:1\n\t" \
    "v_mfma_f32_16x16x128_f8f6f4 a[108:111], v[104:111], v[24:31], a[108:111] cbsz:1 blgp:1\n\t" \
    "v_mfma_f32_16x16x128_f8f6f4 a[120:123], v[112:119], v[24:31], a[120:123] cbsz:1 blgp:1\n\t" \
    "v_mfma_f32_16x16x128_f8f6f4 a[124:127], v[120:127], v[24:31], a[124:127] cbsz:1 blgp:1\n\t" \
    ".Lspinc%=:\n\ts_sleep 1\n\tds_read_b32 v6, %[vs]\n\ts_add_u32 %[sp], %[sp], 1\n\ts_cmp_gt_u32 %[sp], 0x2000000\n\ts_cbranch_scc1 .Ldone%=\n\t" \
    "s_waitcnt lgkmcnt(0)\n\tv_cmp_lt_u32 vcc, %[kk], v6\n\ts_cmp_eq_u64 vcc, -1\n\ts_cbranch_scc0 .Lspinc%=\n\t" \
    "ds_read_b128 v[8:11], %[a0] offset:0\n\tds_read_b128 v[12:15], %[a1] offset:0\n\tds_read_b128 v[16:19], %[a0] offset:2048\n\t" \
    "ds_read_b128 v[20:23], %[a1] offset:2048\n\ts_branch .Lsa%=\n\t" \
      \
    ".Ldone%=:\n\tds_read_b32 v6, %[vs] offset:16\n\tds_read_b32 v7, %[vs] offset:32\n\tv_and_b32 %[a0], 0xf0, %[vq]\n\t" \
    "v_lshlrev_b32 %[a0], 3, %[a0]\n\tv_lshrrev_b32 %[a1], 4, %[vq]\n\tv_xor_b32 %[a1], %[a1], %[vq]\n\tv_and_b32 %[a1], 0x70, %[a1]\n\t" \
    "v_add3_u32 %[a0], %[a0], %[a1], %[db]\n\tv_xor_b32 %[a1], 64, %[a0]\n\ts_add_u32 %[sp], %[sp], 1\n\ts_nop 4\n\ts_waitcnt vmcnt(0) lgkmcnt(0)\n\t" \
    "v_cmp_le_u32 vcc, 32, v6\n\tv_readfirstlane_b32 %[dl], v7\n\tv_readlane_b32 %[sw1], v7, 3\n\ts_cmp_gt_u32 %[sp], 0x2000000\n\t" \
    "s_cbranch_scc1 .Lgo%=\n\ts_cmp_eq_u64 vcc, -1\n\ts_cbranch_scc0 .Ldone%=\n\t" \
    ".Lgo%=:\n\ts_cmp_eq_u32 %[kk], 32\n\ts_cbranch_scc1 .Lok%=\n\tv_mov_b32 v6, 1\n\ts_mov_b64 exec, 1\n\tds_write_b32 %[vs], v6 offset:32\n\t" \
    "s_mov_b64 exec, -1\n\ts_mov_b32 %[dl], 1\n\t" \
    ".Lok%=:\n\tds_write_b128 %[a0], a[0:3] offset:0\n\tds_write_b128 %[a1], a[4:7] offset:0\n\tds_write_b128 %[a0], a[8:11] offset:2048\n\t" \
    "ds_write_b128 %[a1], a[12:15] offset:2048\n\tv_mov_b32 v6, 1\n\ts_mov_b64 exec, %[em]\n\tds_write_b32 %[vs], v6 offset:48\n\t" \
    "s_mov_b64 exec, -1\n\tds_write_b128 %[a0], a[16:19] offset:4096\n\tds_write_b128 %[a1], a[20:23] offset:4096\n\t" \
    "ds_write_b128 %[a0], a[24:27] offset:6144\n\tds_write_b128 %[a1], a[28:31] offset:6144\n\tv_mov_b32 v6, 2\n\ts_mov_b64 exec, %[em]\n\t" \
    "ds_write_b32 %[vs], v6 offset:48\n\ts_mov_b64 exec, -1\n\t" \
    ""

#define HELPER_ASM \
    "s_mov_b32 s66, m0\n\ts_and_b32 s67, %[hq], 1\n\ts_lshl_b32 s67, s67, 3\n\ts_add_u32 s67, s67, 8\n\ts_mov_b32 s68, 0x60606060\n\t" \
    "s_add_u32 s69, %[l0], 114688\n\tv_lshrrev_b32 v61, 3, %[vq]\n\tv_mov_b32 v62, 0xffc\n\tv_lshl_add_u32 v63, %[hq], 2, %[vs]\n\t" \
    "v_mov_b32 v49, %[fm]\n\tds_read_b128 v[40:43], %[vs] offset:16\n\t" \
      \
    "s_cmp_lg_u32 %[q], 0\n\ts_cbranch_scc1 .LGo%=\n\ts_movk_i32 s63, 64\n\ts_branch .LS1%=\n\t" \
    ".LS0%=:\n\ts_sub_u32 s63, s63, 1\n\ts_cbranch_scc1 .Lend%=\n\ts_sleep 2\n\t" \
    ".LS1%=:\n\tglobal_load_dwordx2 v[48:49], %[fm], %[fp] sc0 sc1\n\ts_waitcnt vmcnt(0)\n\tv_cmp_le_u32 vcc, %[call], v48\n\t" \
    "v_readlane_b32 s64, v48, 63\n\ts_cmp_eq_u64 vcc, -1\n\ts_cbranch_scc0 .LS0%=\n\ts_cmp_eq_u32 s64, %[pub]\n\ts_cbranch_scc0 .LS0%=\n\t" \
      \
    ".LGo%=:\n\tbuffer_load_dwordx4 v[64:67], %[vq], %[ra], %[sa] offen offset:0\n\t" \
    "buffer_load_dwordx4 v[68:71], %[vq], %[ra], %[sa] offen offset:1024\n\tbuffer_load_dwordx4 v[72:75], %[vq], %[ra], %[sa] offen offset:2048\n\t" \
    "buffer_load_ushort v76, v61, %[ra], %[sa] offen offset:3072\n\tv_lshrrev_b32 v44, s67, v49\n\tv_and_b32 v60, 0x7f, v49\n\t" \
    "v_and_b32 v44, 0xff, v44\n\tv_lshl_or_b32 v60, v44, 8, v60\n\ts_nop 0\n\tv_readlane_b32 s60, v60, %[k1]\n\ts_nop 0\n\t" \
    "s_sub_u32 s64, s60, 0x4100\n\ts_cmp_lt_u32 s64, 0x4000\n\ts_cselect_b32 s64, 0x80, 0\n\ts_add_u32 s64, s64, %[sa]\n\t" \
    "buffer_load_ushort v77, v61, %[ra], s64 offen offset:3072\n\ts_add_u32 %[sa], %[sa], 0xd00\n\ts_add_u32 %[k1], %[k1], 1\n\t" \
    "s_cmp_gt_u32 %[q], 1\n\ts_cbranch_scc1 .LQa%=\n\ts_sleep 6\n\tglobal_load_dwordx2 v[48:49], %[vb], %[fp] sc0 sc1\n\t" \
    ".LQa%=:\n\tds_write_b32 %[vs], v62 offset:72\n\ts_bitcmp1_b32 %[ld], %[k1]\n\ts_cbranch_scc0 .LPa%=\n\tv_readlane_b32 s61, v60, %[k1]\n\t" \
    "s_lshl_b32 s65, %[k1], 12\n\ts_add_u32 s65, s65, %[sr]\n\ts_cmp_gt_u32 s61, 0x80ff\n\ts_cselect_b32 s65, s65, %[sa]\n\t" \
    "s_sub_u32 s64, s61, 0x4100\n\ts_cmp_lt_u32 s64, 0x4000\n\ts_cselect_b32 s64, 0x80, 0\n\ts_add_u32 s64, s64, s65\n\t" \
    "buffer_load_dwordx4 v[80:83], %[vq], %[ra], s65 offen offset:0\n\tbuffer_load_dwordx4 v[84:87], %[vq], %[ra], s65 offen offset:1024\n\t" \
    "buffer_load_dwordx4 v[88:91], %[vq], %[ra], s65 offen offset:2048\n\tbuffer_load_ushort v92, v61, %[ra], s65 offen offset:3072\n\t" \
    "buffer_load_ushort v93, v61, %[ra], s64 offen offset:3072\n\ts_waitcnt vmcnt(5)\n\ts_branch .LPb%=\n\t" \
    ".LPa%=:\n\ts_waitcnt vmcnt(0)\n\t" \
    ".LPb%=:\n\ts_cmp_gt_u32 %[q], 1\n\ts_cbranch_scc1 .Lloop%=\n\tv_cmp_le_u32 vcc, %[call], v48\n\ts_cmp_eq_u64 vcc, -1\n\t" \
    "s_cbranch_scc0 .Lend%=\n\tv_lshrrev_b32 v44, s67, v49\n\tv_and_b32 v60, 0x7f, v49\n\tv_and_b32 v44, 0xff, v44\n\t" \
    "v_lshl_or_b32 v60, v44, 8, v60\n\t" \
      \
    ".Lloop%=:\n\tv_add_u32 v56, %[lc], %[vq]\n\tv_mov_b32 v47, %[k1]\n\ts_bitcmp1_b32 %[ld], %[k1]\n\ts_cbranch_scc1 .LWa%=\n\t" \
    "s_waitcnt vmcnt(0) lgkmcnt(0)\n\tv_readlane_b32 s61, v60, %[k1]\n\ts_lshl_b32 s65, %[k1], 12\n\ts_add_u32 s65, s65, %[sr]\n\t" \
    "s_cmp_gt_u32 s61, 0x80ff\n\ts_cselect_b32 s65, s65, %[sa]\n\ts_sub_u32 s64, s61, 0x4100\n\ts_cmp_lt_u32 s64, 0x4000\n\t" \
    "s_cselect_b32 s64, 0x80, 0\n\ts_add_u32 s64, s64, s65\n\tbuffer_load_dwordx4 v[80:83], %[vq], %[ra], s65 offen offset:0\n\t" \
    "buffer_load_dwordx4 v[84:87], %[vq], %[ra], s65 offen offset:1024\n\tbuffer_load_dwordx4 v[88:91], %[vq], %[ra], s65 offen offset:2048\n\t" \
    "buffer_load_ushort v92, v61, %[ra], s65 offen offset:3072\n\tbuffer_load_ushort v93, v61, %[ra], s64 offen offset:3072\n\t" \
    ".LWBa%=:\n\ts_add_u32 %[sa], %[sa], 0xd00\n\ts_add_u32 s63, %[k1], 1\n\ts_bitcmp1_b32 %[ld], s63\n\ts_cbranch_scc1 .LIa%=\n\t" \
    ".LIBa%=:\n\t" \
    ".LFCa%=:\n\tv_min_u32 v40, v40, v41\n\tv_min3_u32 v40, v40, v42, v43\n\ts_and_b32 s63, s60, 0x7f\n\tv_readfirstlane_b32 s64, v40\n\t" \
    "s_mul_i32 s63, s63, 0x1010101\n\ts_add_u32 s64, s64, 7\n\ts_cmp_ge_u32 s64, %[k1]\n\ts_cbranch_scc0 .LFa%=\n\ts_cmp_gt_u32 s60, 0x80ff\n\t" \
    "s_cbranch_scc1 .LRa%=\n\ts_add_u32 s64, s63, 0x90801000\n\tv_and_b32 v112, s68, v64\n\tv_and_b32 v116, s68, v68\n\tv_and_b32 v120, s68, v72\n\t" \
    "v_and_b32 v113, s68, v65\n\tv_and_b32 v117, s68, v69\n\tv_and_b32 v121, s68, v73\n\tv_and_b32 v114, s68, v66\n\tv_and_b32 v118, s68, v70\n\t" \
    "v_and_b32 v122, s68, v74\n\tv_and_b32 v115, s68, v67\n\tv_and_b32 v119, s68, v71\n\tv_and_b32 v123, s68, v75\n\t" \
    "v_xad_u32 v64, v64, v112, s63\n\tv_xad_u32 v65, v65, v113, s63\n\tv_xad_u32 v66, v66, v114, s63\n\tv_xad_u32 v67, v67, v115, s63\n\t" \
    "ds_write_b128 v56, v[64:67] offset:0\n\tv_xad_u32 v68, v68, v116, s63\n\tv_xad_u32 v69, v69, v117, s63\n\tv_xad_u32 v70, v70, v118, s63\n\t" \
    "v_xad_u32 v71, v71, v119, s63\n\tds_write_b128 v56, v[68:71] offset:1024\n\tv_xad_u32 v72, v72, v120, s63\n\tv_xad_u32 v73, v73, v121, s63\n\t" \
    "v_xad_u32 v74, v74, v122, s63\n\tv_xad_u32 v75, v75, v123, s63\n\tds_write_b128 v56, v[72:75] offset:2048\n\tv_lshrrev_b32 v120, 5, v120\n\t" \
    "v_lshrrev_b32 v121, 5, v121\n\tv_lshrrev_b32 v122, 5, v122\n\tv_lshrrev_b32 v123, 5, v123\n\tv_lshrrev_b32 v112, 5, v112\n\t" \
    "v_lshrrev_b32 v113, 5, v113\n\tv_lshrrev_b32 v114, 5, v114\n\tv_lshrrev_b32 v115, 5, v115\n\tv_perm_b32 v120, s64, s64, v120\n\t" \
    "v_perm_b32 v121, s64, s64, v121\n\tv_perm_b32 v122, s64, s64, v122\n\tv_perm_b32 v123, s64, s64, v123\n\tv_lshrrev_b32 v116, 3, v116\n\t" \
    "v_lshrrev_b32 v117, 3, v117\n\tv_lshrrev_b32 v118, 3, v118\n\tv_lshrrev_b32 v119, 3, v119\n\tv_add3_u32 v124, v112, v116, v120\n\t" \
    "v_add3_u32 v125, v113, v117, v121\n\tv_add3_u32 v126, v114, v118, v122\n\tv_add3_u32 v127, v115, v119, v123\n\t" \
    "ds_write_b128 v56, v[124:127] offset:3072\n\tv_bfe_i32 v44, v76, 13, 3\n\tv_lshl_add_u32 v45, v76, 3, 5\n\tv_and_or_b32 v46, v76, v62, %[lc]\n\t" \
    "v_lshlrev_b32 v44, v45, v44\n\tds_add_u32 v46, v44\n\ts_cmp_gt_u32 s60, 0x40ff\n\ts_cbranch_scc1 .LXa%=\n\t" \
    ".LXBa%=:\n\tds_write_b32 v63, v47\n\tds_read_b128 v[40:43], %[vs] offset:16\n\ts_add_u32 %[lc], %[lc], 0x4000\n\ts_cmp_eq_u32 %[lc], s69\n\t" \
    "s_cselect_b32 %[lc], %[l0], %[lc]\n\ts_add_u32 %[k1], %[k1], 1\n\t" \
      \
    "v_add_u32 v56, %[lc], %[vq]\n\tv_mov_b32 v47, %[k1]\n\ts_bitcmp1_b32 %[ld], %[k1]\n\ts_cbranch_scc1 .LWb%=\n\ts_waitcnt vmcnt(0) lgkmcnt(0)\n\t" \
    "s_cmp_eq_u32 %[ck], 1\n\ts_cbranch_scc1 .LKx%=\n\t" \
    ".LKBx%=:\n\tv_readlane_b32 s62, v60, %[k1]\n\ts_lshl_b32 s65, %[k1], 12\n\ts_add_u32 s65, s65, %[sr]\n\ts_cmp_gt_u32 s62, 0x80ff\n\t" \
    "s_cselect_b32 s65, s65, %[sa]\n\ts_sub_u32 s64, s62, 0x4100\n\ts_cmp_lt_u32 s64, 0x4000\n\ts_cselect_b32 s64, 0x80, 0\n\t" \
    "s_add_u32 s64, s64, s65\n\tbuffer_load_dwordx4 v[96:99], %[vq], %[ra], s65 offen offset:0\n\t" \
    "buffer_load_dwordx4 v[100:103], %[vq], %[ra], s65 offen offset:1024\n\tbuffer_load_dwordx4 v[104:107], %[vq], %[ra], s65 offen offset:2048\n\t" \
    "buffer_load_ushort v108, v61, %[ra], s65 offen offset:3072\n\tbuffer_load_ushort v109, v61, %[ra], s64 offen offset:3072\n\t" \
    ".LWBb%=:\n\ts_add_u32 %[sa], %[sa], 0xd00\n\ts_add_u32 s63, %[k1], 1\n\ts_bitcmp1_b32 %[ld], s63\n\ts_cbranch_scc1 .LIb%=\n\t" \
    ".LIBb%=:\n\t" \
    ".LFCb%=:\n\tv_min_u32 v40, v40, v41\n\tv_min3_u32 v40, v40, v42, v43\n\ts_and_b32 s63, s61, 0x7f\n\tv_readfirstlane_b32 s64, v40\n\t" \
    "s_mul_i32 s63, s63, 0x1010101\n\ts_add_u32 s64, s64, 7\n\ts_cmp_ge_u32 s64, %[k1]\n\ts_cbranch_scc0 .LFb%=\n\ts_cmp_gt_u32 s61, 0x80ff\n\t" \
    "s_cbranch_scc1 .LRb%=\n\ts_add_u32 s64, s63, 0x90801000\n\tv_and_b32 v112, s68, v80\n\tv_and_b32 v116, s68, v84\n\tv_and_b32 v120, s68, v88\n\t" \
    "v_and_b32 v113, s68, v81\n\tv_and_b32 v117, s68, v85\n\tv_and_b32 v121, s68, v89\n\tv_and_b32 v114, s68, v82\n\tv_and_b32 v118, s68, v86\n\t" \
    "v_and_b32 v122, s68, v90\n\tv_and_b32 v115, s68, v83\n\tv_and_b32 v119, s68, v87\n\tv_and_b32 v123, s68, v91\n\t" \
    "v_xad_u32 v80, v80, v112, s63\n\tv_xad_u32 v81, v81, v113, s63\n\tv_xad_u32 v82, v82, v114, s63\n\tv_xad_u32 v83, v83, v115, s63\n\t" \
    "ds_write_b128 v56, v[80:83] offset:0\n\tv_xad_u32 v84, v84, v116, s63\n\tv_xad_u32 v85, v85, v117, s63\n\tv_xad_u32 v86, v86, v118, s63\n\t" \
    "v_xad_u32 v87, v87, v119, s63\n\tds_write_b128 v56, v[84:87] offset:1024\n\tv_xad_u32 v88, v88, v120, s63\n\tv_xad_u32 v89, v89, v121, s63\n\t" \
    "v_xad_u32 v90, v90, v122, s63\n\tv_xad_u32 v91, v91, v123, s63\n\tds_write_b128 v56, v[88:91] offset:2048\n\tv_lshrrev_b32 v120, 5, v120\n\t" \
    "v_lshrrev_b32 v121, 5, v121\n\tv_lshrrev_b32 v122, 5, v122\n\tv_lshrrev_b32 v123, 5, v123\n\tv_lshrrev_b32 v112, 5, v112\n\t" \
    "v_lshrrev_b32 v113, 5, v113\n\tv_lshrrev_b32 v114, 5, v114\n\tv_lshrrev_b32 v115, 5, v115\n\tv_perm_b32 v120, s64, s64, v120\n\t" \
    "v_perm_b32 v121, s64, s64, v121\n\tv_perm_b32 v122, s64, s64, v122\n\tv_perm_b32 v123, s64, s64, v123\n\tv_lshrrev_b32 v116, 3, v116\n\t" \
    "v_lshrrev_b32 v117, 3, v117\n\tv_lshrrev_b32 v118, 3, v118\n\tv_lshrrev_b32 v119, 3, v119\n\tv_add3_u32 v124, v112, v116, v120\n\t" \
    "v_add3_u32 v125, v113, v117, v121\n\tv_add3_u32 v126, v114, v118, v122\n\tv_add3_u32 v127, v115, v119, v123\n\t" \
    "ds_write_b128 v56, v[124:127] offset:3072\n\tv_bfe_i32 v44, v92, 13, 3\n\tv_lshl_add_u32 v45, v92, 3, 5\n\tv_and_or_b32 v46, v92, v62, %[lc]\n\t" \
    "v_lshlrev_b32 v44, v45, v44\n\tds_add_u32 v46, v44\n\ts_cmp_gt_u32 s61, 0x40ff\n\ts_cbranch_scc1 .LXb%=\n\t" \
    ".LXBb%=:\n\tds_write_b32 v63, v47\n\tds_read_b128 v[40:43], %[vs] offset:16\n\ts_add_u32 %[lc], %[lc], 0x4000\n\ts_cmp_eq_u32 %[lc], s69\n\t" \
    "s_cselect_b32 %[lc], %[l0], %[lc]\n\ts_add_u32 %[k1], %[k1], 1\n\t" \
      \
    "v_add_u32 v56, %[lc], %[vq]\n\tv_mov_b32 v47, %[k1]\n\ts_bitcmp1_b32 %[ld], %[k1]\n\ts_cbranch_scc1 .LWc%=\n\ts_waitcnt vmcnt(0) lgkmcnt(0)\n\t" \
    "v_readlane_b32 s60, v60, %[k1]\n\ts_lshl_b32 s65, %[k1], 12\n\ts_add_u32 s65, s65, %[sr]\n\ts_cmp_gt_u32 s60, 0x80ff\n\t" \
    "s_cselect_b32 s65, s65, %[sa]\n\ts_sub_u32 s64, s60, 0x4100\n\ts_cmp_lt_u32 s64, 0x4000\n\ts_cselect_b32 s64, 0x80, 0\n\t" \
    "s_add_u32 s64, s64, s65\n\tbuffer_load_dwordx4 v[64:67], %[vq], %[ra], s65 offen offset:0\n\t" \
    "buffer_load_dwordx4 v[68:71], %[vq], %[ra], s65 offen offset:1024\n\tbuffer_load_dwordx4 v[72:75], %[vq], %[ra], s65 offen offset:2048\n\t" \
    "buffer_load_ushort v76, v61, %[ra], s65 offen offset:3072\n\tbuffer_load_ushort v77, v61, %[ra], s64 offen offset:3072\n\t" \
    ".LWBc%=:\n\ts_add_u32 %[sa], %[sa], 0xd00\n\ts_add_u32 s63, %[k1], 1\n\ts_bitcmp1_b32 %[ld], s63\n\ts_cbranch_scc1 .LIc%=\n\t" \
    ".LIBc%=:\n\t" \
    ".LFCc%=:\n\tv_min_u32 v40, v40, v41\n\tv_min3_u32 v40, v40, v42, v43\n\ts_and_b32 s63, s62, 0x7f\n\tv_readfirstlane_b32 s64, v40\n\t" \
    "s_mul_i32 s63, s63, 0x1010101\n\ts_add_u32 s64, s64, 7\n\ts_cmp_ge_u32 s64, %[k1]\n\ts_cbranch_scc0 .LFc%=\n\ts_cmp_gt_u32 s62, 0x80ff\n\t" \
    "s_cbranch_scc1 .LRc%=\n\ts_add_u32 s64, s63, 0x90801000\n\tv_and_b32 v112, s68, v96\n\tv_and_b32 v116, s68, v100\n\t" \
    "v_and_b32 v120, s68, v104\n\tv_and_b32 v113, s68, v97\n\tv_and_b32 v117, s68, v101\n\tv_and_b32 v121, s68, v105\n\tv_and_b32 v114, s68, v98\n\t" \
    "v_and_b32 v118, s68, v102\n\tv_and_b32 v122, s68, v106\n\tv_and_b32 v115, s68, v99\n\tv_and_b32 v119, s68, v103\n\tv_and_b32 v123, s68, v107\n\t" \
    "v_xad_u32 v96, v96, v112, s63\n\tv_xad_u32 v97, v97, v113, s63\n\tv_xad_u32 v98, v98, v114, s63\n\tv_xad_u32 v99, v99, v115, s63\n\t" \
    "ds_write_b128 v56, v[96:99] offset:0\n\tv_xad_u32 v100, v100, v116, s63\n\tv_xad_u32 v101, v101, v117, s63\n\t" \
    "v_xad_u32 v102, v102, v118, s63\n\tv_xad_u32 v103, v103, v119, s63\n\tds_write_b128 v56, v[100:103] offset:1024\n\t" \
    "v_xad_u32 v104, v104, v120, s63\n\tv_xad_u32 v105, v105, v121, s63\n\tv_xad_u32 v106, v106, v122, s63\n\tv_xad_u32 v107, v107, v123, s63\n\t" \
    "ds_write_b128 v56, v[104:107] offset:2048\n\tv_lshrrev_b32 v120, 5, v120\n\tv_lshrrev_b32 v121, 5, v121\n\tv_lshrrev_b32 v122, 5, v122\n\t" \
    "v_lshrrev_b32 v123, 5, v123\n\tv_lshrrev_b32 v112, 5, v112\n\tv_lshrrev_b32 v113, 5, v113\n\tv_lshrrev_b32 v114, 5, v114\n\t" \
    "v_lshrrev_b32 v115, 5, v115\n\tv_perm_b32 v120, s64, s64, v120\n\tv_perm_b32 v121, s64, s64, v121\n\tv_perm_b32 v122, s64, s64, v122\n\t" \
    "v_perm_b32 v123, s64, s64, v123\n\tv_lshrrev_b32 v116, 3, v116\n\tv_lshrrev_b32 v117, 3, v117\n\tv_lshrrev_b32 v118, 3, v118\n\t" \
    "v_lshrrev_b32 v119, 3, v119\n\tv_add3_u32 v124, v112, v116, v120\n\tv_add3_u32 v125, v113, v117, v121\n\tv_add3_u32 v126, v114, v118, v122\n\t" \
    "v_add3_u32 v127, v115, v119, v123\n\tds_write_b128 v56, v[124:127] offset:3072\n\tv_bfe_i32 v44, v108, 13, 3\n\t" \
    "v_lshl_add_u32 v45, v108, 3, 5\n\tv_and_or_b32 v46, v108, v62, %[lc]\n\tv_lshlrev_b32 v44, v45, v44\n\tds_add_u32 v46, v44\n\t" \
    "s_cmp_gt_u32 s62, 0x40ff\n\ts_cbranch_scc1 .LXc%=\n\t" \
    ".LXBc%=:\n\tds_write_b32 v63, v47\n\tds_read_b128 v[40:43], %[vs] offset:16\n\ts_add_u32 %[lc], %[lc], 0x4000\n\ts_cmp_eq_u32 %[lc], s69\n\t" \
    "s_cselect_b32 %[lc], %[l0], %[lc]\n\ts_add_u32 %[k1], %[k1], 1\n\ts_cmp_lg_u32 %[ck], 0\n\ts_cbranch_scc1 .LC%=\n\t" \
    ".LCB%=:\n\ts_cmp_lt_u32 %[k1], 31\n\ts_cbranch_scc1 .Lloop%=\n\t" \
      \
    "v_add_u32 v56, %[lc], %[vq]\n\tv_mov_b32 v47, %[k1]\n\ts_bitcmp1_b32 %[ld], %[k1]\n\ts_cbranch_scc1 .LWd%=\n\ts_waitcnt vmcnt(0) lgkmcnt(0)\n\t" \
    "v_readlane_b32 s61, v60, %[k1]\n\ts_lshl_b32 s65, %[k1], 12\n\ts_add_u32 s65, s65, %[sr]\n\ts_cmp_gt_u32 s61, 0x80ff\n\t" \
    "s_cselect_b32 s65, s65, %[sa]\n\ts_sub_u32 s64, s61, 0x4100\n\ts_cmp_lt_u32 s64, 0x4000\n\ts_cselect_b32 s64, 0x80, 0\n\t" \
    "s_add_u32 s64, s64, s65\n\tbuffer_load_dwordx4 v[80:83], %[vq], %[ra], s65 offen offset:0\n\t" \
    "buffer_load_dwordx4 v[84:87], %[vq], %[ra], s65 offen offset:1024\n\tbuffer_load_dwordx4 v[88:91], %[vq], %[ra], s65 offen offset:2048\n\t" \
    "buffer_load_ushort v92, v61, %[ra], s65 offen offset:3072\n\tbuffer_load_ushort v93, v61, %[ra], s64 offen offset:3072\n\t" \
    ".LWBd%=:\n\ts_add_u32 %[sa], %[sa], 0xd00\n\t" \
    ".LFCd%=:\n\tv_min_u32 v40, v40, v41\n\tv_min3_u32 v40, v40, v42, v43\n\ts_and_b32 s63, s60, 0x7f\n\tv_readfirstlane_b32 s64, v40\n\t" \
    "s_mul_i32 s63, s63, 0x1010101\n\ts_add_u32 s64, s64, 7\n\ts_cmp_ge_u32 s64, %[k1]\n\ts_cbranch_scc0 .LFd%=\n\ts_cmp_gt_u32 s60, 0x80ff\n\t" \
    "s_cbranch_scc1 .LRd%=\n\ts_add_u32 s64, s63, 0x90801000\n\tv_and_b32 v112, s68, v64\n\tv_and_b32 v116, s68, v68\n\tv_and_b32 v120, s68, v72\n\t" \
    "v_and_b32 v113, s68, v65\n\tv_and_b32 v117, s68, v69\n\tv_and_b32 v121, s68, v73\n\tv_and_b32 v114, s68, v66\n\tv_and_b32 v118, s68, v70\n\t" \
    "v_and_b32 v122, s68, v74\n\tv_and_b32 v115, s68, v67\n\tv_and_b32 v119, s68, v71\n\tv_and_b32 v123, s68, v75\n\t" \
    "v_xad_u32 v64, v64, v112, s63\n\tv_xad_u32 v65, v65, v113, s63\n\tv_xad_u32 v66, v66, v114, s63\n\tv_xad_u32 v67, v67, v115, s63\n\t" \
    "ds_write_b128 v56, v[64:67] offset:0\n\tv_xad_u32 v68, v68, v116, s63\n\tv_xad_u32 v69, v69, v117, s63\n\tv_xad_u32 v70, v70, v118, s63\n\t" \
    "v_xad_u32 v71, v71, v119, s63\n\tds_write_b128 v56, v[68:71] offset:1024\n\tv_xad_u32 v72, v72, v120, s63\n\tv_xad_u32 v73, v73, v121, s63\n\t" \
    "v_xad_u32 v74, v74, v122, s63\n\tv_xad_u32 v75, v75, v123, s63\n\tds_write_b128 v56, v[72:75] offset:2048\n\tv_lshrrev_b32 v120, 5, v120\n\t" \
    "v_lshrrev_b32 v121, 5, v121\n\tv_lshrrev_b32 v122, 5, v122\n\tv_lshrrev_b32 v123, 5, v123\n\tv_lshrrev_b32 v112, 5, v112\n\t" \
    "v_lshrrev_b32 v113, 5, v113\n\tv_lshrrev_b32 v114, 5, v114\n\tv_lshrrev_b32 v115, 5, v115\n\tv_perm_b32 v120, s64, s64, v120\n\t" \
    "v_perm_b32 v121, s64, s64, v121\n\tv_perm_b32 v122, s64, s64, v122\n\tv_perm_b32 v123, s64, s64, v123\n\tv_lshrrev_b32 v116, 3, v116\n\t" \
    "v_lshrrev_b32 v117, 3, v117\n\tv_lshrrev_b32 v118, 3, v118\n\tv_lshrrev_b32 v119, 3, v119\n\tv_add3_u32 v124, v112, v116, v120\n\t" \
    "v_add3_u32 v125, v113, v117, v121\n\tv_add3_u32 v126, v114, v118, v122\n\tv_add3_u32 v127, v115, v119, v123\n\t" \
    "ds_write_b128 v56, v[124:127] offset:3072\n\tv_bfe_i32 v44, v76, 13, 3\n\tv_lshl_add_u32 v45, v76, 3, 5\n\tv_and_or_b32 v46, v76, v62, %[lc]\n\t" \
    "v_lshlrev_b32 v44, v45, v44\n\tds_add_u32 v46, v44\n\ts_cmp_gt_u32 s60, 0x40ff\n\ts_cbranch_scc1 .LXd%=\n\t" \
    ".LXBd%=:\n\tds_write_b32 v63, v47\n\tds_read_b128 v[40:43], %[vs] offset:16\n\ts_add_u32 %[lc], %[lc], 0x4000\n\ts_cmp_eq_u32 %[lc], s69\n\t" \
    "s_cselect_b32 %[lc], %[l0], %[lc]\n\ts_add_u32 %[k1], %[k1], 1\n\t" \
      \
    "v_add_u32 v56, %[lc], %[vq]\n\tv_mov_b32 v47, %[k1]\n\ts_waitcnt vmcnt(0) lgkmcnt(0)\n\t" \
    ".LFCe%=:\n\tv_min_u32 v40, v40, v41\n\tv_min3_u32 v40, v40, v42, v43\n\ts_and_b32 s63, s61, 0x7f\n\tv_readfirstlane_b32 s64, v40\n\t" \
    "s_mul_i32 s63, s63, 0x1010101\n\ts_add_u32 s64, s64, 7\n\ts_cmp_ge_u32 s64, %[k1]\n\ts_cbranch_scc0 .LFe%=\n\ts_cmp_gt_u32 s61, 0x80ff\n\t" \
    "s_cbranch_scc1 .LRe%=\n\ts_add_u32 s64, s63, 0x90801000\n\tv_and_b32 v112, s68, v80\n\tv_and_b32 v116, s68, v84\n\tv_and_b32 v120, s68, v88\n\t" \
    "v_and_b32 v113, s68, v81\n\tv_and_b32 v117, s68, v85\n\tv_and_b32 v121, s68, v89\n\tv_and_b32 v114, s68, v82\n\tv_and_b32 v118, s68, v86\n\t" \
    "v_and_b32 v122, s68, v90\n\tv_and_b32 v115, s68, v83\n\tv_and_b32 v119, s68, v87\n\tv_and_b32 v123, s68, v91\n\t" \
    "v_xad_u32 v80, v80, v112, s63\n\tv_xad_u32 v81, v81, v113, s63\n\tv_xad_u32 v82, v82, v114, s63\n\tv_xad_u32 v83, v83, v115, s63\n\t" \
    "ds_write_b128 v56, v[80:83] offset:0\n\tv_xad_u32 v84, v84, v116, s63\n\tv_xad_u32 v85, v85, v117, s63\n\tv_xad_u32 v86, v86, v118, s63\n\t" \
    "v_xad_u32 v87, v87, v119, s63\n\tds_write_b128 v56, v[84:87] offset:1024\n\tv_xad_u32 v88, v88, v120, s63\n\tv_xad_u32 v89, v89, v121, s63\n\t" \
    "v_xad_u32 v90, v90, v122, s63\n\tv_xad_u32 v91, v91, v123, s63\n\tds_write_b128 v56, v[88:91] offset:2048\n\tv_lshrrev_b32 v120, 5, v120\n\t" \
    "v_lshrrev_b32 v121, 5, v121\n\tv_lshrrev_b32 v122, 5, v122\n\tv_lshrrev_b32 v123, 5, v123\n\tv_lshrrev_b32 v112, 5, v112\n\t" \
    "v_lshrrev_b32 v113, 5, v113\n\tv_lshrrev_b32 v114, 5, v114\n\tv_lshrrev_b32 v115, 5, v115\n\tv_perm_b32 v120, s64, s64, v120\n\t" \
    "v_perm_b32 v121, s64, s64, v121\n\tv_perm_b32 v122, s64, s64, v122\n\tv_perm_b32 v123, s64, s64, v123\n\tv_lshrrev_b32 v116, 3, v116\n\t" \
    "v_lshrrev_b32 v117, 3, v117\n\tv_lshrrev_b32 v118, 3, v118\n\tv_lshrrev_b32 v119, 3, v119\n\tv_add3_u32 v124, v112, v116, v120\n\t" \
    "v_add3_u32 v125, v113, v117, v121\n\tv_add3_u32 v126, v114, v118, v122\n\tv_add3_u32 v127, v115, v119, v123\n\t" \
    "ds_write_b128 v56, v[124:127] offset:3072\n\tv_bfe_i32 v44, v92, 13, 3\n\tv_lshl_add_u32 v45, v92, 3, 5\n\tv_and_or_b32 v46, v92, v62, %[lc]\n\t" \
    "v_lshlrev_b32 v44, v45, v44\n\tds_add_u32 v46, v44\n\ts_cmp_gt_u32 s61, 0x40ff\n\ts_cbranch_scc1 .LXe%=\n\t" \
    ".LXBe%=:\n\tds_write_b32 v63, v47\n\ts_add_u32 %[k1], %[k1], 1\n\ts_branch .Lend%=\n\t" \
      \
    ".LWa%=:\n\ts_waitcnt vmcnt(5) lgkmcnt(0)\n\ts_branch .LWBa%=\n\t" \
    ".LIa%=:\n\tv_readlane_b32 s62, v60, s63\n\ts_lshl_b32 s65, s63, 12\n\ts_add_u32 s65, s65, %[sr]\n\ts_cmp_gt_u32 s62, 0x80ff\n\t" \
    "s_cselect_b32 s65, s65, %[sa]\n\ts_sub_u32 s64, s62, 0x4100\n\ts_cmp_lt_u32 s64, 0x4000\n\ts_cselect_b32 s64, 0x80, 0\n\t" \
    "s_add_u32 s64, s64, s65\n\tbuffer_load_dwordx4 v[96:99], %[vq], %[ra], s65 offen offset:0\n\t" \
    "buffer_load_dwordx4 v[100:103], %[vq], %[ra], s65 offen offset:1024\n\tbuffer_load_dwordx4 v[104:107], %[vq], %[ra], s65 offen offset:2048\n\t" \
    "buffer_load_ushort v108, v61, %[ra], s65 offen offset:3072\n\tbuffer_load_ushort v109, v61, %[ra], s64 offen offset:3072\n\ts_branch .LIBa%=\n\t" \
    ".LFa%=:\n\ts_sleep 2\n\tds_read_b128 v[40:43], %[vs] offset:16\n\ts_sub_u32 %[sp], %[sp], 1\n\ts_cbranch_scc1 .Lend%=\n\t" \
    "s_waitcnt lgkmcnt(0)\n\ts_branch .LFCa%=\n\t" \
    ".LXa%=:\n\tv_bfe_i32 v44, v77, 13, 3\n\tv_lshl_add_u32 v45, v77, 3, 5\n\tv_and_or_b32 v46, v77, v62, %[lc]\n\tv_lshlrev_b32 v44, v45, v44\n\t" \
    "ds_add_u32 v46, v44\n\ts_branch .LXBa%=\n\t" \
    ".LRa%=:\n\ts_mov_b32 m0, %[lc]\n\ts_lshl_b32 s63, %[k1], 12\n\ts_add_u32 s63, s63, %[sr]\n\ts_sub_u32 s63, s63, 0x1000\n\t" \
    "buffer_load_dwordx4 %[vq], %[ra], s63 offen offset:0 lds\n\tbuffer_load_dwordx4 %[vq], %[ra], s63 offen offset:1024 lds\n\t" \
    "buffer_load_dwordx4 %[vq], %[ra], s63 offen offset:2048 lds\n\tbuffer_load_dwordx4 %[vq], %[ra], s63 offen offset:3072 lds\n\t" \
    "s_waitcnt vmcnt(0)\n\ts_branch .LXBa%=\n\t" \
    ".LKx%=:\n\tglobal_load_dwordx2 v[48:49], %[va], %[fp] sc0 sc1\n\ts_branch .LKBx%=\n\t" \
    ".LKy%=:\n\tglobal_load_dwordx2 v[48:49], %[va], %[fp] sc0 sc1\n\ts_branch .LKBy%=\n\t" \
    ".LWb%=:\n\ts_waitcnt vmcnt(5) lgkmcnt(0)\n\ts_cmp_eq_u32 %[ck], 1\n\ts_cbranch_scc1 .LKy%=\n\t" \
    ".LKBy%=:\n\ts_branch .LWBb%=\n\t" \
    ".LIb%=:\n\tv_readlane_b32 s60, v60, s63\n\ts_lshl_b32 s65, s63, 12\n\ts_add_u32 s65, s65, %[sr]\n\ts_cmp_gt_u32 s60, 0x80ff\n\t" \
    "s_cselect_b32 s65, s65, %[sa]\n\ts_sub_u32 s64, s60, 0x4100\n\ts_cmp_lt_u32 s64, 0x4000\n\ts_cselect_b32 s64, 0x80, 0\n\t" \
    "s_add_u32 s64, s64, s65\n\tbuffer_load_dwordx4 v[64:67], %[vq], %[ra], s65 offen offset:0\n\t" \
    "buffer_load_dwordx4 v[68:71], %[vq], %[ra], s65 offen offset:1024\n\tbuffer_load_dwordx4 v[72:75], %[vq], %[ra], s65 offen offset:2048\n\t" \
    "buffer_load_ushort v76, v61, %[ra], s65 offen offset:3072\n\tbuffer_load_ushort v77, v61, %[ra], s64 offen offset:3072\n\ts_branch .LIBb%=\n\t" \
    ".LFb%=:\n\ts_sleep 2\n\tds_read_b128 v[40:43], %[vs] offset:16\n\ts_sub_u32 %[sp], %[sp], 1\n\ts_cbranch_scc1 .Lend%=\n\t" \
    "s_waitcnt lgkmcnt(0)\n\ts_branch .LFCb%=\n\t" \
    ".LXb%=:\n\tv_bfe_i32 v44, v93, 13, 3\n\tv_lshl_add_u32 v45, v93, 3, 5\n\tv_and_or_b32 v46, v93, v62, %[lc]\n\tv_lshlrev_b32 v44, v45, v44\n\t" \
    "ds_add_u32 v46, v44\n\ts_branch .LXBb%=\n\t" \
    ".LRb%=:\n\ts_mov_b32 m0, %[lc]\n\ts_lshl_b32 s63, %[k1], 12\n\ts_add_u32 s63, s63, %[sr]\n\ts_sub_u32 s63, s63, 0x1000\n\t" \
    "buffer_load_dwordx4 %[vq], %[ra], s63 offen offset:0 lds\n\tbuffer_load_dwordx4 %[vq], %[ra], s63 offen offset:1024 lds\n\t" \
    "buffer_load_dwordx4 %[vq], %[ra], s63 offen offset:2048 lds\n\tbuffer_load_dwordx4 %[vq], %[ra], s63 offen offset:3072 lds\n\t" \
    "s_waitcnt vmcnt(0)\n\ts_branch .LXBb%=\n\t" \
    ".LWc%=:\n\ts_waitcnt vmcnt(5) lgkmcnt(0)\n\ts_branch .LWBc%=\n\t" \
    ".LIc%=:\n\tv_readlane_b32 s61, v60, s63\n\ts_lshl_b32 s65, s63, 12\n\ts_add_u32 s65, s65, %[sr]\n\ts_cmp_gt_u32 s61, 0x80ff\n\t" \
    "s_cselect_b32 s65, s65, %[sa]\n\ts_sub_u32 s64, s61, 0x4100\n\ts_cmp_lt_u32 s64, 0x4000\n\ts_cselect_b32 s64, 0x80, 0\n\t" \
    "s_add_u32 s64, s64, s65\n\tbuffer_load_dwordx4 v[80:83], %[vq], %[ra], s65 offen offset:0\n\t" \
    "buffer_load_dwordx4 v[84:87], %[vq], %[ra], s65 offen offset:1024\n\tbuffer_load_dwordx4 v[88:91], %[vq], %[ra], s65 offen offset:2048\n\t" \
    "buffer_load_ushort v92, v61, %[ra], s65 offen offset:3072\n\tbuffer_load_ushort v93, v61, %[ra], s64 offen offset:3072\n\ts_branch .LIBc%=\n\t" \
    ".LFc%=:\n\ts_sleep 2\n\tds_read_b128 v[40:43], %[vs] offset:16\n\ts_sub_u32 %[sp], %[sp], 1\n\ts_cbranch_scc1 .Lend%=\n\t" \
    "s_waitcnt lgkmcnt(0)\n\ts_branch .LFCc%=\n\t" \
    ".LXc%=:\n\tv_bfe_i32 v44, v109, 13, 3\n\tv_lshl_add_u32 v45, v109, 3, 5\n\tv_and_or_b32 v46, v109, v62, %[lc]\n\tv_lshlrev_b32 v44, v45, v44\n\t" \
    "ds_add_u32 v46, v44\n\ts_branch .LXBc%=\n\t" \
    ".LRc%=:\n\ts_mov_b32 m0, %[lc]\n\ts_lshl_b32 s63, %[k1], 12\n\ts_add_u32 s63, s63, %[sr]\n\ts_sub_u32 s63, s63, 0x1000\n\t" \
    "buffer_load_dwordx4 %[vq], %[ra], s63 offen offset:0 lds\n\tbuffer_load_dwordx4 %[vq], %[ra], s63 offen offset:1024 lds\n\t" \
    "buffer_load_dwordx4 %[vq], %[ra], s63 offen offset:2048 lds\n\tbuffer_load_dwordx4 %[vq], %[ra], s63 offen offset:3072 lds\n\t" \
    "s_waitcnt vmcnt(0)\n\ts_branch .LXBc%=\n\t" \
    ".LWd%=:\n\ts_waitcnt vmcnt(5) lgkmcnt(0)\n\ts_branch .LWBd%=\n\t" \
    ".LFd%=:\n\ts_sleep 2\n\tds_read_b128 v[40:43], %[vs] offset:16\n\ts_sub_u32 %[sp], %[sp], 1\n\ts_cbranch_scc1 .Lend%=\n\t" \
    "s_waitcnt lgkmcnt(0)\n\ts_branch .LFCd%=\n\t" \
    ".LXd%=:\n\tv_bfe_i32 v44, v77, 13, 3\n\tv_lshl_add_u32 v45, v77, 3, 5\n\tv_and_or_b32 v46, v77, v62, %[lc]\n\tv_lshlrev_b32 v44, v45, v44\n\t" \
    "ds_add_u32 v46, v44\n\ts_branch .LXBd%=\n\t" \
    ".LRd%=:\n\ts_mov_b32 m0, %[lc]\n\ts_lshl_b32 s63, %[k1], 12\n\ts_add_u32 s63, s63, %[sr]\n\ts_sub_u32 s63, s63, 0x1000\n\t" \
    "buffer_load_dwordx4 %[vq], %[ra], s63 offen offset:0 lds\n\tbuffer_load_dwordx4 %[vq], %[ra], s63 offen offset:1024 lds\n\t" \
    "buffer_load_dwordx4 %[vq], %[ra], s63 offen offset:2048 lds\n\tbuffer_load_dwordx4 %[vq], %[ra], s63 offen offset:3072 lds\n\t" \
    "s_waitcnt vmcnt(0)\n\ts_branch .LXBd%=\n\t" \
    ".LFe%=:\n\ts_sleep 2\n\tds_read_b128 v[40:43], %[vs] offset:16\n\ts_sub_u32 %[sp], %[sp], 1\n\ts_cbranch_scc1 .Lend%=\n\t" \
    "s_waitcnt lgkmcnt(0)\n\ts_branch .LFCe%=\n\t" \
    ".LXe%=:\n\tv_bfe_i32 v44, v93, 13, 3\n\tv_lshl_add_u32 v45, v93, 3, 5\n\tv_and_or_b32 v46, v93, v62, %[lc]\n\tv_lshlrev_b32 v44, v45, v44\n\t" \
    "ds_add_u32 v46, v44\n\ts_branch .LXBe%=\n\t" \
    ".LRe%=:\n\ts_mov_b32 m0, %[lc]\n\ts_lshl_b32 s63, %[k1], 12\n\ts_add_u32 s63, s63, %[sr]\n\ts_sub_u32 s63, s63, 0x1000\n\t" \
    "buffer_load_dwordx4 %[vq], %[ra], s63 offen offset:0 lds\n\tbuffer_load_dwordx4 %[vq], %[ra], s63 offen offset:1024 lds\n\t" \
    "buffer_load_dwordx4 %[vq], %[ra], s63 offen offset:2048 lds\n\tbuffer_load_dwordx4 %[vq], %[ra], s63 offen offset:3072 lds\n\t" \
    "s_waitcnt vmcnt(0)\n\ts_branch .LXBe%=\n\t" \
    ".LC%=:\n\ts_sub_u32 s63, %[ck], 1\n\ts_cmp_gt_u32 %[ck], 1\n\ts_cselect_b32 %[ck], s63, %[ck]\n\ts_cbranch_scc1 .LCB%=\n\t" \
    "v_cmp_le_u32 vcc, %[call], v48\n\ts_cmp_eq_u64 vcc, -1\n\ts_cbranch_scc0 .Lend%=\n\tv_lshrrev_b32 v44, s67, v49\n\tv_and_b32 v60, 0x7f, v49\n\t" \
    "v_and_b32 v44, 0xff, v44\n\tv_lshl_or_b32 v60, v44, 8, v60\n\ts_mov_b32 %[ck], 0\n\ts_branch .LCB%=\n\t" \
    ".Lend%=:\n\ts_waitcnt vmcnt(0) lgkmcnt(0)\n\ts_mov_b32 m0, s66\n\t" \
    ""

__global__ void __launch_bounds__(512, 2) ag_gemm_kernel(Params p) {
    __shared__ __attribute__((aligned(4096))) unsigned char smem[LDS_BYTES];
    const unsigned lds0 = (unsigned)(size_t)(LAS unsigned char*)smem, sync = lds0 + SYNC_OFF;
    const unsigned lane = threadIdx.x & 63, wid = rfl(threadIdx.x >> 6), bid = blockIdx.x;
    const unsigned cks = ((bid >> 3) + 31) & 31, crb = (bid >> 1) & 3, chf = bid & 1, q2 = wid >> 2, sl = wid & 3;
    const __amdgpu_buffer_rsrc_t rx = __builtin_amdgcn_make_buffer_rsrc((void*)p.x, 0, (unsigned)SHARD, 0x00020000);
    const unsigned xso = (crb * BM + chf * 64) * K_ + cks * BK;
    u32x4 smp, xv; unsigned cv; u32x2 pw;
    {
        const unsigned xo = (32 * q2 + 2 * sl + (lane >> 5)) * K_ + 4 * (lane & 3), c = (lane >> 2) & 7;
        asm volatile("buffer_load_dwordx4 v[36:39], %4, %7, %8 offen\n\tbuffer_load_dword v40, %5, %7, %8 offen\n\tbuffer_load_dword v41, %6, %7, %8 offen\n\t"
                     "buffer_load_dword v42, %5, %7, %9 offen\n\tbuffer_load_dword v43, %6, %7, %9 offen\n\tglobal_load_dword v32, %10, %11 sc1\n\tglobal_load_dwordx2 v[34:35], %12, %13"
                     : "=&{v[36:39]}"(smp), "=&{v[40:43]}"(xv), "=&{v32}"(cv), "=&{v[34:35]}"(pw)
                     : "v"((lane >> 3) * 8 * K_ + (lane & 7) * 16), "v"(xo + ((c ^ sl) << 4)), "v"(xo + 8 * K_ + ((c ^ (4 + sl)) << 4)), "s"(rx), "s"(xso), "s"(xso + 16 * K_),
                       "v"((unsigned)WS_CNT + bid * 64), "s"(p.myws), "v"((lane & 7) * 8), "s"(__builtin_amdgcn_kernarg_segment_ptr()) : "memory");
    }
    const unsigned rank = p.rank & 7, rb = (bid >> 3) & 3, cb = bid >> 5;
    unsigned char* const myws = p.myws;
    GAS unsigned* const tmo = (GAS unsigned*)(myws + WS_TMO);
    const unsigned xcc = __builtin_amdgcn_s_getreg((3 << 11) | 20) & 15u;
    const unsigned long long tstart = __builtin_amdgcn_s_memrealtime();
    const unsigned wcb = bid & 7, wks = (cks + 16) & 31;
    const __amdgpu_buffer_rsrc_t rw = __builtin_amdgcn_make_buffer_rsrc((void*)p.w, 0, (unsigned)(K_ * NL), 0x00020000);
    const unsigned wso = (wks * BK + (wid & 1) * 64) * NL + wcb * BN + ((wid >> 1) & 1) * 128, wld = lds0 + WT_OFF + sl * 8192, vw = (lane >> 3) * NL + (lane & 7) * 16;
    unsigned call, meta;
    {
        const bool inval = bid < 8 && wid == 7;
        if (inval) asm volatile("buffer_inv sc0 sc1" ::: "memory");
        const unsigned tid = threadIdx.x, hu = (crb * 2 + chf) * 32 + cks, qu = (crb * 4 + chf * 2) * 32 + cks;
        const unsigned role = wid < 4 ? (wks < WEARLY ? 1u : bid == 8 && wid == 1 ? 4u : 0u) : wid == 4 + (cb & 3) ? 2u : 0u;
        asm volatile(ENCODE_ASM
            : [call] "=&s"(call), [meta] "=&s"(meta)
            : "{v[36:39]}"(smp), "{v[40:43]}"(xv), "{v32}"(cv), "{v[34:35]}"(pw),
              [rw] "s"(rw), [wso] "s"(wso), [wld] "s"(wld), [role] "s"(rfl(role)), [xc] "s"(xcc & 7), [rp] "s"(myws + WS_TMO + 4096),
              [st] "s"(myws + WS_DATA + qu * QU), [fo] "s"((unsigned)WS_FLAG + (rank * 256 + hu) * 128), [lst] "s"(lds0 + LIST_OFF + 256 * q2),
              [via] "v"(lds0 + (tid < 156 ? LIST_OFF : 0u) + 4 * tid), [viv] "v"(tid < 128 ? lane * 0x80008u + 0x40000u : 0u),
              [vw] "v"(vw), [ven] "v"(lds0 + ENC_OFF + q2 * 3072 + sl * 256 + lane * 4), [vps] "v"(sl * 256 + lane * 4), [vca] "v"(sync + 36 + 4 * q2),
              [vq] "v"(lane * 16), [vrd] "v"(lds0 + ENC_OFF + lane * 16)
            : "memory", "scc", "vcc", S40_59, V44_107);
        if (inval && lane == 0) __hip_atomic_store((GAS unsigned*)(myws + WS_TMO + 128 + xcc * 128), call, __ATOMIC_RELAXED, __HIP_MEMORY_SCOPE_AGENT);
        if (role == 2 && (((meta >> 8) & 0xff) > EXC_CAP || (meta >> 16) > EXC_CAP)) {
            unsigned char* dst = myws + WS_DATA + RAW_OFF + (call & 1) * (unsigned)SHARD + qu * 4096;
#pragma nounroll
            for (unsigned i = 0; i < 8; ++i) {
                const u32x4 d = __builtin_amdgcn_raw_buffer_load_b128(rx, (lane >> 3) * K_ + (((lane & 7) ^ (4 * (i & 1) + (lane >> 4))) << 4), xso + 8 * i * K_, 0);
                asm volatile("global_store_dwordx4 %1, %0, %2 sc0 sc1\n\ts_nop 1" :: "v"(d), "v"(lane * 16 + (i & 3) * 1024 + (i >> 2) * 32 * 4096), "s"(dst) : "memory");
            }
            asm volatile("s_waitcnt vmcnt(0)" ::: "memory");
            const u32x2 fv = {call, meta};
#pragma nounroll
            for (int i = 0; i < NR; ++i) {
                unsigned char* fp = p.ws[i] + WS_FLAG + (((call & 1) * NR + rank) * 256 + hu) * 128;
                asm volatile("s_mov_b64 exec, 1\n\tglobal_store_dwordx2 %0, %1, %2 sc0 sc1\n\ts_mov_b64 exec, -1" :: "v"(0u), "v"(fv), "s"(fp) : "memory");
            }
        }
        if (wid == 0 && lane == 0) __hip_atomic_store((GAS unsigned*)(myws + WS_CNT + bid * 64), call, __ATOMIC_RELAXED, __HIP_MEMORY_SCOPE_AGENT);
    }
    typedef __attribute__((address_space(4))) const float cfloat;
    float scale = __builtin_bit_cast(float, rfl(__builtin_bit_cast(unsigned, *(cfloat*)p.sx * *(cfloat*)p.sw)));
    asm volatile("" : "+s"(scale));
    const __amdgpu_buffer_rsrc_t ro = __builtin_amdgcn_make_buffer_rsrc((void*)p.out, 0, (unsigned)(M_ * NL * 4), 0x00020000);
    unsigned osoff = (rb * BM * NL + cb * BN + sl * 64) * 4;

    if (wid < 4) {
        asm volatile(ZERO_ASM ::: A0_127);
        unsigned char* const wt = myws + WS_WT + (size_t)(call % NWT) * (K_ * NL);
        const unsigned ta = wld + (16 * (lane >> 4) + ((lane & 15) >> 1)) * 128 + (lane & 1) * 8, wfo = (unsigned)WS_WFLAG + (wcb * 32 + wks) * 128;
        unsigned char* const wdst = wt + (((wks * 8 + wcb) * 4 + (wid & 2)) * 2 + (wid & 1)) * 4096;
        if (wks < WEARLY) {
            unsigned t0;
            asm volatile(RETILE_ASM RETILE_FLAG_ASM : [t0] "=&s"(t0)
                : [ta] "v"(ta), [v0] "v"(lane * 16), [v1] "v"(lane * 16 + 8192), [dst] "s"(wdst), [ca] "v"(sync + 64), [vf] "v"(wfo), [fp] "s"(myws), [call] "v"(call) : "memory", "scc", V28_127);
        } else {
            while (rfl(lds_ld(sync + 72)) == 0u && __builtin_amdgcn_s_memrealtime() - tstart < (cb < 2 ? 800u : 250u)) __builtin_amdgcn_s_sleep(2);
            asm volatile(WDMA_ASM :: [rw] "s"(rw), [wso] "s"(wso), [wld] "s"(wld), [vw] "v"(vw) : "memory", "scc", "s52", "s59", "v64", "v65", "v66");
        }
        if (wid == 1) {
            unsigned m;
            const bool ok = poll_flags(myws, (unsigned)WS_WFLAG + cb * 4096 + min(lane, LOOK + 1) * 128, tmo, call, m);
            if (!ok) lds_st(sync + 32, 1u);
            lds_st(sync + 68, ok ? 1u : 2u);
        }
        const bool wok = wait_lds(sync + 68, 1) == 1;
        if (wks >= WEARLY) asm volatile(RETILE_ASM "s_nop 1" :: [ta] "v"(ta), [v0] "v"(lane * 16), [v1] "v"(lane * 16 + 8192), [dst] "s"(wdst) : "memory", V28_127);
        unsigned a0 = lds0 + (lane & 15) * 128 + (((lane >> 4) ^ ((lane & 15) >> 1)) << 4), a1 = a0 ^ 64;
        const __amdgpu_buffer_rsrc_t rwt = __builtin_amdgcn_make_buffer_rsrc((void*)(wt + (cb * 4 + wid) * 8192), 0, wok ? (NK - 1) * (BK * NL) + 8192 : 0, 0x00020000);
        unsigned kk = 0, slot = 0, dl, spin = 0, sw0 = 0, sw1 = 4096;
        asm volatile(CONSUMER_ASM
            : [a0] "+&v"(a0), [a1] "+&v"(a1), [kk] "+&s"(kk), [sl] "+&s"(slot), [dl] "=&s"(dl), [sp] "+&s"(spin), [sw0] "+&s"(sw0), [sw1] "+&s"(sw1)
            : [vs] "v"(sync + 4 * (lane & 3)), [vq] "v"(lane * 16), [em] "s"(1ull << wid), [wf] "s"(rfl(wks >= WEARLY ? wfo : 0u)), [my] "s"(myws), [call] "s"(call),
              [db] "s"(lds0 + wid * 32768), [rw] "s"(rwt), [dp] "s"(STAGE), [dn] "s"(0u - (NSTAGE - 1) * STAGE)
            : "memory", "scc", "vcc", V6_27, V28_127, A0_127);
        unsigned ln;
        asm volatile("v_mbcnt_lo_u32_b32 %0, -1, 0\n\tv_mbcnt_hi_u32_b32 %0, -1, %0" : "=v"(ln));
        osoff += (rfl(sw1) & 7) * (MS * NL * 4);
        const float is = dl != 0u ? __builtin_nanf("") : __builtin_amdgcn_rcpf(scale), nsl = scale * -1.44269504088896340736f;
        const unsigned ovoff = (ln >> 3) * (NL * 4) + (ln & 7) * 16;
        const unsigned wo = a0 - lds0;
        const LAS unsigned char* rd = (const LAS unsigned char*)smem + wid * 32768 + (ln >> 3) * 128 + (((ln & 7) ^ (ln >> 3)) << 4);
#define DUMP(SB) do { asm volatile("ds_write_b128 %0, a[%c2:%c2+3] offset:%c3\n\tds_write_b128 %1, a[%c2+4:%c2+7] offset:%c3\n\t" \
                                   "ds_write_b128 %0, a[%c2+8:%c2+11] offset:%c3+2048\n\tds_write_b128 %1, a[%c2+12:%c2+15] offset:%c3+2048" \
                                   :: "v"(a0), "v"(a1), "n"(16 * (SB)), "n"((SB) * 4096) : "memory"); \
                      lds_st(sync + 48 + 4 * wid, (SB) + 1); } while (0)
#define QUAD(SB, G) do { float q0, q1, q2_, q3; \
            asm volatile("v_accvgpr_read_b32 %0, a[%c4]\n\tv_accvgpr_read_b32 %1, a[%c4+1]\n\tv_accvgpr_read_b32 %2, a[%c4+2]\n\tv_accvgpr_read_b32 %3, a[%c4+3]" \
                         : "=v"(q0), "=v"(q1), "=v"(q2_), "=v"(q3) : "n"(16 * (SB) + 4 * (G))); q[G] = (f32x4){q0, q1, q2_, q3}; } while (0)
#define OWN(SB) do { f32x4 q[4]; QUAD(SB, 0); QUAD(SB, 1); QUAD(SB, 2); QUAD(SB, 3); \
            _Pragma("unroll") for (int g = 0; g < 4; ++g) *(LAS f32x4*)((LAS unsigned char*)smem + (wo ^ (64 * (g & 1))) + (SB) * 4096 + (g >> 1) * 2048) = act4(q[g], nsl, is); \
            _Pragma("unroll") for (int i = 0; i < 4; ++i) q[i] = *(const LAS f32x4*)(rd + (SB) * 4096 + i * 1024); \
            _Pragma("unroll") for (int i = 0; i < 4; ++i) st_out(q[i], ro, ovoff, osoff + (((SB) >> 1) * 32 + 8 * i) * (NL * 4) + ((SB) & 1) * 128); } while (0)
        OWN(4); DUMP(2); OWN(5); DUMP(3); OWN(6); OWN(7);
    } else {
        const unsigned hq = sl;
        const unsigned lead = (0x01010101u << cb) & (rb == 0 ? 0x0fffffffu : rb == 1 ? 0x0000ffffu : 0u);
        unsigned rot = (xcc - bid) & 7, level = 0, src;
        bool dead = false, inv_ok = false;
#pragma nounroll
        for (;;) {
            src = (p.srcmap >> (4 * ((bid + rot) & 7))) & 7;
            lds_st(sync + 44, src);
            const __amdgpu_buffer_rsrc_t ra = __builtin_amdgcn_make_buffer_rsrc((void*)(p.ws[src] + WS_DATA), 0, DATA_BYTES, 0x00020000);
            const unsigned xf0 = WS_FLAG + (((call & 1) * NR + src) * 256 + (rb * 2 + (hq >> 1)) * 32) * 128;
            const unsigned v0 = lane == 63 ? (unsigned)WS_TMO + 4096u : lane == 62 ? (unsigned)WS_TMO + 128u + xcc * 128u : xf0 + (lane & 1) * 128;
            const unsigned vb = xf0 + min(lane, LOOK + 1) * 128;
            const unsigned vall = (lane < 32 ? xf0 : (unsigned)WS_WFLAG + cb * 4096) + (lane & 31) * 128;
            unsigned fmeta = v0;
            if (level == 1) {
                const unsigned long long t0 = __builtin_amdgcn_s_memrealtime();
                bool again = false;
#pragma nounroll
                for (unsigned sp = 0;; ++sp) {
                    u32x2 f;
                    asm volatile("global_load_dwordx2 %0, %1, %2 sc0 sc1\n\ts_waitcnt vmcnt(0)" : "=&v"(f) : "v"(v0), "s"(myws) : "memory");
                    fmeta = f.y;
                    const unsigned pub = __builtin_amdgcn_readlane(f.x, 63);
                    const unsigned long long up = __ballot(f.x >= call);
                    if ((up >> 62) & 1) inv_ok = true;
                    if ((pub >> 3) >= call) {
                        if ((pub & 7) != rot) { rot = pub & 7; again = true; break; }
                        if (inv_ok && (up & 3) == 3) break;
                    }
                    if ((sp & 15) == 15) {
                        if (!inv_ok) { asm volatile("buffer_inv sc0 sc1\n\ts_waitcnt vmcnt(0)" ::: "memory"); inv_ok = true; }
                        if (give_up(t0, tmo, call)) { dead = true; break; }
                    }
                    __builtin_amdgcn_s_sleep(2);
                }
                if (dead) break;
                if (again) continue;
            } else if (level >= 2 && !poll_flags(myws, level == 2 ? vb : vall, tmo, call, fmeta)) { dead = true; break; }
            const unsigned k0 = level < 3 ? 0u : LOOK;
            unsigned sa = (call & 1) * PACK_PAR + ((rb * 4 + hq) * 32 + k0) * QU, spin = 1u << 25, k1 = k0, ck = k0 ? 0u : LOOK / 3, lc = lds0 + hq * 4096 + k0 % NSTAGE * STAGE;
            asm volatile(HELPER_ASM
                : [k1] "+&s"(k1), [lc] "+&s"(lc), [ck] "+&s"(ck), [sa] "+&s"(sa), [sp] "+&s"(spin)
                : [ra] "s"(ra), [sr] "s"(RAW_OFF + (call & 1) * (unsigned)SHARD + (rb * 4 + hq) * 32 * 4096), [call] "s"(call), [fp] "s"(myws), [va] "v"(vall), [vb] "v"(vb),
                  [q] "s"(level), [l0] "s"(lds0 + hq * 4096), [ld] "s"(lead), [hq] "s"(hq), [pub] "s"(call * 8 + rot), [vq] "v"(lane * 16), [vs] "v"(sync), [fm] "v"(fmeta)
                : "memory", "scc", "vcc", S60_69, V40_127);
            k1 = rfl(k1);
            if (rfl(ck) == 0) { dead = k1 != NK + 1; break; }
            if (level == 0 && k1 == 0) level = 1;
            else if (level < 2 && k1 == 1) level = 2;
            else if (k1 == LOOK + 1) level = 3;
            else { dead = true; break; }
        }
        if (dead) lds_st(sync + 32, 1u);
        lds_st(sync + 4 * hq, NK);
        if (rfl(__hip_atomic_load(tmo, __ATOMIC_RELAXED, __HIP_MEMORY_SCOPE_AGENT)) == call) lds_st(sync + 32, 1u);
        osoff += src * (MS * NL * 4);
        const float nsl = scale * -1.44269504088896340736f;
        const unsigned ovoff = (lane >> 3) * (NL * 4) + (lane & 7) * 16;
        const LAS unsigned char* rd = (const LAS unsigned char*)smem + hq * 32768 + (lane >> 3) * 128 + (((lane & 7) ^ (lane >> 3)) << 4);
        wait_lds(sync + 16 + 4 * hq, NK);
        unsigned have = 0, ab = 0;
#pragma nounroll
        for (unsigned sp = 0; sp < (1u << 26) && have == 0; ++sp) { const unsigned v = lds_ld(lane == 1 ? sync + 32 : sync + 48 + 4 * hq); have = rfl(v); ab = __builtin_amdgcn_readlane(v, 1); }
        const float is = ab != 0u ? __builtin_nanf("") : __builtin_amdgcn_rcpf(scale);
#pragma unroll
        for (int sb = 0; sb < 4; ++sb) {
            if (have <= (unsigned)sb) have = wait_lds(sync + 48 + 4 * hq, sb + 1);
            f32x4 v[4];
#pragma unroll
            for (int i = 0; i < 4; ++i) v[i] = *(const LAS f32x4*)(rd + sb * 4096 + i * 1024);
#pragma unroll
            for (int i = 0; i < 4; ++i) st_out(act4(v[i], nsl, is), ro, ovoff, osoff + ((sb >> 1) * 32 + 8 * i) * (NL * 4) + (sb & 1) * 128);
        }
    }
}

unsigned source_map(int rank) {
    unsigned near = ((0xb2u >> rank) & 1) ? 0xb2u : 0x4du;
    int dev = 0; char bus[64] = {0}, path[160], buf[512] = {0};
    if (hipGetDevice(&dev) == hipSuccess && hipDeviceGetPCIBusId(bus, sizeof bus, dev) == hipSuccess) {
        for (char* c = bus; *c; ++c) if (*c >= 'A' && *c <= 'F') *c += 'a' - 'A';
        snprintf(path, sizeof path, "/sys/bus/pci/devices/%s/xgmi_port_num", bus);
        if (FILE* f = fopen(path, "r")) {
            const size_t n = fread(buf, 1, sizeof buf - 1, f); buf[n] = 0; fclose(f);
            unsigned m = 1u << rank, seen = m, a, b, c, d; int used = 0; bool ok = true;
            for (const char* q = buf; sscanf(q, " %x:%x -> %x:%x%n", &a, &b, &c, &d, &used) == 4; q += used) {
                if (a != (unsigned)rank + 1 || c < 1 || c > NR) { ok = false; break; }
                seen |= 1u << (c - 1);
                if (b <= 3) m |= 1u << (c - 1);
            }
            if (ok && seen == 0xffu && __builtin_popcount(m) == 4) near = m;
        }
    }
    (void)hipGetLastError();
    unsigned map = 0; int lo = 0, hi = 4;
    for (int r = 0; r < NR; ++r) map |= (unsigned)r << (4 * (((near >> r) & 1) ? lo++ : hi++));
    return map;
}

}

extern "C" void kernel_launch(int rank, int nranks, ncclComm_t comm, hipStream_t stream,
                              void* const* d_in, const int* in_sizes, int n_in,
                              void* d_out, int out_size, const int* in_dtypes, int out_dtype,
                              void* const* d_out_all, void* const* d_ws_all, size_t ws_bytes) {
    (void)comm; (void)in_dtypes; (void)out_dtype; (void)d_out_all;
    if (nranks != NR || n_in != 4 || (unsigned)rank >= (unsigned)NR || in_sizes[0] != MS * K_ || in_sizes[1] != K_ * NL || out_size != M_ * NL || ws_bytes < WS_NEED) {
        fprintf(stderr, "ag_gemm: unexpected shapes\n");
        return;
    }
    Params p;
    for (int i = 0; i < NR; ++i) p.ws[i] = (unsigned char*)d_ws_all[i];
    p.x = (const unsigned char*)d_in[0]; p.w = (const unsigned char*)d_in[1];
    p.sx = (const float*)d_in[2]; p.sw = (const float*)d_in[3]; p.out = (float*)d_out;
    p.myws = p.ws[rank]; p.rank = (unsigned)rank; p.srcmap = source_map(rank);
    hipLaunchKernelGGL(ag_gemm_kernel, dim3(NWG), dim3(512), 0, stream, p);
    const hipError_t e = hipGetLastError();
    if (e != hipSuccess) fprintf(stderr, "ag_gemm launch: %s\n", hipGetErrorString(e));
}
```
